```python
import numpy as np

kernel_name = "gpu_ag_gemm_m4096_k4096_n32768_f32_relu"


ABSMAX_RTOL = 0.001

COMPUTE_DTYPE = "float32"
EPILOGUE = "relu"
IN_QUANT = None
OUT_QUANT = None
IN_DTYPES = {"x": "float32", "w_mat": "float32"}
WORLD_SIZE = 8

def setup_inputs():

    x = np.random.randn(4096, 4096).astype(np.float32)
    w_mat = (np.random.randn(4096, 32768) / np.sqrt(4096)).astype(np.float32)
    return {"x": x, "w_mat": w_mat}

def _epilogue(y):
    return np.maximum(y, 0.0).astype(np.float32)

def reference(x, w_mat):
    y = (x @ w_mat).astype(np.float32)
    return _epilogue(y)
```

```cpp
#include <hip/hip_runtime.h>
#include <rccl/rccl.h>
#include <cstdio>
#include <cstdint>

#define HIP_CHECK(expr) do { hipError_t _e = (expr); if (_e != hipSuccess) { \
    fprintf(stderr, "HIP error %s at %s:%d\n", hipGetErrorString(_e), __FILE__, __LINE__); \
    return; } } while (0)

namespace {

typedef _Float16 half8 __attribute__((ext_vector_type(8)));
typedef _Float16 half2v __attribute__((ext_vector_type(2)));
typedef float f32x4 __attribute__((ext_vector_type(4)));
typedef float f32x2 __attribute__((ext_vector_type(2)));
typedef unsigned u32x4 __attribute__((ext_vector_type(4)));
#define LAS __attribute__((address_space(3)))

constexpr int WORLD = 8;
constexpr int GM = 4096, GK = 4096, GN = 4096;
constexpr int MS = GM / WORLD;
constexpr int NT = GK / 64;
constexpr int HTB = 16384;
constexpr size_t PANEL_B = (size_t)NT * 2 * HTB;
constexpr size_t OPER_B = 16 * PANEL_B;
constexpr size_t WS_CTL = 0;
constexpr size_t WS_A0 = 1u << 20;
constexpr size_t WS_B = WS_A0 + 2 * OPER_B;
constexpr size_t WS_END = WS_B + OPER_B;
constexpr int CW_TICKET0 = 0, CW_TICKET1 = 32, CW_DONE0 = 64, CW_TMO = 96, CW_FLAG = 128;
constexpr unsigned long long DEADLINE_TICKS = 50000000ull;
constexpr int LDS_BYTES = 8 * HTB;
constexpr int GRID = 256;

struct Params {
    const float* x; const float* w; float* out;
    char* ws[WORLD];
    int rank;
};

__device__ __forceinline__ u32x4 cvt8(const f32x4 a, const f32x4 b) {
    half8 h;
    h[0] = (_Float16)a[0]; h[1] = (_Float16)a[1]; h[2] = (_Float16)a[2]; h[3] = (_Float16)a[3];
    h[4] = (_Float16)b[0]; h[5] = (_Float16)b[1]; h[6] = (_Float16)b[2]; h[7] = (_Float16)b[3];
    return __builtin_bit_cast(u32x4, h);
}

__global__ __launch_bounds__(512, 2) void fused_kernel(Params p, int phase) {
    extern __shared__ __attribute__((aligned(16))) unsigned char smem[];
    LAS unsigned char* lds = (LAS unsigned char*)smem;
    const int tid = threadIdx.x, wid = __builtin_amdgcn_readfirstlane(tid >> 6), lane = tid & 63;
    const int rank = p.rank;
    char* wsl = p.ws[rank];
    unsigned* ctl = (unsigned*)(wsl + WS_CTL);

    LAS unsigned* bc = (LAS unsigned*)lds;
    if (tid == 0) {
        unsigned t = __hip_atomic_fetch_add(ctl + (phase == 0 ? CW_TICKET0 : CW_TICKET1), 1u, __ATOMIC_RELAXED, __HIP_MEMORY_SCOPE_AGENT);
        bc[0] = t / GRID + 1u;
    }
    __syncthreads();
    const unsigned call = __builtin_amdgcn_readfirstlane(bc[0]);
    __syncthreads();
    const size_t abuf = WS_A0 + (size_t)(call & 1u) * OPER_B;

    if (phase == 0) {
        const int gw = blockIdx.x * 8 + wid;
        const int fr = lane & 15, fq = lane >> 4;
        for (int i = 0; i < 2; ++i) {
            const int f = gw * 2 + i;
            const int kh = f >> 5, rg = f & 31;
            const float* src = p.x + (size_t)(rg * 16 + fr) * GK + kh * 32 + fq * 8;
            const f32x4 a = *(const f32x4*)src, b = *(const f32x4*)(src + 4);
            const u32x4 v = cvt8(a, b);
            const int grow = rank * MS + rg * 16;
            const int pm = grow >> 8, h = (grow >> 7) & 1, m8 = (grow >> 4) & 7;
            const size_t off = abuf + (size_t)pm * PANEL_B + ((size_t)(kh >> 1) * 2 + h) * HTB + (size_t)(m8 * 2 + (kh & 1)) * 1024 + lane * 16;
#pragma unroll
            for (int d = 0; d < WORLD; ++d) {
                const int dst = (rank + d) & 7;
                *(u32x4*)(p.ws[dst] + off) = v;
            }
        }
        for (int i = 0; i < 4; ++i) {
            const int u = i * 2048 + gw;
            const int kh = u >> 6, ng = u & 63;
            const float* src = p.w + (size_t)(kh * 32 + fq * 8) * GN + ng * 64 + fr * 4;
            f32x4 r[8];
#pragma unroll
            for (int j = 0; j < 8; ++j) r[j] = *(const f32x4*)(src + (size_t)j * GN);
            const int pn = ng >> 2, wc = ng & 3;
#pragma unroll
            for (int c = 0; c < 4; ++c) {
                const f32x4 a = {r[0][c], r[1][c], r[2][c], r[3][c]}, b = {r[4][c], r[5][c], r[6][c], r[7][c]};
                const u32x4 v = cvt8(a, b);
                const int bj = c >> 1, n = c & 1;
                const size_t off = WS_B + (size_t)pn * PANEL_B + ((size_t)(kh >> 1) * 2 + bj) * HTB + (size_t)((wc * 2 + n) * 2 + (kh & 1)) * 1024 + lane * 16;
                *(u32x4*)(wsl + off) = v;
            }
        }
        asm volatile("s_waitcnt vmcnt(0)" ::: "memory");
        __syncthreads();
        if (tid == 0) {
            __builtin_amdgcn_fence(__ATOMIC_RELEASE, "");
            asm volatile("s_waitcnt vmcnt(0)" ::: "memory");
            const unsigned t = __hip_atomic_fetch_add(ctl + CW_DONE0, 1u, __ATOMIC_ACQ_REL, __HIP_MEMORY_SCOPE_AGENT);
            if ((t % GRID) == GRID - 1) {
                __builtin_amdgcn_fence(__ATOMIC_ACQUIRE, "agent");
                __builtin_amdgcn_fence(__ATOMIC_RELEASE, "");
                asm volatile("s_waitcnt vmcnt(0)" ::: "memory");
                for (int d = 1; d < WORLD; ++d) {
                    const int dst = (rank + d) & 7;
                    __hip_atomic_store((unsigned*)(p.ws[dst] + WS_CTL) + CW_FLAG + 32 * rank, call, __ATOMIC_RELAXED, __HIP_MEMORY_SCOPE_SYSTEM);
                }
            }
        }
        return;
    }

    const int xcd = blockIdx.x & 7, idx = blockIdx.x >> 3;
    const int pm = (xcd >> 1) * 4 + (idx & 3), pn = (xcd & 1) * 8 + (idx >> 2);
    const int src_rank = pm >> 1;
    unsigned* tmo = ctl + CW_TMO;
    if (tid == 0 && src_rank != rank) {
        const unsigned* flag = ctl + CW_FLAG + 32 * src_rank;
        const unsigned long long t0 = __builtin_amdgcn_s_memrealtime();
        while (__hip_atomic_load(flag, __ATOMIC_RELAXED, __HIP_MEMORY_SCOPE_SYSTEM) < call) {
            if (__hip_atomic_load(tmo, __ATOMIC_RELAXED, __HIP_MEMORY_SCOPE_AGENT) == call
                || __builtin_amdgcn_s_memrealtime() - t0 > DEADLINE_TICKS) {
                __hip_atomic_store(tmo, call, __ATOMIC_RELAXED, __HIP_MEMORY_SCOPE_AGENT);
                break;
            }
            __builtin_amdgcn_s_sleep(2);
        }
        __builtin_amdgcn_fence(__ATOMIC_ACQUIRE, "");
        asm volatile("s_waitcnt vmcnt(0)" ::: "memory");
    }
    __syncthreads();

    const int wr = wid >> 2, wc = wid & 3;
    const char* cA = wsl + abuf + (size_t)pm * PANEL_B;
    const char* cB = wsl + WS_B + (size_t)pn * PANEL_B;
    const unsigned voff = (unsigned)(wid * 1024 + lane * 16);
    const unsigned ldsw = (unsigned)wid * 1024u;
    const int aoff = wr * 8192 + lane * 16, boff = wc * 4096 + lane * 16;
#define SA(b, h) (((b) * 2 + (h)) * HTB)
#define SB(b, h) ((4 + (b) * 2 + (h)) * HTB)
#define STAGE(bufoff, gbase) do { _Pragma("unroll") for (int _i = 0; _i < 2; ++_i) \
        __builtin_amdgcn_global_load_lds((const unsigned*)((const char*)(gbase) + voff + _i * 8192), (LAS unsigned*)(lds + (bufoff) + ldsw + _i * 8192), 16, 0, 0); } while (0)
#define LDA(dst, b, h) do { _Pragma("unroll") for (int m = 0; m < 4; ++m) _Pragma("unroll") for (int k = 0; k < 2; ++k) dst[m][k] = *(const LAS half8*)(lds + SA(b, h) + aoff + m * 2048 + k * 1024); } while (0)
#define LDB(dst, b, h) do { _Pragma("unroll") for (int n = 0; n < 2; ++n) _Pragma("unroll") for (int k = 0; k < 2; ++k) dst[n][k] = *(const LAS half8*)(lds + SB(b, h) + boff + n * 2048 + k * 1024); } while (0)
#define MMA(ai, bj, At, Bt) do { __builtin_amdgcn_s_setprio(1); _Pragma("unroll") for (int m = 0; m < 4; ++m) _Pragma("unroll") for (int n = 0; n < 2; ++n) _Pragma("unroll") for (int k = 0; k < 2; ++k) \
        acc[ai][bj][m][n] = __builtin_amdgcn_mfma_f32_16x16x32_f16(At[m][k], Bt[n][k], acc[ai][bj][m][n], 0, 0, 0); __builtin_amdgcn_s_setprio(0); } while (0)
#define WAIT_V(n) asm volatile("s_waitcnt vmcnt(" #n ")" ::: "memory")
#define WAIT_L(n) asm volatile("s_waitcnt lgkmcnt(" #n ")" ::: "memory")
#define BAR __builtin_amdgcn_s_barrier()
#define SCHED __builtin_amdgcn_sched_barrier(0)
    f32x4 acc[2][2][4][2];
#pragma unroll
    for (int a = 0; a < 2; ++a)
#pragma unroll
        for (int b = 0; b < 2; ++b)
#pragma unroll
            for (int m = 0; m < 4; ++m)
#pragma unroll
                for (int n = 0; n < 2; ++n) acc[a][b][m][n] = (f32x4){0.f, 0.f, 0.f, 0.f};
    half8 At[4][2], B0[2][2], B1[2][2];
    constexpr size_t KS = 2 * HTB;
    STAGE(SB(0, 0), cB); STAGE(SA(0, 0), cA); STAGE(SB(0, 1), cB + HTB); STAGE(SA(0, 1), cA + HTB);
    if (wr == 1) BAR;
    WAIT_V(4); BAR;
    STAGE(SB(1, 0), cB + KS); STAGE(SA(1, 0), cA + KS); STAGE(SB(1, 1), cB + KS + HTB);
    WAIT_V(6); BAR;
    for (int t = 0; t < NT; t += 2) {
        const bool last = (t == NT - 2);
        const char* a1 = cA + (size_t)(t + 1) * KS;
        const char* a2 = last ? cA : cA + (size_t)(t + 2) * KS; const char* b2 = last ? cB : cB + (size_t)(t + 2) * KS;
        const char* a3 = a2 + KS; const char* b3 = b2 + KS;
        LDB(B0, 0, 0); SCHED; LDA(At, 0, 0); STAGE(SA(1, 1), a1 + HTB);
        WAIT_L(8); BAR; WAIT_L(0); MMA(0, 0, At, B0); BAR; SCHED;
        LDB(B1, 0, 1); STAGE(SB(0, 0), b2);
        BAR; WAIT_L(0); MMA(0, 1, At, B1); BAR;
        LDA(At, 0, 1); STAGE(SA(0, 0), a2);
        BAR; WAIT_L(0); MMA(1, 0, At, B0); BAR; SCHED;
        STAGE(SB(0, 1), b2 + HTB);
        WAIT_V(6); BAR; MMA(1, 1, At, B1); BAR;
        LDB(B0, 1, 0); SCHED; LDA(At, 1, 0); STAGE(SA(0, 1), a2 + HTB);
        WAIT_L(8); BAR; WAIT_L(0); MMA(0, 0, At, B0); BAR; SCHED;
        LDB(B1, 1, 1); STAGE(SB(1, 0), b3);
        BAR; WAIT_L(0); MMA(0, 1, At, B1); BAR;
        LDA(At, 1, 1); STAGE(SA(1, 0), a3);
        BAR; WAIT_L(0); MMA(1, 0, At, B0); BAR; SCHED;
        STAGE(SB(1, 1), b3 + HTB);
        WAIT_V(6); BAR; MMA(1, 1, At, B1); BAR;
    }
    WAIT_V(0);
    if (wr == 0) BAR;
    {
        int tz = threadIdx.x; asm volatile("" : "+v"(tz));
        const int ln = tz & 63, fr = ln & 15, fq = ln >> 4;
        const bool bad = __hip_atomic_load(tmo, __ATOMIC_RELAXED, __HIP_MEMORY_SCOPE_AGENT) == call;
        const float qn = __builtin_nanf("");
        float* cp = p.out + (size_t)(pm * 256 + wr * 64 + fq * 4) * GN + pn * 256 + wc * 64 + fr * 4;
#pragma unroll
        for (int ai = 0; ai < 2; ++ai)
#pragma unroll
            for (int m = 0; m < 4; ++m)
#pragma unroll
                for (int j = 0; j < 4; ++j) {
                    f32x4 v = {acc[ai][0][m][0][j], acc[ai][0][m][1][j], acc[ai][1][m][0][j], acc[ai][1][m][1][j]};
                    v[0] = fmaxf(v[0], 0.f); v[1] = fmaxf(v[1], 0.f); v[2] = fmaxf(v[2], 0.f); v[3] = fmaxf(v[3], 0.f);
                    if (bad) v = (f32x4){qn, qn, qn, qn};
                    *(f32x4*)(cp + (size_t)(ai * 128 + m * 16 + j) * GN) = v;
                }
    }
}

}

extern "C" void kernel_launch(int rank, int nranks, ncclComm_t comm,
                              hipStream_t stream,
                              void* const* d_in, const int* in_sizes, int n_in,
                              void* d_out, int out_size,
                              const int* in_dtypes, int out_dtype,
                              void* const* d_out_all,
                              void* const* d_ws_all, size_t ws_bytes) {
    if (nranks != WORLD || ws_bytes < WS_END || n_in < 2) { fprintf(stderr, "unsupported configuration\n"); return; }
    static bool init = false;
    if (!init) {
        HIP_CHECK(hipFuncSetAttribute((const void*)fused_kernel, hipFuncAttributeMaxDynamicSharedMemorySize, LDS_BYTES));
        init = true;
    }
    Params p;
    p.x = (const float*)d_in[0]; p.w = (const float*)d_in[1]; p.out = (float*)d_out; p.rank = rank;
    for (int i = 0; i < WORLD; ++i) p.ws[i] = (char*)d_ws_all[i];
    hipLaunchKernelGGL(fused_kernel, dim3(GRID), dim3(512), 256, stream, p, 0);
    hipLaunchKernelGGL(fused_kernel, dim3(GRID), dim3(512), LDS_BYTES, stream, p, 1);
    HIP_CHECK(hipGetLastError());
}
```

```cpp
#include <hip/hip_runtime.h>
#include <rccl/rccl.h>
#include <cstdio>
#include <cstdint>
#include <utility>

#define HIP_CHECK(expr) do { hipError_t _e = (expr); if (_e != hipSuccess) { \
    fprintf(stderr, "HIP error %s at %s:%d\n", hipGetErrorString(_e), __FILE__, __LINE__); \
    return; } } while (0)

namespace {

template <int... Is, class F> __device__ __forceinline__ void sfor_impl(std::integer_sequence<int, Is...>, F&& f) { (f(std::integral_constant<int, Is>{}), ...); }
template <int N, class F> __device__ __forceinline__ void sfor(F&& f) { sfor_impl(std::make_integer_sequence<int, N>{}, f); }

typedef _Float16 half8 __attribute__((ext_vector_type(8)));
typedef float f32x4 __attribute__((ext_vector_type(4)));
typedef unsigned u32x4 __attribute__((ext_vector_type(4)));
typedef unsigned u32x2 __attribute__((ext_vector_type(2)));
#define LAS __attribute__((address_space(3)))

constexpr int WORLD = 8;
constexpr int GM = 4096, GK = 4096, GN = 4096;
constexpr int MS = GM / WORLD;
constexpr int NT = GK / 64;
constexpr int TB = 32768;
constexpr size_t PANEL_B = (size_t)NT * TB;
constexpr size_t OPER_B = 16 * PANEL_B;
constexpr size_t WS_CTL = 0;
constexpr size_t WS_SLOT = 4096;
constexpr size_t WS_FLAGA = 65536;
constexpr size_t WS_FLAGB = 262144;
constexpr size_t WS_DBG = 512u << 10;
constexpr size_t WS_A0 = 1u << 20;
constexpr size_t WS_B = WS_A0 + 2 * OPER_B;
constexpr size_t WS_X32 = WS_B + OPER_B;
constexpr size_t WS_END = WS_X32 + 4 * OPER_B;
constexpr int CW_TICKET = 0, CW_TMO = 256;
constexpr unsigned long long DEADLINE_TICKS = 50000000ull;
constexpr int LDS_X = 4 * TB;
constexpr int LDS_BYTES = 4 * TB + 16384;
constexpr int GRID = 256;
#define NTHREADS 256
#define PACE 66
#define KEARLY 8
#define KBURST 128
#define TILE_TICKS 140
#define SLOPE 43
#define OFFL 230
#define OFFR 440
#define ABL_NOPOLL 0
#define ABL_NOW 0
#define ABL_NOPUSH 0
#define SPINSLEEP 8
#define POLLP 1
#define PAHEAD 10
#define PBHEAD 7
#define APF 5

struct Params { const float* x; const float* w; float* out; char* ws[WORLD]; int rank; };

template <class T> __device__ __forceinline__ T* uptr(T* q) {
    unsigned long long v = (unsigned long long)q;
    unsigned lo = __builtin_amdgcn_readfirstlane((unsigned)v), hi = __builtin_amdgcn_readfirstlane((unsigned)(v >> 32));
    return (T*)(((unsigned long long)hi << 32) | lo);
}
__device__ __forceinline__ u32x4 cvt8(const f32x4 a, const f32x4 b, float s) {
    half8 h;
    h[0] = (_Float16)(a[0] * s); h[1] = (_Float16)(a[1] * s); h[2] = (_Float16)(a[2] * s); h[3] = (_Float16)(a[3] * s);
    h[4] = (_Float16)(b[0] * s); h[5] = (_Float16)(b[1] * s); h[6] = (_Float16)(b[2] * s); h[7] = (_Float16)(b[3] * s);
    return __builtin_bit_cast(u32x4, h);
}
__device__ __forceinline__ unsigned absbits(float v) { return __float_as_uint(v) & 0x7fffffffu; }
__device__ __forceinline__ unsigned amax4(const f32x4 a) { return max(max(absbits(a[0]), absbits(a[1])), max(absbits(a[2]), absbits(a[3]))); }
__device__ __forceinline__ unsigned wave_umax(unsigned v) {
    v = max(v, (unsigned)__builtin_amdgcn_update_dpp(0, (int)v, 0x111, 0xf, 0xf, false));
    v = max(v, (unsigned)__builtin_amdgcn_update_dpp(0, (int)v, 0x112, 0xf, 0xf, false));
    v = max(v, (unsigned)__builtin_amdgcn_update_dpp(0, (int)v, 0x114, 0xf, 0xf, false));
    v = max(v, (unsigned)__builtin_amdgcn_update_dpp(0, (int)v, 0x118, 0xf, 0xf, false));
    v = max(v, (unsigned)__builtin_amdgcn_update_dpp(0, (int)v, 0x142, 0xa, 0xf, false));
    v = max(v, (unsigned)__builtin_amdgcn_update_dpp(0, (int)v, 0x143, 0xc, 0xf, false));
    return (unsigned)__builtin_amdgcn_readlane((int)v, 63);
}
__device__ __forceinline__ int scale_exp(unsigned maxbits, int target) {
    const int ef = (int)(maxbits >> 23);
    if (ef == 0 || ef == 255) return 0;
    int e = target - (ef - 127);
    return e > 126 ? 126 : (e < -126 ? -126 : e);
}
__device__ __forceinline__ float exp2i(int e) { return __uint_as_float((unsigned)(e + 127) << 23); }

constexpr int R_PB = 58, R_PA = 60, R_W = 64, R_AF = 96, R_B = 128, R_X = 128, R_S = 96, R_SL = 112;
#define OWNED_REGS "a0","a1","a2","a3","a4","a5","a6","a7","a8","a9","a10","a11","a12","a13","a14","a15","a16","a17","a18","a19","a20","a21","a22","a23","a24","a25","a26","a27","a28","a29","a30","a31","a32","a33","a34","a35","a36","a37","a38","a39","a40","a41","a42","a43","a44","a45","a46","a47","a48","a49","a50","a51","a52","a53","a54","a55","a56","a57","a58","a59","a60","a61","a62","a63","a64","a65","a66","a67","a68","a69","a70","a71","a72","a73","a74","a75","a76","a77","a78","a79","a80","a81","a82","a83","a84","a85","a86","a87","a88","a89","a90","a91","a92","a93","a94","a95","a96","a97","a98","a99","a100","a101","a102","a103","a104","a105","a106","a107","a108","a109","a110","a111","a112","a113","a114","a115","a116","a117","a118","a119","a120","a121","a122","a123","a124","a125","a126","a127","a128","a129","a130","a131","a132","a133","a134","a135","a136","a137","a138","a139","a140","a141","a142","a143","a144","a145","a146","a147","a148","a149","a150","a151","a152","a153","a154","a155","a156","a157","a158","a159","a160","a161","a162","a163","a164","a165","a166","a167","a168","a169","a170","a171","a172","a173","a174","a175","a176","a177","a178","a179","a180","a181","a182","a183","a184","a185","a186","a187","a188","a189","a190","a191","a192","a193","a194","a195","a196","a197","a198","a199","a200","a201","a202","a203","a204","a205","a206","a207","a208","a209","a210","a211","a212","a213","a214","a215","a216","a217","a218","a219","a220","a221","a222","a223","a224","a225","a226","a227","a228","a229","a230","a231","a232","a233","a234","a235","a236","a237","a238","a239","a240","a241","a242","a243","a244","a245","a246","a247","a248","a249","a250","a251","a252","a253","a254","a255","v58","v59","v60","v61","v62","v63","v64","v65","v66","v67","v68","v69","v70","v71","v72","v73","v74","v75","v76","v77","v78","v79","v80","v81","v82","v83","v84","v85","v86","v87","v88","v89","v90","v91","v92","v93","v94","v95","v96","v97","v98","v99","v100","v101","v102","v103","v104","v105","v106","v107","v108","v109","v110","v111","v112","v113","v114","v115","v116","v117","v118","v119","v120","v121","v122","v123","v124","v125","v126","v127","v128","v129","v130","v131","v132","v133","v134","v135","v136","v137","v138","v139","v140","v141","v142","v143","v144","v145","v146","v147","v148","v149","v150","v151","v152","v153","v154","v155","v156","v157","v158","v159","v160","v161","v162","v163","v164","v165","v166","v167","v168","v169","v170","v171","v172","v173","v174","v175","v176","v177","v178","v179","v180","v181","v182","v183","v184","v185","v186","v187","v188","v189","v190","v191","v192","v193","v194","v195","v196","v197","v198","v199","v200","v201","v202","v203","v204","v205","v206","v207","v208","v209","v210","v211","v212","v213","v214","v215","v216","v217","v218","v219","v220","v221","v222","v223","v224","v225","v226","v227","v228","v229","v230","v231","v232","v233","v234","v235","v236","v237","v238","v239","v240","v241","v242","v243","v244","v245","v246","v247","v248","v249","v250","v251","v252","v253","v254","v255"
#define ALL_AGPRS "a0","a1","a2","a3","a4","a5","a6","a7","a8","a9","a10","a11","a12","a13","a14","a15","a16","a17","a18","a19","a20","a21","a22","a23","a24","a25","a26","a27","a28","a29","a30","a31","a32","a33","a34","a35","a36","a37","a38","a39","a40","a41","a42","a43","a44","a45","a46","a47","a48","a49","a50","a51","a52","a53","a54","a55","a56","a57","a58","a59","a60","a61","a62","a63","a64","a65","a66","a67","a68","a69","a70","a71","a72","a73","a74","a75","a76","a77","a78","a79","a80","a81","a82","a83","a84","a85","a86","a87","a88","a89","a90","a91","a92","a93","a94","a95","a96","a97","a98","a99","a100","a101","a102","a103","a104","a105","a106","a107","a108","a109","a110","a111","a112","a113","a114","a115","a116","a117","a118","a119","a120","a121","a122","a123","a124","a125","a126","a127","a128","a129","a130","a131","a132","a133","a134","a135","a136","a137","a138","a139","a140","a141","a142","a143","a144","a145","a146","a147","a148","a149","a150","a151","a152","a153","a154","a155","a156","a157","a158","a159","a160","a161","a162","a163","a164","a165","a166","a167","a168","a169","a170","a171","a172","a173","a174","a175","a176","a177","a178","a179","a180","a181","a182","a183","a184","a185","a186","a187","a188","a189","a190","a191","a192","a193","a194","a195","a196","a197","a198","a199","a200","a201","a202","a203","a204","a205","a206","a207","a208","a209","a210","a211","a212","a213","a214","a215","a216","a217","a218","a219","a220","a221","a222","a223","a224","a225","a226","a227","a228","a229","a230","a231","a232","a233","a234","a235","a236","a237","a238","a239","a240","a241","a242","a243","a244","a245","a246","a247","a248","a249","a250","a251","a252","a253","a254","a255"
template <int R, int OFF> __device__ __forceinline__ void gload16(unsigned voff, const char* sbase) {
    asm volatile("global_load_dwordx4 v[%c2:%c2+3], %0, %1 offset:%c3" :: "v"(voff), "s"(sbase), "i"(R), "i"(OFF) : "memory");
}
template <int R> __device__ __forceinline__ void gload16_sys(unsigned voff, const char* sbase) {
    asm volatile("global_load_dwordx4 v[%c2:%c2+3], %0, %1 sc0 sc1" :: "v"(voff), "s"(sbase), "i"(R) : "memory");
}
template <int R> __device__ __forceinline__ void gload16_agent(unsigned voff, const char* sbase) {
    asm volatile("global_load_dwordx4 v[%c2:%c2+3], %0, %1 sc1" :: "v"(voff), "s"(sbase), "i"(R) : "memory");
}
template <int R> __device__ __forceinline__ void gload8_agent(unsigned voff, const char* sbase) {
    asm volatile("global_load_dwordx2 v[%c2:%c2+1], %0, %1 sc1" :: "v"(voff), "s"(sbase), "i"(R) : "memory");
}
template <int R> __device__ __forceinline__ unsigned rdreg() { unsigned v; asm volatile("v_mov_b32 %0, v%c1" : "=v"(v) : "i"(R)); return v; }
template <int OFF> __device__ __forceinline__ void gstore16_sys(unsigned voff, const u32x4& v, char* sbase) {
    asm volatile("global_store_dwordx4 %0, %1, %2 offset:%3 sc0 sc1\n\ts_nop 1" :: "v"(voff), "v"(v), "s"(sbase), "i"(OFF) : "memory");
}
template <int OFF> __device__ __forceinline__ void gstore16_agent(unsigned voff, const u32x4& v, char* sbase) {
    asm volatile("global_store_dwordx4 %0, %1, %2 offset:%3 sc1\n\ts_nop 1" :: "v"(voff), "v"(v), "s"(sbase), "i"(OFF) : "memory");
}
__device__ __forceinline__ void gstore64_sys_lane(unsigned voff, const u32x4& v0, const u32x4& v1, const u32x4& v2, const u32x4& v3, unsigned plo, unsigned phi, int ln) {
    const unsigned long long b = ((unsigned long long)(unsigned)__builtin_amdgcn_readlane((int)phi, ln) << 32) | (unsigned)__builtin_amdgcn_readlane((int)plo, ln);
    asm volatile("s_nop 4\n\tglobal_store_dwordx4 %0, %1, %5 sc0 sc1\n\tglobal_store_dwordx4 %0, %2, %5 offset:1024 sc0 sc1\n\t"
                 "global_store_dwordx4 %0, %3, %5 offset:2048 sc0 sc1\n\tglobal_store_dwordx4 %0, %4, %5 offset:3072 sc0 sc1\n\ts_nop 1"
                 :: "v"(voff), "v"(v0), "v"(v1), "v"(v2), "v"(v3), "s"(b) : "memory");
}
template <int R, int OFF> __device__ __forceinline__ void storeown_sys(unsigned voff, char* sbase) {
    asm volatile("s_nop 4\n\tglobal_store_dwordx4 %0, v[%c2:%c2+3], %1 offset:%c3 sc0 sc1\n\ts_nop 1" :: "v"(voff), "s"(sbase), "i"(R), "i"(OFF) : "memory");
}
__device__ __forceinline__ void gstore8_sys(char* addr, const u32x2& v) {
    asm volatile("global_store_dwordx2 %0, %1, off sc0 sc1\n\ts_nop 1" :: "v"(addr), "v"(v) : "memory");
}
__device__ __forceinline__ void gstore8_agent(unsigned voff, const u32x2& v, char* sbase) {
    asm volatile("global_store_dwordx2 %0, %1, %2 sc1\n\ts_nop 1" :: "v"(voff), "v"(v), "s"(sbase) : "memory");
}
__device__ __forceinline__ void gstore4_agent(unsigned voff, unsigned v, char* sbase) {
    asm volatile("global_store_dword %0, %1, %2 sc1\n\ts_nop 1" :: "v"(voff), "v"(v), "s"(sbase) : "memory");
}
__device__ __forceinline__ unsigned long long rtclock() {
    unsigned long long t; asm volatile("s_memrealtime %0\n\ts_waitcnt lgkmcnt(0)" : "=s"(t) :: "memory"); return t;
}
__device__ __forceinline__ unsigned load_word(const char* sbase) {
    unsigned v; asm volatile("global_load_dword %0, %1, %2 sc1\n\ts_waitcnt vmcnt(0)" : "=&v"(v) : "v"(0u), "s"(sbase) : "memory");
    return __builtin_amdgcn_readfirstlane(v);
}
__device__ __forceinline__ int give_up(char* tmo, unsigned call, unsigned long long t0) {
    if (load_word(tmo) == call) return 1;
    if (rtclock() - t0 > DEADLINE_TICKS) { gstore4_agent(0u, call, tmo); return 1; }
    __builtin_amdgcn_s_sleep(1);
    return 0;
}
template <int R, int OFF> __device__ __forceinline__ void ldsrd(unsigned lo, unsigned hi) {
    if constexpr (OFF < 65536) asm volatile("ds_read_b128 v[%c1:%c1+3], %0 offset:%c2" :: "v"(lo), "i"(R), "i"(OFF) : "memory");
    else asm volatile("ds_read_b128 v[%c1:%c1+3], %0 offset:%c2" :: "v"(hi), "i"(R), "i"(OFF - 65536) : "memory");
}
template <int OFF> __device__ __forceinline__ void lds_write16(unsigned a, const u32x4& v) { asm volatile("ds_write_b128 %0, %1 offset:%2" :: "v"(a), "v"(v), "i"(OFF) : "memory"); }
template <int OFF> __device__ __forceinline__ u32x4 lds_read16_wait(unsigned a) { u32x4 v; asm volatile("ds_read_b128 %0, %1 offset:%2\n\ts_waitcnt lgkmcnt(0)" : "=&v"(v) : "v"(a), "i"(OFF) : "memory"); return v; }
template <int N> __device__ __forceinline__ void wait_vm() { asm volatile("s_waitcnt vmcnt(%0)" :: "i"(N) : "memory"); }
template <int R> __device__ __forceinline__ void amax4r(float& m) {
    asm volatile("v_max3_f32 %0, |v%c1|, |v[%c1+1]|, %0\n\tv_max3_f32 %0, |v[%c1+2]|, |v[%c1+3]|, %0" : "+v"(m) : "i"(R));
}
template <int R, int N> __device__ __forceinline__ unsigned amax_regs() {
    float m = 0.f;
    sfor<N / 4>([&](auto I) __attribute__((always_inline)) { amax4r<R + 4 * I>(m); });
    return __float_as_uint(m);
}
template <int R, int S> __device__ __forceinline__ u32x4 cvt8r(float s) {
    unsigned a, b, c, d;
    asm volatile("v_fma_mixlo_f16 %0, v%c5, %4, 0\n\tv_fma_mixlo_f16 %1, v[%c5+2*%c6], %4, 0\n\tv_fma_mixlo_f16 %2, v[%c5+4*%c6], %4, 0\n\tv_fma_mixlo_f16 %3, v[%c5+6*%c6], %4, 0\n\t"
                 "v_fma_mixhi_f16 %0, v[%c5+%c6], %4, 0\n\tv_fma_mixhi_f16 %1, v[%c5+3*%c6], %4, 0\n\tv_fma_mixhi_f16 %2, v[%c5+5*%c6], %4, 0\n\tv_fma_mixhi_f16 %3, v[%c5+7*%c6], %4, 0"
                 : "=&v"(a), "=&v"(b), "=&v"(c), "=&v"(d) : "v"(s), "i"(R), "i"(S));
    return (u32x4){a, b, c, d};
}
template <int R, int W, int A, int B> __device__ __forceinline__ void mfma_group() {
    asm volatile("s_waitcnt lgkmcnt(%c0)\n\t"
                 "v_mfma_f32_16x16x32_f16 a[%c1:%c1+3], v[%c2:%c2+3], v[%c3:%c3+3], a[%c1:%c1+3]\n\t"
                 "v_mfma_f32_16x16x32_f16 a[%c1+4:%c1+7], v[%c2:%c2+3], v[%c3+4:%c3+7], a[%c1+4:%c1+7]\n\t"
                 "v_mfma_f32_16x16x32_f16 a[%c1+8:%c1+11], v[%c2:%c2+3], v[%c3+8:%c3+11], a[%c1+8:%c1+11]\n\t"
                 "v_mfma_f32_16x16x32_f16 a[%c1+12:%c1+15], v[%c2:%c2+3], v[%c3+12:%c3+15], a[%c1+12:%c1+15]"
                 :: "i"(W), "i"(R), "i"(A), "i"(B) : "memory", ALL_AGPRS);
}
template <int R, int A, int B> __device__ __forceinline__ void mfma1() {
    asm volatile("v_mfma_f32_16x16x32_f16 a[%c0:%c0+3], v[%c1:%c1+3], v[%c2:%c2+3], a[%c0:%c0+3]" :: "i"(R), "i"(A), "i"(B) : "memory");
}
template <int R, int W, int A, int B> __device__ __forceinline__ void mfma1w() {
    asm volatile("s_waitcnt lgkmcnt(%c0)\n\tv_mfma_f32_16x16x32_f16 a[%c1:%c1+3], v[%c2:%c2+3], v[%c3:%c3+3], a[%c1:%c1+3]" :: "i"(W), "i"(R), "i"(A), "i"(B) : "memory", ALL_AGPRS);
}
template <int R0, int R1> __device__ __forceinline__ unsigned long long ready_mask(unsigned thr) {
    unsigned long long m; unsigned t;
    asm volatile("v_min_u32 %1, v%c3, v%c4\n\tv_cmp_le_u32 %0, %2, %1" : "=s"(m), "=&v"(t) : "s"(thr), "i"(R0), "i"(R1));
    return m;
}
template <int R> __device__ __forceinline__ void acc_zero4() {
    asm volatile("v_accvgpr_write_b32 a%c0, 0\n\tv_accvgpr_write_b32 a[%c0+1], 0\n\tv_accvgpr_write_b32 a[%c0+2], 0\n\tv_accvgpr_write_b32 a[%c0+3], 0" :: "i"(R) : ALL_AGPRS);
}
template <int R> __device__ __forceinline__ void acc_zero16(const u32x4& z) {
    asm volatile("v_mfma_f32_32x32x16_f16 a[%c1:%c1+15], %0, %0, 0" :: "v"(z), "i"(R) : ALL_AGPRS);
}
__device__ __forceinline__ void acc_zero_all() {
    u32x4 z = {0u, 0u, 0u, 0u}; asm volatile("s_nop 1" : "+v"(z));
    sfor<16>([&](auto I) __attribute__((always_inline)) { acc_zero16<I * 16>(z); });
    asm volatile("s_nop 15\n\ts_nop 3" ::: "memory");
}
template <int R> __device__ __forceinline__ f32x4 acc_read_row() {
    float a, b, c, d;
    asm volatile("v_accvgpr_read_b32 %0, a%c4\n\tv_accvgpr_read_b32 %1, a[%c4+4]\n\tv_accvgpr_read_b32 %2, a[%c4+8]\n\tv_accvgpr_read_b32 %3, a[%c4+12]"
                 : "=v"(a), "=v"(b), "=v"(c), "=v"(d) : "i"(R));
    return (f32x4){a, b, c, d};
}
template <int R> __device__ __forceinline__ void mfma_f32(float a, float b) {
    asm volatile("s_nop 1\n\tv_mfma_f32_16x16x4_f32 a[%c2:%c2+3], %0, %1, a[%c2:%c2+3]" :: "v"(a), "v"(b), "i"(R) : ALL_AGPRS);
}

#define STAMP(i) do {} while (0)

__global__ __launch_bounds__(256, 1) __attribute__((amdgpu_num_vgpr(58))) void fused_kernel(Params p) {
    extern __shared__ __attribute__((aligned(16))) unsigned char smem[];
    LAS unsigned char* lds = (LAS unsigned char*)smem;
    const int tid = threadIdx.x, wid = __builtin_amdgcn_readfirstlane(tid >> 6), lane = tid & 63;
    const int fr = lane & 15, fq = lane >> 4;
    const int rank = p.rank;
    char* wsl = uptr(p.ws[0]);
    unsigned* ctl = (unsigned*)(wsl + WS_CTL);
    char* tmo = wsl + WS_CTL + CW_TMO * 4;
    asm volatile("" ::: OWNED_REGS);
    STAMP(0);
    const int bid = blockIdx.x, xcd = bid & 7, idx = bid >> 3;
    const int pm = (idx >> 2) * 2 + (xcd & 1), pn = (xcd >> 1) * 4 + (idx & 3);
    const bool local = (pm >> 1) == rank;
    const int jwg = local ? 224 + (idx & 3) * 8 + xcd : (((idx >> 2) < rank ? (idx >> 2) : (idx >> 2) - 1) * 4 + (idx & 3)) * 8 + xcd;
    const int unit = (jwg >> 2) + 64 * wid, uq = jwg & 3;
    const int ukh = unit >> 1, upl = unit & 1, upm = rank * 2 + upl;
    const bool burst = ukh < KBURST;

    const unsigned voff0 = lane * 16, voff1 = lane * 16 + 4096;
    unsigned ticket = 0;
    if (lane == 0) asm volatile("global_atomic_add %0, %1, %2, %3 sc0" : "=v"(ticket) : "v"(0u), "v"(1u), "s"(wsl + WS_CTL + CW_TICKET * 4 + wid * 128) : "memory");
    const char* xb = (const char*)uptr(p.x) + ((size_t)(upl * 256 + uq * 64) * GK + ukh * 32) * 4;
    const unsigned xv = (unsigned)(fr * GK + fq * 8) * 4u;
    sfor<4>([&](auto I) __attribute__((always_inline)) { constexpr int i = I; gload16<R_S + 16 + 4 * i, 0>((unsigned)((lane * 8 + i * 2 + 1) * GK + ((lane * 68 + i * 1028) & (GK - 1))) * 4u, (const char*)uptr(p.x)); });
    sfor<4>([&](auto I) __attribute__((always_inline)) { constexpr int i = I; gload16<R_X + 8 * i, 0>(xv, xb + (size_t)i * 16 * GK * 4); gload16<R_X + 8 * i + 4, 16>(xv, xb + (size_t)i * 16 * GK * 4); });
    const unsigned wvoff = (unsigned)(fq * 8 * GN + fr * 4) * 4u;
    const char* wbase = (const char*)uptr(p.w) + ((size_t)pm * 32 * GN + pn * 256 + wid * 64) * 4;
    sfor<4>([&](auto I) __attribute__((always_inline)) { constexpr int i = I; gload16<R_S + 4 * i, 0>((unsigned)((lane * 64 + i * 16 + 5) * GN + ((lane * 68 + i * 1028) & (GN - 1))) * 4u, (const char*)uptr(p.w)); });
    sfor<8>([&](auto J) __attribute__((always_inline)) { constexpr int j = J; gload16<R_W + 4 * j, 0>(wvoff, wbase + (size_t)j * GN * 4); });
    {
        unsigned long long pc; unsigned sz;
        asm volatile("s_getpc_b64 %0\n.Lagk_begin:\n\ts_mov_b32 %1, .Lagk_end-.Lagk_begin" : "=s"(pc), "=s"(sz));
        gload16<R_B + 124, 0>(min((unsigned)(idx * 4 + wid) * 1024u, sz - 1024u) + lane * 16, (const char*)pc);
    }
    const int r0 = (unit * 4 + uq * 2) % 7;
    char* fdst = wsl;
#pragma unroll
    for (int i = 1; i < WORLD; ++i) { char* q = p.ws[i]; fdst = (lane & 7) == i ? q : fdst; }
    STAMP(15);
    asm volatile("s_waitcnt vmcnt(13)" : "+v"(ticket) :: "memory");
    STAMP(14);
    const unsigned call = __builtin_amdgcn_readfirstlane(ticket) / GRID + 1u;
    const size_t abuf = WS_A0 + (size_t)(call & 1u) * OPER_B;
    const size_t faoff = WS_FLAGA + (size_t)(call & 1u) * 65536;
    int dead = 0;
    const int ex = scale_exp(wave_umax(amax_regs<R_S + 16, 16>()), 4);
    const unsigned tx0 = (unsigned)rtclock();
    STAMP(3);
    u32x4 xq[4];
    bool xbad;
    {
        const float sx = exp2i(ex);
        sfor<4>([&](auto I) __attribute__((always_inline)) { constexpr int i = I; xq[i] = cvt8r<R_X + 8 * i, 1>(sx); });
        const float a = __uint_as_float(amax_regs<R_X, 32>()) * sx;
        xbad = __ballot(!(a < 60000.0f)) != 0ull || (__ballot(a >= 0.000244140625f) == 0ull && __ballot(a != 0.0f) != 0ull);
    }
    const unsigned xoff = (unsigned)(abuf + (size_t)upm * PANEL_B + (size_t)ukh * 16384 + uq * 4096 + lane * 16);
    char* fladdr = fdst + faoff + (((size_t)upm * 128 + ukh) * 4 + uq) * 8;
    const u32x2 flval = {call, (unsigned)(ex & 0xffff) | (xbad ? 0x10000u : 0u)};
    const unsigned plo = (unsigned)(unsigned long long)fdst, phi = (unsigned)((unsigned long long)fdst >> 32);
    const unsigned xlds = (unsigned)(uintptr_t)lds + LDS_X + wid * 4096 + lane * 16;
    sfor<4>([&](auto I) __attribute__((always_inline)) { constexpr int i = I; lds_write16<i * 1024>(xlds, xq[i]); });
    int pst = 0;
    auto push_all = [&]() __attribute__((always_inline)) {
        STAMP(11);
        const u32x4 v0 = lds_read16_wait<0>(xlds), v1 = lds_read16_wait<1024>(xlds), v2 = lds_read16_wait<2048>(xlds), v3 = lds_read16_wait<3072>(xlds);
        int q = r0;
#pragma unroll 1
        for (int d = 0; d < 7; ++d) {
            gstore64_sys_lane(xoff, v0, v1, v2, v3, plo, phi, q + 1);
            q = q == 6 ? 0 : q + 1;
        }
        pst = 1;
    };
    auto push_flag = [&]() __attribute__((always_inline)) { if (lane >= 1 && lane < 8) gstore8_sys(fladdr, flval); pst = 5; STAMP(12); };
    const unsigned due = tx0 + (unsigned)(ukh * PACE);
    if (__builtin_expect(xbad, 0)) {
        const unsigned so = (unsigned)(WS_X32 + (size_t)(call & 1u) * 2 * OPER_B + ((size_t)(upm * 256 + uq * 64) * GK + ukh * 32) * 4) + xv;
#pragma unroll 1
        for (int d = 0; d < WORLD; ++d) {
            const unsigned long long bq = ((unsigned long long)(unsigned)__builtin_amdgcn_readlane((int)phi, d) << 32) | (unsigned)__builtin_amdgcn_readlane((int)plo, d);
            sfor<8>([&](auto I) __attribute__((always_inline)) { constexpr int i = I; storeown_sys<R_X + 4 * i, (i & 1) * 16>(so + (unsigned)(i >> 1) * 16u * GK * 4u, (char*)bq); });
        }
        wait_vm<0>();
    }
    if (ukh == 0) push_all();
    sfor<4>([&](auto I) __attribute__((always_inline)) { constexpr int i = I; gstore16_sys<i * 1024>(xoff, xq[i], wsl); });
    if (ukh == 0) wait_vm<33>(); else wait_vm<5>();
    const int ew = scale_exp(wave_umax(amax_regs<R_S, 16>()), 0);
    const float sw = exp2i(ew);
    char* bdst = wsl + WS_B + (size_t)pn * PANEL_B + (size_t)(pm >> 1) * TB + wid * 8192 + (pm & 1) * 4096;
    char* fbdst = wsl + WS_FLAGB + ((size_t)(pn * 4 + wid) * 128 + pm) * 4;
    auto wbits = [&]() __attribute__((always_inline)) -> unsigned {
        const float a = __uint_as_float(amax_regs<R_W, 32>()) * sw;
        return call * 4u + ((__ballot(!(a < 60000.0f)) ? 1u : 0u) | (__ballot(a >= 0.015625f) ? 2u : 0u));
    };
    auto wstore = [&](auto C, char* dst) __attribute__((always_inline)) { constexpr int c = C; gstore16_agent<c * 1024>(voff0, cvt8r<R_W + c, 4>(sw), dst); };
    sfor<4>([&](auto C) __attribute__((always_inline)) { wstore(C, bdst); });
    const unsigned fb0 = wbits();
    wait_vm<0>();
    gstore4_agent(0u, fb0, fbdst);
    if (lane == 0) gstore8_sys(fladdr, flval);
    if (pst == 1) push_flag();
    if (pst == 0 && ukh < KEARLY) {
        while ((int)((unsigned)rtclock() - due) < 0) __builtin_amdgcn_s_sleep(1);
        push_all(); wait_vm<0>(); push_flag();
    }
    STAMP(4);

    const char* gA = wsl + abuf + (size_t)pm * PANEL_B + wid * 8192;
    const char* gB = wsl + WS_B + (size_t)pn * PANEL_B + wid * 8192;
    const char* fA = wsl + faoff + (size_t)pm * 4096;
    const char* fB = wsl + WS_FLAGB + (size_t)(pn * 4 + wid) * 512;
    int RA = 0, RB = 0, exa = 0;
    int plim = 0;
    int qa = 0, qb = 0;
    int tsend = 0;
    int evt = 0;
    int nslow = 0;
    const unsigned call4 = call * 4u;
    auto pollA = [&](int last) __attribute__((always_inline)) { qa = RA; gload16_sys<R_PA>((unsigned)min(qa + (lane >> 2), last) * 64u + (lane & 3) * 16u, fA); };
    auto pollB = [&](int last) __attribute__((always_inline)) { qb = RB; gload8_agent<R_PB>((unsigned)min(qb + fr, last) * 8u, fB); };
    auto digestA = [&](int lim) __attribute__((always_inline)) { const unsigned long long nm = ~ready_mask<R_PA, R_PA + 2>(call); RA = max(RA, min(qa + (nm ? (int)__builtin_ctzll(nm) >> 2 : 16), lim)); };
    auto digestB = [&](int lim) __attribute__((always_inline)) { RB = max(RB, min(qb + (int)__builtin_ctz(~(unsigned)ready_mask<R_PB, R_PB + 1>(call4) | 0x10000u), lim)); };
    auto limits = [&]() __attribute__((always_inline)) { nslow = min(min(RA >= NT ? 1000 : RA - 3, RB >= NT ? 1000 : min(RB - 2, NT - 4)), evt); };
    auto spin = [&](int na, int nb, int tcur) __attribute__((always_inline)) {
        const unsigned long long t0 = rtclock();
        wait_vm<0>();
        if (pst == 1 || pst == 2) push_flag();
        bool wide = true;
        for (int it = 1; !dead; ++it) {
            if (pst == 0 && (int)((unsigned)rtclock() - due) >= 0) { push_all(); wait_vm<0>(); push_flag(); }
            const int la = wide ? NT - 1 : min(max(na, RA + 1) + 2, NT) - 1, lb = wide ? NT - 1 : min(max(nb, RB + 1) + 4, NT) - 1;
            pollA(la); pollB(lb);
            wait_vm<0>();
            digestA(la + 1); digestB(lb + 1);
            plim = 0;
            if (__builtin_amdgcn_readfirstlane(rdreg<R_PA>()) == call) exa = (int)(short)__builtin_amdgcn_readfirstlane((int)rdreg<R_PA + 1>());
            if (RA >= na && RB >= nb) break;
            wide = false;
            const unsigned long long now = rtclock();
            if ((it & 15) == 0 || now - t0 > DEADLINE_TICKS) dead = give_up(tmo, call, t0);
            __builtin_amdgcn_s_sleep(SPINSLEEP);
        }
        if (pst == 0) { const int rem = (int)(due - (unsigned)rtclock()); tsend = tcur + (rem <= 0 ? 0 : rem / TILE_TICKS); }
    };
    acc_zero_all();
    const unsigned ldsb = (unsigned)(uintptr_t)lds + lane * 16, ldsh = ldsb + 65536;
#define GLDS_A(tile, st, i) __builtin_amdgcn_global_load_lds((const unsigned*)(gA + (size_t)(tile) * TB + ((i) >> 2) * 4096 + voff0), (LAS unsigned*)(lds + (st) * TB + wid * 8192 + ((i) >> 2) * 4096), 16, ((i) & 3) * 1024, 0)
#define LOAD_B(buf, tile, i) gload16<R_B + (buf) * 32 + (i) * 4, ((i) & 3) * 1024>(((i) >> 2) ? voff1 : voff0, gB + (size_t)(tile) * TB)
    sfor<8>([&](auto J) __attribute__((always_inline)) { constexpr int j = J; gload16<R_W + 4 * j, 0>(wvoff, wbase + (size_t)(512 + j) * GN * 4); });
    spin(3, 3, 0);
    sfor<8>([&](auto I) __attribute__((always_inline)) { constexpr int i = I; GLDS_A(0, 0, i); });
    sfor<8>([&](auto I) __attribute__((always_inline)) { constexpr int i = I; LOAD_B(0, 0, i); });
    sfor<8>([&](auto I) __attribute__((always_inline)) { constexpr int i = I; GLDS_A(1, 1, i); });
    sfor<8>([&](auto I) __attribute__((always_inline)) { constexpr int i = I; LOAD_B(1, 1, i); });
    sfor<8>([&](auto I) __attribute__((always_inline)) { constexpr int i = I; GLDS_A(2, 2, i); });
    wait_vm<16>();
    __builtin_amdgcn_s_barrier();
    STAMP(5);
    sfor<APF>([&](auto G) __attribute__((always_inline)) { constexpr int g = G; ldsrd<R_AF + 4 * g, g * 1024>(ldsb, ldsh); });
    evt = pst == 5 ? 1000 : tsend;
    limits();
    float wmax = 0.f; unsigned wfl = 0;
    auto tile = [&](auto PH_, auto LASTP_, const int t) __attribute__((always_inline)) {
        constexpr int PH = PH_; constexpr bool LASTQ = LASTP_, LASTP = LASTQ || ABL_NOW;
        constexpr int cur = PH & 3, PP = (PH + 7) & 7;
        constexpr bool PREV_W = !ABL_NOW && !(LASTP && PH > 0);
        constexpr int e1p = (PREV_W && PP == 6 ? 8 : 0) + (PREV_W && PP == 2 ? 1 : 0), e2p = (PREV_W && PP == 0 ? 4 : 0) + ((PP & 1) ? 2 : 0);
        constexpr int base = 16 + e1p + e2p;
        constexpr bool FIN = LASTQ && PH == 4;
        if (__builtin_expect(t >= nslow, 0)) {
            if (pst == 1) { wait_vm<base + 28>(); pst = 2; } else { wait_vm<base>(); if (pst == 2) push_flag(); }
            const int na = t + 4 < NT ? t + 4 : NT, nb = FIN ? NT : t + 3;
            if (RA < na || RB < nb) spin(na, nb, t);
            if (pst == 0 && t >= tsend) push_all();
            evt = pst == 5 ? 1000 : (pst == 0 ? tsend : t + 1);
            limits();
        } else wait_vm<base>();
        const int slab = (t >> 3) + 1;
        sfor<32>([&](auto G) __attribute__((always_inline)) { constexpr int g = G;
            constexpr int RC = (g & 15) * 16, RAF = R_AF + 4 * (g & 7), RBF = R_B + cur * 32 + (g >> 4) * 16;
            if constexpr (g == 16) __builtin_amdgcn_s_barrier();
            mfma1w<RC, (FIN && g + APF > 32) ? (32 - g - 1) : APF - 1, RAF, RBF>();
            if constexpr (g + APF < 32) ldsrd<R_AF + 4 * ((g + APF) & 7), (PH & 3) * TB + (g + APF) * 1024>(ldsb, ldsh);
            else if constexpr (!FIN) ldsrd<R_AF + 4 * ((g + APF) & 7), ((PH + 1) & 3) * TB + (g + APF - 32) * 1024>(ldsb, ldsh);
            mfma1<RC + 4, RAF, RBF + 4>();
            if constexpr ((PH & 1) == 1 && g == 8) digestA(plim);
            if constexpr ((PH & 1) == 1 && g == 10) digestB(plim);
            if constexpr ((PH & 1) == 1 && g == 12) limits();
            if constexpr (!LASTP && PH == 0 && g >= 8 && g < 16) { if constexpr (g == 8) wmax = 0.f; amax4r<R_W + 4 * (g - 8)>(wmax); }
            if constexpr (!LASTP && PH == 0 && g == 17) { const float a = wmax * sw; wfl = call4 + ((__ballot(!(a < 60000.0f)) ? 1u : 0u) | (__ballot(a >= 0.015625f) ? 2u : 0u)); }
            u32x4 wq;
            if constexpr (!LASTP && PH == 0 && g >= 25 && g < 29) wq = cvt8r<R_W + (g - 25), 4>(sw);
            mfma1<RC + 8, RAF, RBF + 8>();
            if constexpr (g < 8) LOAD_B((cur + 2) & 3, t + 2, g);
            if constexpr (FIN && g >= 8 && g < 16) LOAD_B(3, t + 3, g - 8);
            if constexpr (FIN && g >= 26 && g < 30) gload16_sys<R_W + 4 * (g - 26)>(lane * 16 + (g - 26) * 1024, fA);
            if constexpr (FIN && g == 16) gload8_agent<R_PB>(lane * 8, fB);
            if constexpr (!LASTP && PH == 6 && g >= 8 && g < 16) gload16<R_W + 4 * (g - 8), 0>(wvoff, wbase + ((size_t)min(slab + 1, 7) * 512 + (g - 8)) * GN * 4);
            if constexpr (!LASTP && PH == 2 && g == 9) gstore4_agent(0u, wfl, fbdst + slab * 64);
            if constexpr (g >= 17 && g < 25) GLDS_A(t + 3, (PH + 3) & 3, g - 17);
            if constexpr (!LASTP && PH == 0 && g >= 25 && g < 29) gstore16_agent<(g - 25) * 1024>(voff0, wq, bdst + (size_t)slab * 8 * TB);
            if constexpr ((PH & 1) == 1 && g == 29) { pollA(NT - 1); plim = NT; }
            if constexpr ((PH & 1) == 1 && g == 30) pollB(NT - 1);
            mfma1<RC + 12, RAF, RBF + 12>();
        });
    };
    for (int s = 0; s < 7; ++s) {
        sfor<8>([&](auto PH) __attribute__((always_inline)) { tile(PH, std::false_type{}, s * 8 + PH); });
    }
    sfor<5>([&](auto PH) __attribute__((always_inline)) { tile(PH, std::true_type{}, 56 + PH); });
    if (pst == 0) push_all();
    wait_vm<0>();
    if (pst != 5) push_flag();
    __builtin_amdgcn_s_barrier();
    STAMP(8);
    bool wbad;
    { const unsigned o = rdreg<R_PB>() | rdreg<R_PB + 1>(); wbad = __ballot(o & 1u) != 0ull || __ballot(o & 2u) == 0ull;
      unsigned xo = 0; sfor<8>([&](auto I) __attribute__((always_inline)) { xo |= rdreg<R_W + 2 * I + 1>(); }); wbad = wbad || __ballot(xo & 0x10000u) != 0ull; }
    float fx = exp2i(-exa), fw = exp2i(-ew);
    bool bad = false;
    float* cp = p.out + (size_t)(pm * 256 + fq * 4) * GN + pn * 256 + wid * 64 + fr * 4;
    auto store_rows = [&](auto M_) __attribute__((always_inline)) { constexpr int m = M_;
        sfor<4>([&](auto J) __attribute__((always_inline)) { constexpr int j = J;
            f32x4 v = acc_read_row<m * 16 + j>();
#pragma unroll
            for (int e = 0; e < 4; ++e) { float y = v[e]; y = y < 0.f ? 0.f : y; v[e] = bad ? __builtin_nanf("") : (y * fx) * fw; }
            *(f32x4*)(cp + (size_t)(m * 16 + j) * GN) = v;
        });
    };
    if (!wbad) {
        bad = dead || load_word(tmo) == call;
        sfor<APF>([&](auto I) __attribute__((always_inline)) { constexpr int i = I, m = i / 6, q = i % 6; ldsrd<R_AF + 4 * (i & 7), (1 + (q >> 1)) * TB + ((q & 1) * 16 + m) * 1024>(ldsb, ldsh); });
        sfor<96>([&](auto I) __attribute__((always_inline)) { constexpr int i = I, m = i / 6, q = i % 6, i2 = i + APF, m2 = i2 / 6, q2 = i2 % 6;
            if constexpr (i2 < 96) ldsrd<R_AF + 4 * (i2 & 7), (1 + (q2 >> 1)) * TB + ((q2 & 1) * 16 + m2) * 1024>(ldsb, ldsh);
            mfma_group<m * 16, (i2 < 96) ? APF : (95 - i), R_AF + 4 * (i & 7), R_B + (1 + (q >> 1)) * 32 + (q & 1) * 16>();
            if constexpr (q == 1 && m > 0) store_rows(std::integral_constant<int, m - 1>{});
        });
        asm volatile("s_nop 15\n\ts_nop 15" ::: "memory");
        store_rows(std::integral_constant<int, 15>{});
    } else {
        bad = dead || load_word(tmo) == call;
        asm volatile("s_nop 15\n\ts_nop 15" ::: "memory");
        acc_zero_all();
        const _Float16* ax = (const _Float16*)(wsl + abuf + (size_t)pm * PANEL_B);
        const float* x32 = (const float*)(wsl + WS_X32 + (size_t)(call & 1u) * 2 * OPER_B) + (size_t)(pm * 256 + fr) * GK;
        const unsigned* fl = (const unsigned*)fA;
        const float* wq = p.w + pn * 256 + wid * 64 + fr * 4;
        for (int k0 = 0; k0 < GK; k0 += 4) {
            const int k = k0 + fq;
            const f32x4 b = *(const f32x4*)(wq + (size_t)k * GN);
            const _Float16* ap = ax + (size_t)(k >> 5) * 8192 + (size_t)(((k >> 3) & 3) * 16 + fr) * 8 + (k & 7);
            sfor<4>([&](auto Q_) __attribute__((always_inline)) { constexpr int q = Q_;
                const bool f32src = (__hip_atomic_load(fl + ((k0 >> 5) * 4 + q) * 2 + 1, __ATOMIC_RELAXED, __HIP_MEMORY_SCOPE_SYSTEM) & 0x10000u) != 0u;
                sfor<4>([&](auto M_) __attribute__((always_inline)) { constexpr int m = q * 4 + M_;
                    const float a = f32src ? __hip_atomic_load(x32 + (size_t)m * 16 * GK + k, __ATOMIC_RELAXED, __HIP_MEMORY_SCOPE_SYSTEM) : (float)ap[m * 512] * fx;
                    mfma_f32<m * 16>(a, b[0]); mfma_f32<m * 16 + 4>(a, b[1]); mfma_f32<m * 16 + 8>(a, b[2]); mfma_f32<m * 16 + 12>(a, b[3]);
                });
            });
        }
        fx = 1.0f; fw = 1.0f;
        asm volatile("s_nop 15\n\ts_nop 15" ::: "memory");
        sfor<16>([&](auto M_) __attribute__((always_inline)) { store_rows(M_); });
    }
    asm volatile(".Lagk_end:" ::: "memory");
    STAMP(9);
}

}

extern "C" void kernel_launch(int rank, int nranks, ncclComm_t comm,
                              hipStream_t stream,
                              void* const* d_in, const int* in_sizes, int n_in,
                              void* d_out, int out_size,
                              const int* in_dtypes, int out_dtype,
                              void* const* d_out_all,
                              void* const* d_ws_all, size_t ws_bytes) {
    if (nranks != WORLD || rank < 0 || rank >= WORLD || ws_bytes < WS_END || n_in < 2) { fprintf(stderr, "unsupported configuration\n"); return; }
    static bool init[WORLD] = {};
    if (!init[rank]) {
        HIP_CHECK(hipFuncSetAttribute((const void*)fused_kernel, hipFuncAttributeMaxDynamicSharedMemorySize, LDS_BYTES));
        init[rank] = true;
    }
    Params p;
    p.x = (const float*)d_in[0]; p.w = (const float*)d_in[1]; p.out = (float*)d_out; p.rank = rank;
    for (int i = 0; i < WORLD; ++i) p.ws[i] = (char*)d_ws_all[(rank + i) & 7];
    hipLaunchKernelGGL(fused_kernel, dim3(GRID), dim3(NTHREADS), LDS_BYTES, stream, p);
    HIP_CHECK(hipGetLastError());
}
```

```cpp
#include <hip/hip_runtime.h>
#include <rccl/rccl.h>
#include <cstdio>
#include <cstdint>
#include <utility>

#define HIP_CHECK(expr) do { hipError_t _e = (expr); if (_e != hipSuccess) { \
    fprintf(stderr, "HIP error %s at %s:%d\n", hipGetErrorString(_e), __FILE__, __LINE__); \
    return; } } while (0)

namespace {

template <int... Is, class F> __device__ __forceinline__ void sfor_impl(std::integer_sequence<int, Is...>, F&& f) { (f(std::integral_constant<int, Is>{}), ...); }
template <int N, class F> __device__ __forceinline__ void sfor(F&& f) { sfor_impl(std::make_integer_sequence<int, N>{}, f); }

typedef _Float16 half8 __attribute__((ext_vector_type(8)));
typedef float f32x4 __attribute__((ext_vector_type(4)));
typedef unsigned u32x4 __attribute__((ext_vector_type(4)));
typedef unsigned u32x2 __attribute__((ext_vector_type(2)));
#define LAS __attribute__((address_space(3)))

constexpr int WORLD = 8;
constexpr int GM = 4096, GK = 4096, GN = 4096;
constexpr int MS = GM / WORLD;
constexpr int NT = GK / 64;
constexpr int TB = 32768;
constexpr size_t PANEL_B = (size_t)NT * TB;
constexpr size_t OPER_B = 16 * PANEL_B;
constexpr size_t WS_CTL = 0;
constexpr size_t WS_SLOT = 4096;
constexpr size_t WS_FLAGA = 65536;
constexpr size_t WS_FLAGB = 262144;
constexpr size_t WS_DBG = 512u << 10;
constexpr size_t WS_A0 = 1u << 20;
constexpr size_t WS_B = WS_A0 + 2 * OPER_B;
constexpr size_t WS_X32 = WS_B + OPER_B;
constexpr size_t WS_END = WS_X32 + 4 * OPER_B;
constexpr int CW_TICKET = 0, CW_TMO = 256;
constexpr unsigned long long DEADLINE_TICKS = 50000000ull;
constexpr int LDS_X = 4 * TB;
constexpr int LDS_BYTES = 4 * TB + 16384;
constexpr int GRID = 256;
#define NTHREADS 256
#define PACE 72
#define PWIN 16
#define KNOW 6
#define K1 24
#define P1 56
#define POFF 0
#define KEARLY 8
#define KBURST 128
#define TILE_TICKS 140
#define SLOPE 43
#define OFFL 230
#define OFFR 440
#define ABL_NOPOLL 0
#define ABL_NOW 0
#define ABL_NOPUSH 0
#define SPINSLEEP 8
#define POLLP 1
#define PAHEAD 10
#define PBHEAD 7
#define APF 5

struct Params { const float* x; const float* w; float* out; char* ws[WORLD]; int rank; };

template <class T> __device__ __forceinline__ T* uptr(T* q) {
    unsigned long long v = (unsigned long long)q;
    unsigned lo = __builtin_amdgcn_readfirstlane((unsigned)v), hi = __builtin_amdgcn_readfirstlane((unsigned)(v >> 32));
    return (T*)(((unsigned long long)hi << 32) | lo);
}
__device__ __forceinline__ u32x4 cvt8(const f32x4 a, const f32x4 b, float s) {
    half8 h;
    h[0] = (_Float16)(a[0] * s); h[1] = (_Float16)(a[1] * s); h[2] = (_Float16)(a[2] * s); h[3] = (_Float16)(a[3] * s);
    h[4] = (_Float16)(b[0] * s); h[5] = (_Float16)(b[1] * s); h[6] = (_Float16)(b[2] * s); h[7] = (_Float16)(b[3] * s);
    return __builtin_bit_cast(u32x4, h);
}
__device__ __forceinline__ unsigned absbits(float v) { return __float_as_uint(v) & 0x7fffffffu; }
__device__ __forceinline__ unsigned amax4(const f32x4 a) { return max(max(absbits(a[0]), absbits(a[1])), max(absbits(a[2]), absbits(a[3]))); }
__device__ __forceinline__ unsigned wave_umax(unsigned v) {
    v = max(v, (unsigned)__builtin_amdgcn_update_dpp(0, (int)v, 0x111, 0xf, 0xf, false));
    v = max(v, (unsigned)__builtin_amdgcn_update_dpp(0, (int)v, 0x112, 0xf, 0xf, false));
    v = max(v, (unsigned)__builtin_amdgcn_update_dpp(0, (int)v, 0x114, 0xf, 0xf, false));
    v = max(v, (unsigned)__builtin_amdgcn_update_dpp(0, (int)v, 0x118, 0xf, 0xf, false));
    v = max(v, (unsigned)__builtin_amdgcn_update_dpp(0, (int)v, 0x142, 0xa, 0xf, false));
    v = max(v, (unsigned)__builtin_amdgcn_update_dpp(0, (int)v, 0x143, 0xc, 0xf, false));
    return (unsigned)__builtin_amdgcn_readlane((int)v, 63);
}
__device__ __forceinline__ int scale_exp(unsigned maxbits, int target) {
    const int ef = (int)(maxbits >> 23);
    if (ef == 0 || ef == 255) return 0;
    int e = target - (ef - 127);
    return e > 126 ? 126 : (e < -126 ? -126 : e);
}
__device__ __forceinline__ float exp2i(int e) { return __uint_as_float((unsigned)(e + 127) << 23); }

constexpr int R_PB = 58, R_PA = 60, R_W = 64, R_AF = 96, R_B = 128, R_X = 128, R_S = 96, R_SL = 112, R_W2 = 160;
#define OWNED_REGS "a0","a1","a2","a3","a4","a5","a6","a7","a8","a9","a10","a11","a12","a13","a14","a15","a16","a17","a18","a19","a20","a21","a22","a23","a24","a25","a26","a27","a28","a29","a30","a31","a32","a33","a34","a35","a36","a37","a38","a39","a40","a41","a42","a43","a44","a45","a46","a47","a48","a49","a50","a51","a52","a53","a54","a55","a56","a57","a58","a59","a60","a61","a62","a63","a64","a65","a66","a67","a68","a69","a70","a71","a72","a73","a74","a75","a76","a77","a78","a79","a80","a81","a82","a83","a84","a85","a86","a87","a88","a89","a90","a91","a92","a93","a94","a95","a96","a97","a98","a99","a100","a101","a102","a103","a104","a105","a106","a107","a108","a109","a110","a111","a112","a113","a114","a115","a116","a117","a118","a119","a120","a121","a122","a123","a124","a125","a126","a127","a128","a129","a130","a131","a132","a133","a134","a135","a136","a137","a138","a139","a140","a141","a142","a143","a144","a145","a146","a147","a148","a149","a150","a151","a152","a153","a154","a155","a156","a157","a158","a159","a160","a161","a162","a163","a164","a165","a166","a167","a168","a169","a170","a171","a172","a173","a174","a175","a176","a177","a178","a179","a180","a181","a182","a183","a184","a185","a186","a187","a188","a189","a190","a191","a192","a193","a194","a195","a196","a197","a198","a199","a200","a201","a202","a203","a204","a205","a206","a207","a208","a209","a210","a211","a212","a213","a214","a215","a216","a217","a218","a219","a220","a221","a222","a223","a224","a225","a226","a227","a228","a229","a230","a231","a232","a233","a234","a235","a236","a237","a238","a239","a240","a241","a242","a243","a244","a245","a246","a247","a248","a249","a250","a251","a252","a253","a254","a255","v58","v59","v60","v61","v62","v63","v64","v65","v66","v67","v68","v69","v70","v71","v72","v73","v74","v75","v76","v77","v78","v79","v80","v81","v82","v83","v84","v85","v86","v87","v88","v89","v90","v91","v92","v93","v94","v95","v96","v97","v98","v99","v100","v101","v102","v103","v104","v105","v106","v107","v108","v109","v110","v111","v112","v113","v114","v115","v116","v117","v118","v119","v120","v121","v122","v123","v124","v125","v126","v127","v128","v129","v130","v131","v132","v133","v134","v135","v136","v137","v138","v139","v140","v141","v142","v143","v144","v145","v146","v147","v148","v149","v150","v151","v152","v153","v154","v155","v156","v157","v158","v159","v160","v161","v162","v163","v164","v165","v166","v167","v168","v169","v170","v171","v172","v173","v174","v175","v176","v177","v178","v179","v180","v181","v182","v183","v184","v185","v186","v187","v188","v189","v190","v191","v192","v193","v194","v195","v196","v197","v198","v199","v200","v201","v202","v203","v204","v205","v206","v207","v208","v209","v210","v211","v212","v213","v214","v215","v216","v217","v218","v219","v220","v221","v222","v223","v224","v225","v226","v227","v228","v229","v230","v231","v232","v233","v234","v235","v236","v237","v238","v239","v240","v241","v242","v243","v244","v245","v246","v247","v248","v249","v250","v251","v252","v253","v254","v255"
#define ALL_AGPRS "a0","a1","a2","a3","a4","a5","a6","a7","a8","a9","a10","a11","a12","a13","a14","a15","a16","a17","a18","a19","a20","a21","a22","a23","a24","a25","a26","a27","a28","a29","a30","a31","a32","a33","a34","a35","a36","a37","a38","a39","a40","a41","a42","a43","a44","a45","a46","a47","a48","a49","a50","a51","a52","a53","a54","a55","a56","a57","a58","a59","a60","a61","a62","a63","a64","a65","a66","a67","a68","a69","a70","a71","a72","a73","a74","a75","a76","a77","a78","a79","a80","a81","a82","a83","a84","a85","a86","a87","a88","a89","a90","a91","a92","a93","a94","a95","a96","a97","a98","a99","a100","a101","a102","a103","a104","a105","a106","a107","a108","a109","a110","a111","a112","a113","a114","a115","a116","a117","a118","a119","a120","a121","a122","a123","a124","a125","a126","a127","a128","a129","a130","a131","a132","a133","a134","a135","a136","a137","a138","a139","a140","a141","a142","a143","a144","a145","a146","a147","a148","a149","a150","a151","a152","a153","a154","a155","a156","a157","a158","a159","a160","a161","a162","a163","a164","a165","a166","a167","a168","a169","a170","a171","a172","a173","a174","a175","a176","a177","a178","a179","a180","a181","a182","a183","a184","a185","a186","a187","a188","a189","a190","a191","a192","a193","a194","a195","a196","a197","a198","a199","a200","a201","a202","a203","a204","a205","a206","a207","a208","a209","a210","a211","a212","a213","a214","a215","a216","a217","a218","a219","a220","a221","a222","a223","a224","a225","a226","a227","a228","a229","a230","a231","a232","a233","a234","a235","a236","a237","a238","a239","a240","a241","a242","a243","a244","a245","a246","a247","a248","a249","a250","a251","a252","a253","a254","a255"
template <int R, int OFF> __device__ __forceinline__ void gload16(unsigned voff, const char* sbase) {
    asm volatile("global_load_dwordx4 v[%c2:%c2+3], %0, %1 offset:%c3" :: "v"(voff), "s"(sbase), "i"(R), "i"(OFF) : "memory");
}
template <int R> __device__ __forceinline__ void gload16_sys(unsigned voff, const char* sbase) {
    asm volatile("global_load_dwordx4 v[%c2:%c2+3], %0, %1 sc0 sc1" :: "v"(voff), "s"(sbase), "i"(R) : "memory");
}
template <int R> __device__ __forceinline__ void gload16_agent(unsigned voff, const char* sbase) {
    asm volatile("global_load_dwordx4 v[%c2:%c2+3], %0, %1 sc1" :: "v"(voff), "s"(sbase), "i"(R) : "memory");
}
template <int R> __device__ __forceinline__ void gload8_agent(unsigned voff, const char* sbase) {
    asm volatile("global_load_dwordx2 v[%c2:%c2+1], %0, %1 sc1" :: "v"(voff), "s"(sbase), "i"(R) : "memory");
}
template <int R> __device__ __forceinline__ unsigned rdreg() { unsigned v; asm volatile("v_mov_b32 %0, v%c1" : "=v"(v) : "i"(R)); return v; }
template <int OFF> __device__ __forceinline__ void gstore16_sys(unsigned voff, const u32x4& v, char* sbase) {
    asm volatile("global_store_dwordx4 %0, %1, %2 offset:%3 sc0 sc1\n\ts_nop 1" :: "v"(voff), "v"(v), "s"(sbase), "i"(OFF) : "memory");
}
template <int OFF> __device__ __forceinline__ void gstore16_agent(unsigned voff, const u32x4& v, char* sbase) {
    asm volatile("global_store_dwordx4 %0, %1, %2 offset:%3 sc1\n\ts_nop 1" :: "v"(voff), "v"(v), "s"(sbase), "i"(OFF) : "memory");
}
__device__ __forceinline__ void gstore64_sys_lane(unsigned voff, const u32x4& v0, const u32x4& v1, const u32x4& v2, const u32x4& v3, unsigned plo, unsigned phi, int ln) {
    const unsigned long long b = ((unsigned long long)(unsigned)__builtin_amdgcn_readlane((int)phi, ln) << 32) | (unsigned)__builtin_amdgcn_readlane((int)plo, ln);
    asm volatile("s_nop 4\n\tglobal_store_dwordx4 %0, %1, %5 sc0 sc1\n\tglobal_store_dwordx4 %0, %2, %5 offset:1024 sc0 sc1\n\t"
                 "global_store_dwordx4 %0, %3, %5 offset:2048 sc0 sc1\n\tglobal_store_dwordx4 %0, %4, %5 offset:3072 sc0 sc1\n\ts_nop 1"
                 :: "v"(voff), "v"(v0), "v"(v1), "v"(v2), "v"(v3), "s"(b) : "memory");
}
template <int R, int OFF> __device__ __forceinline__ void storeown_sys(unsigned voff, char* sbase) {
    asm volatile("s_nop 4\n\tglobal_store_dwordx4 %0, v[%c2:%c2+3], %1 offset:%c3 sc0 sc1\n\ts_nop 1" :: "v"(voff), "s"(sbase), "i"(R), "i"(OFF) : "memory");
}
__device__ __forceinline__ void gstore8_sys(char* addr, const u32x2& v) {
    asm volatile("global_store_dwordx2 %0, %1, off sc0 sc1\n\ts_nop 1" :: "v"(addr), "v"(v) : "memory");
}
__device__ __forceinline__ void gstore8_agent(unsigned voff, const u32x2& v, char* sbase) {
    asm volatile("global_store_dwordx2 %0, %1, %2 sc1\n\ts_nop 1" :: "v"(voff), "v"(v), "s"(sbase) : "memory");
}
__device__ __forceinline__ void gstore4_agent(unsigned voff, unsigned v, char* sbase) {
    asm volatile("global_store_dword %0, %1, %2 sc1\n\ts_nop 1" :: "v"(voff), "v"(v), "s"(sbase) : "memory");
}
__device__ __forceinline__ unsigned long long rtclock() {
    unsigned long long t; asm volatile("s_memrealtime %0\n\ts_waitcnt lgkmcnt(0)" : "=s"(t) :: "memory"); return t;
}
__device__ __forceinline__ unsigned load_word(const char* sbase) {
    unsigned v; asm volatile("global_load_dword %0, %1, %2 sc1\n\ts_waitcnt vmcnt(0)" : "=&v"(v) : "v"(0u), "s"(sbase) : "memory");
    return __builtin_amdgcn_readfirstlane(v);
}
__device__ __forceinline__ int give_up(char* tmo, unsigned call, unsigned long long t0) {
    if (load_word(tmo) == call) return 1;
    if (rtclock() - t0 > DEADLINE_TICKS) { gstore4_agent(0u, call, tmo); return 1; }
    __builtin_amdgcn_s_sleep(1);
    return 0;
}
template <int R, int OFF> __device__ __forceinline__ void ldsrd(unsigned lo, unsigned hi) {
    if constexpr (OFF < 65536) asm volatile("ds_read_b128 v[%c1:%c1+3], %0 offset:%c2" :: "v"(lo), "i"(R), "i"(OFF) : "memory");
    else asm volatile("ds_read_b128 v[%c1:%c1+3], %0 offset:%c2" :: "v"(hi), "i"(R), "i"(OFF - 65536) : "memory");
}
template <int OFF> __device__ __forceinline__ void lds_write16(unsigned a, const u32x4& v) { asm volatile("ds_write_b128 %0, %1 offset:%2" :: "v"(a), "v"(v), "i"(OFF) : "memory"); }
template <int OFF> __device__ __forceinline__ u32x4 lds_read16_wait(unsigned a) { u32x4 v; asm volatile("ds_read_b128 %0, %1 offset:%2\n\ts_waitcnt lgkmcnt(0)" : "=&v"(v) : "v"(a), "i"(OFF) : "memory"); return v; }
template <int N> __device__ __forceinline__ void wait_vm() { asm volatile("s_waitcnt vmcnt(%0)" :: "i"(N) : "memory"); }
template <int R> __device__ __forceinline__ void amax4r(float& m) {
    asm volatile("v_max3_f32 %0, |v%c1|, |v[%c1+1]|, %0\n\tv_max3_f32 %0, |v[%c1+2]|, |v[%c1+3]|, %0" : "+v"(m) : "i"(R));
}
template <int R, int N> __device__ __forceinline__ unsigned amax_regs() {
    float m = 0.f;
    sfor<N / 4>([&](auto I) __attribute__((always_inline)) { amax4r<R + 4 * I>(m); });
    return __float_as_uint(m);
}
template <int R, int S> __device__ __forceinline__ u32x4 cvt8r(float s) {
    unsigned a, b, c, d;
    asm volatile("v_fma_mixlo_f16 %0, v%c5, %4, 0\n\tv_fma_mixlo_f16 %1, v[%c5+2*%c6], %4, 0\n\tv_fma_mixlo_f16 %2, v[%c5+4*%c6], %4, 0\n\tv_fma_mixlo_f16 %3, v[%c5+6*%c6], %4, 0\n\t"
                 "v_fma_mixhi_f16 %0, v[%c5+%c6], %4, 0\n\tv_fma_mixhi_f16 %1, v[%c5+3*%c6], %4, 0\n\tv_fma_mixhi_f16 %2, v[%c5+5*%c6], %4, 0\n\tv_fma_mixhi_f16 %3, v[%c5+7*%c6], %4, 0"
                 : "=&v"(a), "=&v"(b), "=&v"(c), "=&v"(d) : "v"(s), "i"(R), "i"(S));
    return (u32x4){a, b, c, d};
}
template <int R, int W, int A, int B> __device__ __forceinline__ void mfma_group() {
    asm volatile("s_waitcnt lgkmcnt(%c0)\n\t"
                 "v_mfma_f32_16x16x32_f16 a[%c1:%c1+3], v[%c2:%c2+3], v[%c3:%c3+3], a[%c1:%c1+3]\n\t"
                 "v_mfma_f32_16x16x32_f16 a[%c1+4:%c1+7], v[%c2:%c2+3], v[%c3+4:%c3+7], a[%c1+4:%c1+7]\n\t"
                 "v_mfma_f32_16x16x32_f16 a[%c1+8:%c1+11], v[%c2:%c2+3], v[%c3+8:%c3+11], a[%c1+8:%c1+11]\n\t"
                 "v_mfma_f32_16x16x32_f16 a[%c1+12:%c1+15], v[%c2:%c2+3], v[%c3+12:%c3+15], a[%c1+12:%c1+15]"
                 :: "i"(W), "i"(R), "i"(A), "i"(B) : "memory", ALL_AGPRS);
}
template <int R, int A, int B> __device__ __forceinline__ void mfma1() {
    asm volatile("v_mfma_f32_16x16x32_f16 a[%c0:%c0+3], v[%c1:%c1+3], v[%c2:%c2+3], a[%c0:%c0+3]" :: "i"(R), "i"(A), "i"(B) : "memory");
}
template <int R, int W, int A, int B> __device__ __forceinline__ void mfma1w() {
    asm volatile("s_waitcnt lgkmcnt(%c0)\n\tv_mfma_f32_16x16x32_f16 a[%c1:%c1+3], v[%c2:%c2+3], v[%c3:%c3+3], a[%c1:%c1+3]" :: "i"(W), "i"(R), "i"(A), "i"(B) : "memory", ALL_AGPRS);
}
template <int R0, int R1> __device__ __forceinline__ unsigned long long ready_mask(unsigned thr) {
    unsigned long long m; unsigned t;
    asm volatile("v_min_u32 %1, v%c3, v%c4\n\tv_cmp_le_u32 %0, %2, %1" : "=s"(m), "=&v"(t) : "s"(thr), "i"(R0), "i"(R1));
    return m;
}
template <int R> __device__ __forceinline__ void acc_zero4() {
    asm volatile("v_accvgpr_write_b32 a%c0, 0\n\tv_accvgpr_write_b32 a[%c0+1], 0\n\tv_accvgpr_write_b32 a[%c0+2], 0\n\tv_accvgpr_write_b32 a[%c0+3], 0" :: "i"(R) : ALL_AGPRS);
}
template <int R> __device__ __forceinline__ void acc_zero16(const u32x4& z) {
    asm volatile("v_mfma_f32_32x32x16_f16 a[%c1:%c1+15], %0, %0, 0" :: "v"(z), "i"(R) : ALL_AGPRS);
}
__device__ __forceinline__ void acc_zero_all() {
    u32x4 z = {0u, 0u, 0u, 0u}; asm volatile("s_nop 1" : "+v"(z));
    sfor<16>([&](auto I) __attribute__((always_inline)) { acc_zero16<I * 16>(z); });
    asm volatile("s_nop 15\n\ts_nop 3" ::: "memory");
}
template <int R> __device__ __forceinline__ f32x4 acc_read_row() {
    float a, b, c, d;
    asm volatile("v_accvgpr_read_b32 %0, a%c4\n\tv_accvgpr_read_b32 %1, a[%c4+4]\n\tv_accvgpr_read_b32 %2, a[%c4+8]\n\tv_accvgpr_read_b32 %3, a[%c4+12]"
                 : "=v"(a), "=v"(b), "=v"(c), "=v"(d) : "i"(R));
    return (f32x4){a, b, c, d};
}
template <int R> __device__ __forceinline__ void mfma_f32(float a, float b) {
    asm volatile("s_nop 1\n\tv_mfma_f32_16x16x4_f32 a[%c2:%c2+3], %0, %1, a[%c2:%c2+3]" :: "v"(a), "v"(b), "i"(R) : ALL_AGPRS);
}

#define STAMP(i) do {} while (0)

__global__ __launch_bounds__(256, 1) __attribute__((amdgpu_num_vgpr(58))) void fused_kernel(Params p) {
    extern __shared__ __attribute__((aligned(16))) unsigned char smem[];
    LAS unsigned char* lds = (LAS unsigned char*)smem;
    const int tid = threadIdx.x, wid = __builtin_amdgcn_readfirstlane(tid >> 6), lane = tid & 63;
    const int fr = lane & 15, fq = lane >> 4;
    const int rank = p.rank;
    char* wsl = uptr(p.ws[0]);
    unsigned* ctl = (unsigned*)(wsl + WS_CTL);
    char* tmo = wsl + WS_CTL + CW_TMO * 4;
    asm volatile("" ::: OWNED_REGS);
    STAMP(0);
    const int bid = blockIdx.x, xcd = bid & 7, idx = bid >> 3;
    const int pm = (idx >> 2) * 2 + (xcd & 1), pn = (xcd >> 1) * 4 + (idx & 3);
    const bool local = (pm >> 1) == rank;
    const int jwg = local ? 224 + (idx & 3) * 8 + xcd : (((idx >> 2) < rank ? (idx >> 2) : (idx >> 2) - 1) * 4 + (idx & 3)) * 8 + xcd;
    const int unit = (jwg >> 2) + 64 * wid, uq = jwg & 3;
    const int ukh = unit >> 1, upl = unit & 1, upm = rank * 2 + upl;
    const bool burst = ukh < KBURST;

    const unsigned voff0 = lane * 16, voff1 = lane * 16 + 4096;
    unsigned ticket = 0;
    if (lane == 0) asm volatile("global_atomic_add %0, %1, %2, %3 sc0" : "=v"(ticket) : "v"(0u), "v"(1u), "s"(wsl + WS_CTL + CW_TICKET * 4 + wid * 128) : "memory");
    const char* xb = (const char*)uptr(p.x) + ((size_t)(upl * 256 + uq * 64) * GK + ukh * 32) * 4;
    const unsigned xv = (unsigned)(fr * GK + fq * 8) * 4u;
    sfor<4>([&](auto I) __attribute__((always_inline)) { constexpr int i = I; gload16<R_S + 16 + 4 * i, 0>((unsigned)((lane * 8 + i * 2 + 1) * GK + ((lane * 68 + i * 1028) & (GK - 1))) * 4u, (const char*)uptr(p.x)); });
    sfor<4>([&](auto I) __attribute__((always_inline)) { constexpr int i = I; gload16<R_X + 8 * i, 0>(xv, xb + (size_t)i * 16 * GK * 4); gload16<R_X + 8 * i + 4, 16>(xv, xb + (size_t)i * 16 * GK * 4); });
    const unsigned wvoff = (unsigned)(fq * 8 * GN + fr * 4) * 4u;
    const char* wbase = (const char*)uptr(p.w) + ((size_t)pm * 32 * GN + pn * 256 + wid * 64) * 4;
    sfor<4>([&](auto I) __attribute__((always_inline)) { constexpr int i = I; gload16<R_S + 4 * i, 0>((unsigned)((lane * 64 + i * 16 + 5) * GN + ((lane * 68 + i * 1028) & (GN - 1))) * 4u, (const char*)uptr(p.w)); });
    sfor<8>([&](auto J) __attribute__((always_inline)) { constexpr int j = J; gload16<R_W + 4 * j, 0>(wvoff, wbase + (size_t)j * GN * 4); });
    {
        unsigned long long pc; unsigned sz;
        asm volatile("s_getpc_b64 %0\n.Lagk_begin:\n\ts_mov_b32 %1, .Lagk_end-.Lagk_begin" : "=s"(pc), "=s"(sz));
        gload16<R_B + 124, 0>(min((unsigned)(idx * 4 + wid) * 1024u, sz - 1024u) + lane * 16, (const char*)pc);
    }
    const int r0 = (unit * 4 + uq * 2) % 7;
    char* fdst = wsl;
#pragma unroll
    for (int i = 1; i < WORLD; ++i) { char* q = p.ws[i]; fdst = (lane & 7) == i ? q : fdst; }
    STAMP(15);
    asm volatile("s_waitcnt vmcnt(13)" : "+v"(ticket) :: "memory");
    STAMP(14);
    const unsigned call = __builtin_amdgcn_readfirstlane(ticket) / GRID + 1u;
    const size_t abuf = WS_A0 + (size_t)(call & 1u) * OPER_B;
    const size_t faoff = WS_FLAGA + (size_t)(call & 1u) * 65536;
    int dead = 0;
    const int ex = scale_exp(wave_umax(amax_regs<R_S + 16, 16>()), 4);
    const unsigned tx0 = (unsigned)rtclock();
    STAMP(3);
    u32x4 xq[4];
    bool xbad;
    {
        const float sx = exp2i(ex);
        sfor<4>([&](auto I) __attribute__((always_inline)) { constexpr int i = I; xq[i] = cvt8r<R_X + 8 * i, 1>(sx); });
        const float a = __uint_as_float(amax_regs<R_X, 32>()) * sx;
        xbad = __ballot(!(a < 60000.0f)) != 0ull || (__ballot(a >= 0.000244140625f) == 0ull && __ballot(a != 0.0f) != 0ull);
    }
    const unsigned xoff = (unsigned)(abuf + (size_t)upm * PANEL_B + (size_t)ukh * 16384 + uq * 4096 + lane * 16);
    char* fladdr = fdst + faoff + (((size_t)upm * 128 + ukh) * 4 + uq) * 8;
    const u32x2 flval = {call, (unsigned)(ex & 0xffff) | (xbad ? 0x10000u : 0u)};
    const unsigned plo = (unsigned)(unsigned long long)fdst, phi = (unsigned)((unsigned long long)fdst >> 32);
    const unsigned xlds = (unsigned)(uintptr_t)lds + LDS_X + wid * 4096 + lane * 16;
    sfor<4>([&](auto I) __attribute__((always_inline)) { constexpr int i = I; lds_write16<i * 1024>(xlds, xq[i]); });
    int pst = 0;
    auto push_all = [&]() __attribute__((always_inline)) {
        STAMP(11);
        const u32x4 v0 = lds_read16_wait<0>(xlds), v1 = lds_read16_wait<1024>(xlds), v2 = lds_read16_wait<2048>(xlds), v3 = lds_read16_wait<3072>(xlds);
        int q = r0;
#pragma unroll 1
        for (int d = 0; d < 7; ++d) {
            gstore64_sys_lane(xoff, v0, v1, v2, v3, plo, phi, q + 1);
            q = q == 6 ? 0 : q + 1;
        }
        pst = 1;
    };
    auto push_flag = [&]() __attribute__((always_inline)) { if (lane >= 1 && lane < 8) gstore8_sys(fladdr, flval); pst = 5; STAMP(12); };
    const unsigned due = tx0 + (unsigned)(ukh < KNOW ? 0 : ukh * PACE);
    if (__builtin_expect(xbad, 0)) {
        const unsigned so = (unsigned)(WS_X32 + (size_t)(call & 1u) * 2 * OPER_B + ((size_t)(upm * 256 + uq * 64) * GK + ukh * 32) * 4) + xv;
#pragma unroll 1
        for (int d = 0; d < WORLD; ++d) {
            const unsigned long long bq = ((unsigned long long)(unsigned)__builtin_amdgcn_readlane((int)phi, d) << 32) | (unsigned)__builtin_amdgcn_readlane((int)plo, d);
            sfor<8>([&](auto I) __attribute__((always_inline)) { constexpr int i = I; storeown_sys<R_X + 4 * i, (i & 1) * 16>(so + (unsigned)(i >> 1) * 16u * GK * 4u, (char*)bq); });
        }
        wait_vm<0>();
    }
    if (ukh < KNOW) push_all();
    sfor<4>([&](auto I) __attribute__((always_inline)) { constexpr int i = I; gstore16_sys<i * 1024>(xoff, xq[i], wsl); });
    sfor<8>([&](auto J) __attribute__((always_inline)) { constexpr int j = J; gload16<R_W2 + 4 * j, 0>(wvoff, wbase + (size_t)(512 + j) * GN * 4); });
    if (ukh < KNOW) wait_vm<41>(); else wait_vm<13>();
    const int ew = scale_exp(wave_umax(amax_regs<R_S, 16>()), 0);
    const float sw = exp2i(ew);
    char* bdst = wsl + WS_B + (size_t)pn * PANEL_B + (size_t)(pm >> 1) * TB + wid * 8192 + (pm & 1) * 4096;
    char* fbdst = wsl + WS_FLAGB + ((size_t)(pn * 4 + wid) * 128 + pm) * 4;
    auto wbits = [&]() __attribute__((always_inline)) -> unsigned {
        const float a = __uint_as_float(amax_regs<R_W, 32>()) * sw;
        return call * 4u + ((__ballot(!(a < 60000.0f)) ? 1u : 0u) | (__ballot(a >= 0.015625f) ? 2u : 0u));
    };
    auto wstore = [&](auto C, char* dst) __attribute__((always_inline)) { constexpr int c = C; gstore16_agent<c * 1024>(voff0, cvt8r<R_W + c, 4>(sw), dst); };
    sfor<4>([&](auto C) __attribute__((always_inline)) { wstore(C, bdst); });
    const unsigned fb0 = wbits();
    wait_vm<0>();
    gstore4_agent(0u, fb0, fbdst);
    if (lane == 0) gstore8_sys(fladdr, flval);
    if (pst == 1) push_flag();
    if (pst == 0 && ukh < KEARLY) {
        while ((int)((unsigned)rtclock() - due) < 0) __builtin_amdgcn_s_sleep(1);
        push_all(); wait_vm<0>(); push_flag();
    }
    {
        sfor<4>([&](auto C) __attribute__((always_inline)) { constexpr int c = C; gstore16_agent<c * 1024>(voff0, cvt8r<R_W2 + c, 4>(sw), bdst + 8 * TB); });
        const float a = __uint_as_float(amax_regs<R_W2, 32>()) * sw;
        const unsigned f1 = call * 4u + ((__ballot(!(a < 60000.0f)) ? 1u : 0u) | (__ballot(a >= 0.015625f) ? 2u : 0u));
        wait_vm<0>();
        gstore4_agent(64u, f1, fbdst);
    }
    STAMP(4);

    const char* gA = wsl + abuf + (size_t)pm * PANEL_B + wid * 8192;
    const char* gB = wsl + WS_B + (size_t)pn * PANEL_B + wid * 8192;
    const char* fA = wsl + faoff + (size_t)pm * 4096;
    const char* fB = wsl + WS_FLAGB + (size_t)(pn * 4 + wid) * 512;
    int RA = 0, RB = 0, exa = 0;
    int plim = 0;
    int qa = 0, qb = 0;
    int tsend = 0;
    int evt = 0;
    int nslow = 0;
    const unsigned call4 = call * 4u;
    auto pollA = [&](int last) __attribute__((always_inline)) { qa = RA; last = min(last, qa + PWIN - 1);
        gload16_sys<R_PA>((unsigned)min(qa + (lane >> 2), last) * 64u + (lane & 3) * 16u, fA);
    };
    auto pollB = [&](int last) __attribute__((always_inline)) { qb = RB; gload8_agent<R_PB>((unsigned)min(qb + fr, last) * 8u, fB); };
    auto digestA = [&](int lim) __attribute__((always_inline)) { const unsigned long long nm = ~ready_mask<R_PA, R_PA + 2>(call); RA = max(RA, min(qa + min(nm ? (int)__builtin_ctzll(nm) >> 2 : 16, PWIN), lim)); };
    auto digestB = [&](int lim) __attribute__((always_inline)) { RB = max(RB, min(qb + (int)__builtin_ctz(~(unsigned)ready_mask<R_PB, R_PB + 1>(call4) | 0x10000u), lim)); };
    auto limits = [&]() __attribute__((always_inline)) { nslow = min(min(RA >= NT ? 1000 : RA - 3, RB >= NT ? 1000 : min(RB - 2, NT - 4)), evt); };
    auto retime = [&](int tcur) __attribute__((always_inline)) {
        const int rem = (int)(due - (unsigned)rtclock());
        tsend = rem <= TILE_TICKS / 2 ? tcur : tcur + max(1, (rem * 7) / (8 * TILE_TICKS));
    };
    auto spin = [&](int na, int nb, int tcur) __attribute__((always_inline)) {
        const unsigned long long t0 = rtclock();
        wait_vm<0>();
        if (pst == 1 || pst == 2) push_flag();
        bool wide = true;
        for (int it = 1; !dead; ++it) {
            if (pst == 0 && (int)((unsigned)rtclock() - due) >= 0) { push_all(); wait_vm<0>(); push_flag(); }
            const int la = wide ? NT - 1 : min(max(na, RA + 1) + 2, NT) - 1, lb = wide ? NT - 1 : min(max(nb, RB + 1) + 4, NT) - 1;
            pollA(la); pollB(lb);
            wait_vm<0>();
            digestA(la + 1); digestB(lb + 1);
            plim = 0;
            if (__builtin_amdgcn_readfirstlane(rdreg<R_PA>()) == call) exa = (int)(short)__builtin_amdgcn_readfirstlane((int)rdreg<R_PA + 1>());
            if (RA >= na && RB >= nb) break;
            wide = false;
            const unsigned long long now = rtclock();
            if ((it & 15) == 0 || now - t0 > DEADLINE_TICKS) dead = give_up(tmo, call, t0);
            __builtin_amdgcn_s_sleep(SPINSLEEP);
        }
        if (pst == 0) retime(tcur);
    };
    acc_zero_all();
    const unsigned ldsb = (unsigned)(uintptr_t)lds + lane * 16, ldsh = ldsb + 65536;
#define GLDS_A(tile, st, i) __builtin_amdgcn_global_load_lds((const unsigned*)(gA + (size_t)(tile) * TB + ((i) >> 2) * 4096 + voff0), (LAS unsigned*)(lds + (st) * TB + wid * 8192 + ((i) >> 2) * 4096), 16, ((i) & 3) * 1024, 0)
#define LOAD_B(buf, tile, i) gload16<R_B + (buf) * 32 + (i) * 4, ((i) & 3) * 1024>(((i) >> 2) ? voff1 : voff0, gB + (size_t)(tile) * TB)
    sfor<8>([&](auto J) __attribute__((always_inline)) { constexpr int j = J; gload16<R_W + 4 * j, 0>(wvoff, wbase + (size_t)(1024 + j) * GN * 4); });
    spin(3, 3, 0);
    sfor<8>([&](auto I) __attribute__((always_inline)) { constexpr int i = I; GLDS_A(0, 0, i); });
    sfor<8>([&](auto I) __attribute__((always_inline)) { constexpr int i = I; LOAD_B(0, 0, i); });
    sfor<8>([&](auto I) __attribute__((always_inline)) { constexpr int i = I; GLDS_A(1, 1, i); });
    sfor<8>([&](auto I) __attribute__((always_inline)) { constexpr int i = I; LOAD_B(1, 1, i); });
    sfor<8>([&](auto I) __attribute__((always_inline)) { constexpr int i = I; GLDS_A(2, 2, i); });
    wait_vm<16>();
    __builtin_amdgcn_s_barrier();
    STAMP(5);
    sfor<APF>([&](auto G) __attribute__((always_inline)) { constexpr int g = G; ldsrd<R_AF + 4 * g, g * 1024>(ldsb, ldsh); });
    evt = pst == 5 ? 1000 : tsend;
    limits();
    float wmax = 0.f; unsigned wfl = 0;
    auto tile = [&](auto PH_, auto LASTP_, const int t) __attribute__((always_inline)) {
        constexpr int PH = PH_; constexpr bool LASTQ = LASTP_, LASTP = LASTQ || ABL_NOW;
        constexpr int cur = PH & 3, PP = (PH + 7) & 7;
        constexpr bool PREV_W = !ABL_NOW && !(LASTP && PH > 0);
        constexpr int e1p = (PREV_W && PP == 6 ? 8 : 0) + (PREV_W && PP == 2 ? 1 : 0), e2p = (PREV_W && PP == 0 ? 4 : 0) + ((PP & 1) ? 2 : 0);
        constexpr int base = 16 + e1p + e2p;
        constexpr bool FIN = LASTQ && PH == 4;
        if (__builtin_expect(t >= nslow, 0)) {
            if (pst == 1) { wait_vm<base + 28>(); pst = 2; } else { wait_vm<base>(); if (pst == 2) push_flag(); }
            const int na = t + 4 < NT ? t + 4 : NT, nb = FIN ? NT : t + 3;
            if (RA < na || RB < nb) spin(na, nb, t);
            if (pst == 0 && t >= tsend) { retime(t); if (t >= tsend) push_all(); }
            evt = pst == 5 ? 1000 : (pst == 0 ? tsend : t + 1);
            limits();
        } else wait_vm<base>();
        const int slab = min((t >> 3) + 2, 7);
        sfor<32>([&](auto G) __attribute__((always_inline)) { constexpr int g = G;
            constexpr int RC = (g & 15) * 16, RAF = R_AF + 4 * (g & 7), RBF = R_B + cur * 32 + (g >> 4) * 16;
            if constexpr (g == 16) __builtin_amdgcn_s_barrier();
            mfma1w<RC, (FIN && g + APF > 32) ? (32 - g - 1) : APF - 1, RAF, RBF>();
            if constexpr (g + APF < 32) ldsrd<R_AF + 4 * ((g + APF) & 7), (PH & 3) * TB + (g + APF) * 1024>(ldsb, ldsh);
            else if constexpr (!FIN) ldsrd<R_AF + 4 * ((g + APF) & 7), ((PH + 1) & 3) * TB + (g + APF - 32) * 1024>(ldsb, ldsh);
            mfma1<RC + 4, RAF, RBF + 4>();
            if constexpr ((PH & 1) == 1 && g == 8) digestA(plim);
            if constexpr ((PH & 1) == 1 && g == 10) digestB(plim);
            if constexpr ((PH & 1) == 1 && g == 12) limits();
            if constexpr (!LASTP && PH == 0 && g >= 8 && g < 16) { if constexpr (g == 8) wmax = 0.f; amax4r<R_W + 4 * (g - 8)>(wmax); }
            if constexpr (!LASTP && PH == 0 && g == 17) { const float a = wmax * sw; wfl = call4 + ((__ballot(!(a < 60000.0f)) ? 1u : 0u) | (__ballot(a >= 0.015625f) ? 2u : 0u)); }
            u32x4 wq;
            if constexpr (!LASTP && PH == 0 && g >= 25 && g < 29) wq = cvt8r<R_W + (g - 25), 4>(sw);
            mfma1<RC + 8, RAF, RBF + 8>();
            if constexpr (g < 8) LOAD_B((cur + 2) & 3, t + 2, g);
            if constexpr (FIN && g >= 8 && g < 16) LOAD_B(3, t + 3, g - 8);
            if constexpr (FIN && g >= 26 && g < 30) gload16_sys<R_W + 4 * (g - 26)>(lane * 16 + (g - 26) * 1024, fA);
            if constexpr (FIN && g == 16) gload8_agent<R_PB>(lane * 8, fB);
            if constexpr (!LASTP && PH == 6 && g >= 8 && g < 16) gload16<R_W + 4 * (g - 8), 0>(wvoff, wbase + ((size_t)min(slab + 1, 7) * 512 + (g - 8)) * GN * 4);
            if constexpr (!LASTP && PH == 2 && g == 9) gstore4_agent(0u, wfl, fbdst + slab * 64);
            if constexpr (g >= 17 && g < 25) GLDS_A(t + 3, (PH + 3) & 3, g - 17);
            if constexpr (!LASTP && PH == 0 && g >= 25 && g < 29) gstore16_agent<(g - 25) * 1024>(voff0, wq, bdst + (size_t)slab * 8 * TB);
            if constexpr ((PH & 1) == 1 && g == 29) { pollA(NT - 1); plim = NT; }
            if constexpr ((PH & 1) == 1 && g == 30) pollB(NT - 1);
            mfma1<RC + 12, RAF, RBF + 12>();
        });
    };
    for (int s = 0; s < 7; ++s) {
        sfor<8>([&](auto PH) __attribute__((always_inline)) { tile(PH, std::false_type{}, s * 8 + PH); });
    }
    sfor<5>([&](auto PH) __attribute__((always_inline)) { tile(PH, std::true_type{}, 56 + PH); });
    if (pst == 0) push_all();
    wait_vm<0>();
    if (pst != 5) push_flag();
    __builtin_amdgcn_s_barrier();
    STAMP(8);
    bool wbad;
    { const unsigned o = rdreg<R_PB>() | rdreg<R_PB + 1>(); wbad = __ballot(o & 1u) != 0ull || __ballot(o & 2u) == 0ull;
      unsigned xo = 0; sfor<8>([&](auto I) __attribute__((always_inline)) { xo |= rdreg<R_W + 2 * I + 1>(); }); wbad = wbad || __ballot(xo & 0x10000u) != 0ull; }
    float fx = exp2i(-exa), fw = exp2i(-ew);
    bool bad = false;
    float* cp = p.out + (size_t)(pm * 256 + fq * 4) * GN + pn * 256 + wid * 64 + fr * 4;
    auto store_rows = [&](auto M_) __attribute__((always_inline)) { constexpr int m = M_;
        sfor<4>([&](auto J) __attribute__((always_inline)) { constexpr int j = J;
            f32x4 v = acc_read_row<m * 16 + j>();
#pragma unroll
            for (int e = 0; e < 4; ++e) { float y = v[e]; y = y < 0.f ? 0.f : y; v[e] = bad ? __builtin_nanf("") : (y * fx) * fw; }
            __builtin_nontemporal_store(v, (f32x4*)(cp + (size_t)(m * 16 + j) * GN));
        });
    };
    if (!wbad) {
        bad = dead || load_word(tmo) == call;
        sfor<APF>([&](auto I) __attribute__((always_inline)) { constexpr int i = I, m = i / 6, q = i % 6; ldsrd<R_AF + 4 * (i & 7), (1 + (q >> 1)) * TB + ((q & 1) * 16 + m) * 1024>(ldsb, ldsh); });
        sfor<96>([&](auto I) __attribute__((always_inline)) { constexpr int i = I, m = i / 6, q = i % 6, i2 = i + APF, m2 = i2 / 6, q2 = i2 % 6;
            if constexpr (i2 < 96) ldsrd<R_AF + 4 * (i2 & 7), (1 + (q2 >> 1)) * TB + ((q2 & 1) * 16 + m2) * 1024>(ldsb, ldsh);
            mfma_group<m * 16, (i2 < 96) ? APF : (95 - i), R_AF + 4 * (i & 7), R_B + (1 + (q >> 1)) * 32 + (q & 1) * 16>();
            if constexpr (q == 1 && m > 0) store_rows(std::integral_constant<int, m - 1>{});
        });
        asm volatile("s_nop 15\n\ts_nop 15" ::: "memory");
        store_rows(std::integral_constant<int, 15>{});
    } else {
        bad = dead || load_word(tmo) == call;
        asm volatile("s_nop 15\n\ts_nop 15" ::: "memory");
        acc_zero_all();
        const _Float16* ax = (const _Float16*)(wsl + abuf + (size_t)pm * PANEL_B);
        const float* x32 = (const float*)(wsl + WS_X32 + (size_t)(call & 1u) * 2 * OPER_B) + (size_t)(pm * 256 + fr) * GK;
        const unsigned* fl = (const unsigned*)fA;
        const float* wq = p.w + pn * 256 + wid * 64 + fr * 4;
        for (int k0 = 0; k0 < GK; k0 += 4) {
            const int k = k0 + fq;
            const f32x4 b = *(const f32x4*)(wq + (size_t)k * GN);
            const _Float16* ap = ax + (size_t)(k >> 5) * 8192 + (size_t)(((k >> 3) & 3) * 16 + fr) * 8 + (k & 7);
            sfor<4>([&](auto Q_) __attribute__((always_inline)) { constexpr int q = Q_;
                const bool f32src = (__hip_atomic_load(fl + ((k0 >> 5) * 4 + q) * 2 + 1, __ATOMIC_RELAXED, __HIP_MEMORY_SCOPE_SYSTEM) & 0x10000u) != 0u;
                sfor<4>([&](auto M_) __attribute__((always_inline)) { constexpr int m = q * 4 + M_;
                    const float a = f32src ? __hip_atomic_load(x32 + (size_t)m * 16 * GK + k, __ATOMIC_RELAXED, __HIP_MEMORY_SCOPE_SYSTEM) : (float)ap[m * 512] * fx;
                    mfma_f32<m * 16>(a, b[0]); mfma_f32<m * 16 + 4>(a, b[1]); mfma_f32<m * 16 + 8>(a, b[2]); mfma_f32<m * 16 + 12>(a, b[3]);
                });
            });
        }
        fx = 1.0f; fw = 1.0f;
        asm volatile("s_nop 15\n\ts_nop 15" ::: "memory");
        sfor<16>([&](auto M_) __attribute__((always_inline)) { store_rows(M_); });
    }
    asm volatile(".Lagk_end:" ::: "memory");
    STAMP(9);
}

}

extern "C" void kernel_launch(int rank, int nranks, ncclComm_t comm,
                              hipStream_t stream,
                              void* const* d_in, const int* in_sizes, int n_in,
                              void* d_out, int out_size,
                              const int* in_dtypes, int out_dtype,
                              void* const* d_out_all,
                              void* const* d_ws_all, size_t ws_bytes) {
    if (nranks != WORLD || rank < 0 || rank >= WORLD || ws_bytes < WS_END || n_in < 2) { fprintf(stderr, "unsupported configuration\n"); return; }
    static bool init[WORLD] = {};
    if (!init[rank]) {
        HIP_CHECK(hipFuncSetAttribute((const void*)fused_kernel, hipFuncAttributeMaxDynamicSharedMemorySize, LDS_BYTES));
        init[rank] = true;
    }
    Params p;
    p.x = (const float*)d_in[0]; p.w = (const float*)d_in[1]; p.out = (float*)d_out; p.rank = rank;
    for (int i = 0; i < WORLD; ++i) p.ws[i] = (char*)d_ws_all[(rank + i) & 7];
    hipLaunchKernelGGL(fused_kernel, dim3(GRID), dim3(NTHREADS), LDS_BYTES, stream, p);
    HIP_CHECK(hipGetLastError());
}
```

```cpp
#include <hip/hip_runtime.h>
#include <rccl/rccl.h>
#include <cstdio>
#include <cstdint>
#include <utility>

#define HIP_CHECK(expr) do { hipError_t _e = (expr); if (_e != hipSuccess) { \
    fprintf(stderr, "HIP error %s at %s:%d\n", hipGetErrorString(_e), __FILE__, __LINE__); \
    return; } } while (0)

namespace {

template <int... Is, class F> __device__ __forceinline__ void sfor_impl(std::integer_sequence<int, Is...>, F&& f) { (f(std::integral_constant<int, Is>{}), ...); }
template <int N, class F> __device__ __forceinline__ void sfor(F&& f) { sfor_impl(std::make_integer_sequence<int, N>{}, f); }

typedef _Float16 half8 __attribute__((ext_vector_type(8)));
typedef float f32x4 __attribute__((ext_vector_type(4)));
typedef unsigned u32x4 __attribute__((ext_vector_type(4)));
typedef unsigned u32x2 __attribute__((ext_vector_type(2)));
#define LAS __attribute__((address_space(3)))

constexpr int WORLD = 8;
constexpr int GM = 4096, GK = 4096, GN = 4096;
constexpr int MS = GM / WORLD;
constexpr int NT = GK / 64;
constexpr int TB = 32768;
constexpr size_t PANEL_B = (size_t)NT * TB;
constexpr size_t OPER_B = 16 * PANEL_B;
constexpr size_t WS_CTL = 0;
constexpr size_t WS_SLOT = 4096;
constexpr size_t WS_FLAGA = 65536;
constexpr size_t WS_FLAGB = 262144;
constexpr size_t WS_DBG = 512u << 10;
constexpr size_t WS_A0 = 1u << 20;
constexpr size_t WS_B = WS_A0 + 2 * OPER_B;
constexpr size_t WS_X32 = WS_B + OPER_B;
constexpr size_t WS_END = WS_X32 + 4 * OPER_B;
constexpr int CW_TICKET = 0, CW_TMO = 256;
constexpr unsigned long long DEADLINE_TICKS = 50000000ull;
constexpr int LDS_X = 4 * TB;
constexpr int LDS_BYTES = 4 * TB + 16384;
constexpr int GRID = 256;
#define NTHREADS 256
#define PACE 68
#define PWIN 16
#define KNOW 6
#define K1 24
#define P1 56
#define POFF 0
#define SNAKE 1
#define CLOSED 0
#define FLAG1 1
#define KEARLY 8
#define KBURST 128
#define TILE_TICKS 145
#define SLOPE 43
#define OFFL 230
#define OFFR 440
#define ABL_NOPOLL 0
#define ABL_NOW 0
#define ABL_NOPUSH 0
#define SPINSLEEP 8
#define POLLP 1
#define PAHEAD 10
#define PBHEAD 7
#define APF 5

struct Params { const float* x; const float* w; float* out; char* ws[WORLD]; int rank; };

template <class T> __device__ __forceinline__ T* uptr(T* q) {
    unsigned long long v = (unsigned long long)q;
    unsigned lo = __builtin_amdgcn_readfirstlane((unsigned)v), hi = __builtin_amdgcn_readfirstlane((unsigned)(v >> 32));
    return (T*)(((unsigned long long)hi << 32) | lo);
}
__device__ __forceinline__ u32x4 cvt8(const f32x4 a, const f32x4 b, float s) {
    half8 h;
    h[0] = (_Float16)(a[0] * s); h[1] = (_Float16)(a[1] * s); h[2] = (_Float16)(a[2] * s); h[3] = (_Float16)(a[3] * s);
    h[4] = (_Float16)(b[0] * s); h[5] = (_Float16)(b[1] * s); h[6] = (_Float16)(b[2] * s); h[7] = (_Float16)(b[3] * s);
    return __builtin_bit_cast(u32x4, h);
}
__device__ __forceinline__ unsigned absbits(float v) { return __float_as_uint(v) & 0x7fffffffu; }
__device__ __forceinline__ unsigned amax4(const f32x4 a) { return max(max(absbits(a[0]), absbits(a[1])), max(absbits(a[2]), absbits(a[3]))); }
__device__ __forceinline__ unsigned wave_umax(unsigned v) {
    v = max(v, (unsigned)__builtin_amdgcn_update_dpp(0, (int)v, 0x111, 0xf, 0xf, false));
    v = max(v, (unsigned)__builtin_amdgcn_update_dpp(0, (int)v, 0x112, 0xf, 0xf, false));
    v = max(v, (unsigned)__builtin_amdgcn_update_dpp(0, (int)v, 0x114, 0xf, 0xf, false));
    v = max(v, (unsigned)__builtin_amdgcn_update_dpp(0, (int)v, 0x118, 0xf, 0xf, false));
    v = max(v, (unsigned)__builtin_amdgcn_update_dpp(0, (int)v, 0x142, 0xa, 0xf, false));
    v = max(v, (unsigned)__builtin_amdgcn_update_dpp(0, (int)v, 0x143, 0xc, 0xf, false));
    return (unsigned)__builtin_amdgcn_readlane((int)v, 63);
}
__device__ __forceinline__ int scale_exp(unsigned maxbits, int target) {
    const int ef = (int)(maxbits >> 23);
    if (ef == 0 || ef == 255) return 0;
    int e = target - (ef - 127);
    return e > 126 ? 126 : (e < -126 ? -126 : e);
}
__device__ __forceinline__ float exp2i(int e) { return __uint_as_float((unsigned)(e + 127) << 23); }

constexpr int R_PB = 58, R_PA = 60, R_W = 64, R_AF = 96, R_B = 128, R_X = 128, R_S = 96, R_SL = 112, R_W2 = 160;
#define OWNED_REGS "a0","a1","a2","a3","a4","a5","a6","a7","a8","a9","a10","a11","a12","a13","a14","a15","a16","a17","a18","a19","a20","a21","a22","a23","a24","a25","a26","a27","a28","a29","a30","a31","a32","a33","a34","a35","a36","a37","a38","a39","a40","a41","a42","a43","a44","a45","a46","a47","a48","a49","a50","a51","a52","a53","a54","a55","a56","a57","a58","a59","a60","a61","a62","a63","a64","a65","a66","a67","a68","a69","a70","a71","a72","a73","a74","a75","a76","a77","a78","a79","a80","a81","a82","a83","a84","a85","a86","a87","a88","a89","a90","a91","a92","a93","a94","a95","a96","a97","a98","a99","a100","a101","a102","a103","a104","a105","a106","a107","a108","a109","a110","a111","a112","a113","a114","a115","a116","a117","a118","a119","a120","a121","a122","a123","a124","a125","a126","a127","a128","a129","a130","a131","a132","a133","a134","a135","a136","a137","a138","a139","a140","a141","a142","a143","a144","a145","a146","a147","a148","a149","a150","a151","a152","a153","a154","a155","a156","a157","a158","a159","a160","a161","a162","a163","a164","a165","a166","a167","a168","a169","a170","a171","a172","a173","a174","a175","a176","a177","a178","a179","a180","a181","a182","a183","a184","a185","a186","a187","a188","a189","a190","a191","a192","a193","a194","a195","a196","a197","a198","a199","a200","a201","a202","a203","a204","a205","a206","a207","a208","a209","a210","a211","a212","a213","a214","a215","a216","a217","a218","a219","a220","a221","a222","a223","a224","a225","a226","a227","a228","a229","a230","a231","a232","a233","a234","a235","a236","a237","a238","a239","a240","a241","a242","a243","a244","a245","a246","a247","a248","a249","a250","a251","a252","a253","a254","a255","v58","v59","v60","v61","v62","v63","v64","v65","v66","v67","v68","v69","v70","v71","v72","v73","v74","v75","v76","v77","v78","v79","v80","v81","v82","v83","v84","v85","v86","v87","v88","v89","v90","v91","v92","v93","v94","v95","v96","v97","v98","v99","v100","v101","v102","v103","v104","v105","v106","v107","v108","v109","v110","v111","v112","v113","v114","v115","v116","v117","v118","v119","v120","v121","v122","v123","v124","v125","v126","v127","v128","v129","v130","v131","v132","v133","v134","v135","v136","v137","v138","v139","v140","v141","v142","v143","v144","v145","v146","v147","v148","v149","v150","v151","v152","v153","v154","v155","v156","v157","v158","v159","v160","v161","v162","v163","v164","v165","v166","v167","v168","v169","v170","v171","v172","v173","v174","v175","v176","v177","v178","v179","v180","v181","v182","v183","v184","v185","v186","v187","v188","v189","v190","v191","v192","v193","v194","v195","v196","v197","v198","v199","v200","v201","v202","v203","v204","v205","v206","v207","v208","v209","v210","v211","v212","v213","v214","v215","v216","v217","v218","v219","v220","v221","v222","v223","v224","v225","v226","v227","v228","v229","v230","v231","v232","v233","v234","v235","v236","v237","v238","v239","v240","v241","v242","v243","v244","v245","v246","v247","v248","v249","v250","v251","v252","v253","v254","v255"
#define ALL_AGPRS "a0","a1","a2","a3","a4","a5","a6","a7","a8","a9","a10","a11","a12","a13","a14","a15","a16","a17","a18","a19","a20","a21","a22","a23","a24","a25","a26","a27","a28","a29","a30","a31","a32","a33","a34","a35","a36","a37","a38","a39","a40","a41","a42","a43","a44","a45","a46","a47","a48","a49","a50","a51","a52","a53","a54","a55","a56","a57","a58","a59","a60","a61","a62","a63","a64","a65","a66","a67","a68","a69","a70","a71","a72","a73","a74","a75","a76","a77","a78","a79","a80","a81","a82","a83","a84","a85","a86","a87","a88","a89","a90","a91","a92","a93","a94","a95","a96","a97","a98","a99","a100","a101","a102","a103","a104","a105","a106","a107","a108","a109","a110","a111","a112","a113","a114","a115","a116","a117","a118","a119","a120","a121","a122","a123","a124","a125","a126","a127","a128","a129","a130","a131","a132","a133","a134","a135","a136","a137","a138","a139","a140","a141","a142","a143","a144","a145","a146","a147","a148","a149","a150","a151","a152","a153","a154","a155","a156","a157","a158","a159","a160","a161","a162","a163","a164","a165","a166","a167","a168","a169","a170","a171","a172","a173","a174","a175","a176","a177","a178","a179","a180","a181","a182","a183","a184","a185","a186","a187","a188","a189","a190","a191","a192","a193","a194","a195","a196","a197","a198","a199","a200","a201","a202","a203","a204","a205","a206","a207","a208","a209","a210","a211","a212","a213","a214","a215","a216","a217","a218","a219","a220","a221","a222","a223","a224","a225","a226","a227","a228","a229","a230","a231","a232","a233","a234","a235","a236","a237","a238","a239","a240","a241","a242","a243","a244","a245","a246","a247","a248","a249","a250","a251","a252","a253","a254","a255"
template <int R, int OFF> __device__ __forceinline__ void gload16(unsigned voff, const char* sbase) {
    asm volatile("global_load_dwordx4 v[%c2:%c2+3], %0, %1 offset:%c3" :: "v"(voff), "s"(sbase), "i"(R), "i"(OFF) : "memory");
}
template <int R> __device__ __forceinline__ void gload16_sys(unsigned voff, const char* sbase) {
    asm volatile("global_load_dwordx4 v[%c2:%c2+3], %0, %1 sc0 sc1" :: "v"(voff), "s"(sbase), "i"(R) : "memory");
}
template <int R> __device__ __forceinline__ void gload16_agent(unsigned voff, const char* sbase) {
    asm volatile("global_load_dwordx4 v[%c2:%c2+3], %0, %1 sc1" :: "v"(voff), "s"(sbase), "i"(R) : "memory");
}
template <int R> __device__ __forceinline__ void gload8_agent(unsigned voff, const char* sbase) {
    asm volatile("global_load_dwordx2 v[%c2:%c2+1], %0, %1 sc1" :: "v"(voff), "s"(sbase), "i"(R) : "memory");
}
template <int R> __device__ __forceinline__ unsigned rdreg() { unsigned v; asm volatile("v_mov_b32 %0, v%c1" : "=v"(v) : "i"(R)); return v; }
template <int OFF> __device__ __forceinline__ void gstore16_sys(unsigned voff, const u32x4& v, char* sbase) {
    asm volatile("global_store_dwordx4 %0, %1, %2 offset:%3 sc0 sc1\n\ts_nop 1" :: "v"(voff), "v"(v), "s"(sbase), "i"(OFF) : "memory");
}
template <int OFF> __device__ __forceinline__ void gstore16_agent(unsigned voff, const u32x4& v, char* sbase) {
    asm volatile("global_store_dwordx4 %0, %1, %2 offset:%3 sc1\n\ts_nop 1" :: "v"(voff), "v"(v), "s"(sbase), "i"(OFF) : "memory");
}
__device__ __forceinline__ void gstore64_sys_lane(unsigned voff, const u32x4& v0, const u32x4& v1, const u32x4& v2, const u32x4& v3, unsigned plo, unsigned phi, int ln) {
    const unsigned long long b = ((unsigned long long)(unsigned)__builtin_amdgcn_readlane((int)phi, ln) << 32) | (unsigned)__builtin_amdgcn_readlane((int)plo, ln);
    asm volatile("s_nop 4\n\tglobal_store_dwordx4 %0, %1, %5 sc0 sc1\n\tglobal_store_dwordx4 %0, %2, %5 offset:1024 sc0 sc1\n\t"
                 "global_store_dwordx4 %0, %3, %5 offset:2048 sc0 sc1\n\tglobal_store_dwordx4 %0, %4, %5 offset:3072 sc0 sc1\n\ts_nop 1"
                 :: "v"(voff), "v"(v0), "v"(v1), "v"(v2), "v"(v3), "s"(b) : "memory");
}
template <int R, int OFF> __device__ __forceinline__ void storeown_sys(unsigned voff, char* sbase) {
    asm volatile("s_nop 4\n\tglobal_store_dwordx4 %0, v[%c2:%c2+3], %1 offset:%c3 sc0 sc1\n\ts_nop 1" :: "v"(voff), "s"(sbase), "i"(R), "i"(OFF) : "memory");
}
__device__ __forceinline__ void gstore8_sys(char* addr, const u32x2& v) {
    asm volatile("global_store_dwordx2 %0, %1, off sc0 sc1\n\ts_nop 1" :: "v"(addr), "v"(v) : "memory");
}
__device__ __forceinline__ void gstore8_agent(unsigned voff, const u32x2& v, char* sbase) {
    asm volatile("global_store_dwordx2 %0, %1, %2 sc1\n\ts_nop 1" :: "v"(voff), "v"(v), "s"(sbase) : "memory");
}
__device__ __forceinline__ void gstore4_agent(unsigned voff, unsigned v, char* sbase) {
    asm volatile("global_store_dword %0, %1, %2 sc1\n\ts_nop 1" :: "v"(voff), "v"(v), "s"(sbase) : "memory");
}
__device__ __forceinline__ unsigned long long rtclock() {
    unsigned long long t; asm volatile("s_memrealtime %0\n\ts_waitcnt lgkmcnt(0)" : "=s"(t) :: "memory"); return t;
}
__device__ __forceinline__ unsigned load_word(const char* sbase) {
    unsigned v; asm volatile("global_load_dword %0, %1, %2 sc1\n\ts_waitcnt vmcnt(0)" : "=&v"(v) : "v"(0u), "s"(sbase) : "memory");
    return __builtin_amdgcn_readfirstlane(v);
}
__device__ __forceinline__ int give_up(char* tmo, unsigned call, unsigned long long t0) {
    if (load_word(tmo) == call) return 1;
    if (rtclock() - t0 > DEADLINE_TICKS) { gstore4_agent(0u, call, tmo); return 1; }
    __builtin_amdgcn_s_sleep(1);
    return 0;
}
template <int R, int OFF> __device__ __forceinline__ void ldsrd(unsigned lo, unsigned hi) {
    if constexpr (OFF < 65536) asm volatile("ds_read_b128 v[%c1:%c1+3], %0 offset:%c2" :: "v"(lo), "i"(R), "i"(OFF) : "memory");
    else asm volatile("ds_read_b128 v[%c1:%c1+3], %0 offset:%c2" :: "v"(hi), "i"(R), "i"(OFF - 65536) : "memory");
}
template <int OFF> __device__ __forceinline__ void lds_write16(unsigned a, const u32x4& v) { asm volatile("ds_write_b128 %0, %1 offset:%2" :: "v"(a), "v"(v), "i"(OFF) : "memory"); }
template <int OFF> __device__ __forceinline__ u32x4 lds_read16_wait(unsigned a) { u32x4 v; asm volatile("ds_read_b128 %0, %1 offset:%2\n\ts_waitcnt lgkmcnt(0)" : "=&v"(v) : "v"(a), "i"(OFF) : "memory"); return v; }
template <int N> __device__ __forceinline__ void wait_vm() { asm volatile("s_waitcnt vmcnt(%0)" :: "i"(N) : "memory"); }
template <int R> __device__ __forceinline__ void amax4r(float& m) {
    asm volatile("v_max3_f32 %0, |v%c1|, |v[%c1+1]|, %0\n\tv_max3_f32 %0, |v[%c1+2]|, |v[%c1+3]|, %0" : "+v"(m) : "i"(R));
}
template <int R, int N> __device__ __forceinline__ unsigned amax_regs() {
    float m = 0.f;
    sfor<N / 4>([&](auto I) __attribute__((always_inline)) { amax4r<R + 4 * I>(m); });
    return __float_as_uint(m);
}
template <int R, int S> __device__ __forceinline__ u32x4 cvt8r(float s) {
    unsigned a, b, c, d;
    asm volatile("v_fma_mixlo_f16 %0, v%c5, %4, 0\n\tv_fma_mixlo_f16 %1, v[%c5+2*%c6], %4, 0\n\tv_fma_mixlo_f16 %2, v[%c5+4*%c6], %4, 0\n\tv_fma_mixlo_f16 %3, v[%c5+6*%c6], %4, 0\n\t"
                 "v_fma_mixhi_f16 %0, v[%c5+%c6], %4, 0\n\tv_fma_mixhi_f16 %1, v[%c5+3*%c6], %4, 0\n\tv_fma_mixhi_f16 %2, v[%c5+5*%c6], %4, 0\n\tv_fma_mixhi_f16 %3, v[%c5+7*%c6], %4, 0"
                 : "=&v"(a), "=&v"(b), "=&v"(c), "=&v"(d) : "v"(s), "i"(R), "i"(S));
    return (u32x4){a, b, c, d};
}
template <int R, int W, int A, int B> __device__ __forceinline__ void mfma_group() {
    asm volatile("s_waitcnt lgkmcnt(%c0)\n\t"
                 "v_mfma_f32_16x16x32_f16 a[%c1:%c1+3], v[%c2:%c2+3], v[%c3:%c3+3], a[%c1:%c1+3]\n\t"
                 "v_mfma_f32_16x16x32_f16 a[%c1+4:%c1+7], v[%c2:%c2+3], v[%c3+4:%c3+7], a[%c1+4:%c1+7]\n\t"
                 "v_mfma_f32_16x16x32_f16 a[%c1+8:%c1+11], v[%c2:%c2+3], v[%c3+8:%c3+11], a[%c1+8:%c1+11]\n\t"
                 "v_mfma_f32_16x16x32_f16 a[%c1+12:%c1+15], v[%c2:%c2+3], v[%c3+12:%c3+15], a[%c1+12:%c1+15]"
                 :: "i"(W), "i"(R), "i"(A), "i"(B) : "memory", ALL_AGPRS);
}
template <int R, int A, int B> __device__ __forceinline__ void mfma1() {
    asm volatile("v_mfma_f32_16x16x32_f16 a[%c0:%c0+3], v[%c1:%c1+3], v[%c2:%c2+3], a[%c0:%c0+3]" :: "i"(R), "i"(A), "i"(B) : "memory");
}
template <int R, int W, int A, int B> __device__ __forceinline__ void mfma1w() {
    asm volatile("s_waitcnt lgkmcnt(%c0)\n\tv_mfma_f32_16x16x32_f16 a[%c1:%c1+3], v[%c2:%c2+3], v[%c3:%c3+3], a[%c1:%c1+3]" :: "i"(W), "i"(R), "i"(A), "i"(B) : "memory", ALL_AGPRS);
}
template <int R0, int R1> __device__ __forceinline__ unsigned long long ready_mask(unsigned thr) {
    unsigned long long m; unsigned t;
    asm volatile("v_min_u32 %1, v%c3, v%c4\n\tv_cmp_le_u32 %0, %2, %1" : "=s"(m), "=&v"(t) : "s"(thr), "i"(R0), "i"(R1));
    return m;
}
template <int R> __device__ __forceinline__ void acc_zero4() {
    asm volatile("v_accvgpr_write_b32 a%c0, 0\n\tv_accvgpr_write_b32 a[%c0+1], 0\n\tv_accvgpr_write_b32 a[%c0+2], 0\n\tv_accvgpr_write_b32 a[%c0+3], 0" :: "i"(R) : ALL_AGPRS);
}
template <int R> __device__ __forceinline__ void acc_zero16(const u32x4& z) {
    asm volatile("v_mfma_f32_32x32x16_f16 a[%c1:%c1+15], %0, %0, 0" :: "v"(z), "i"(R) : ALL_AGPRS);
}
__device__ __forceinline__ void acc_zero_all() {
    u32x4 z = {0u, 0u, 0u, 0u}; asm volatile("s_nop 1" : "+v"(z));
    sfor<16>([&](auto I) __attribute__((always_inline)) { acc_zero16<I * 16>(z); });
    asm volatile("s_nop 15\n\ts_nop 3" ::: "memory");
}
template <int R> __device__ __forceinline__ f32x4 acc_read_row() {
    float a, b, c, d;
    asm volatile("v_accvgpr_read_b32 %0, a%c4\n\tv_accvgpr_read_b32 %1, a[%c4+4]\n\tv_accvgpr_read_b32 %2, a[%c4+8]\n\tv_accvgpr_read_b32 %3, a[%c4+12]"
                 : "=v"(a), "=v"(b), "=v"(c), "=v"(d) : "i"(R));
    return (f32x4){a, b, c, d};
}
template <int R> __device__ __forceinline__ void mfma_f32(float a, float b) {
    asm volatile("s_nop 1\n\tv_mfma_f32_16x16x4_f32 a[%c2:%c2+3], %0, %1, a[%c2:%c2+3]" :: "v"(a), "v"(b), "i"(R) : ALL_AGPRS);
}

#define STAMP(i) do {} while (0)

__global__ __launch_bounds__(256, 1) __attribute__((amdgpu_num_vgpr(58))) void fused_kernel(Params p) {
    extern __shared__ __attribute__((aligned(16))) unsigned char smem[];
    LAS unsigned char* lds = (LAS unsigned char*)smem;
    const int tid = threadIdx.x, wid = __builtin_amdgcn_readfirstlane(tid >> 6), lane = tid & 63;
    const int fr = lane & 15, fq = lane >> 4;
    const int rank = p.rank;
    char* wsl = uptr(p.ws[0]);
    unsigned* ctl = (unsigned*)(wsl + WS_CTL);
    char* tmo = wsl + WS_CTL + CW_TMO * 4;
    asm volatile("" ::: OWNED_REGS);
    STAMP(0);
    const int bid = blockIdx.x, xcd = bid & 7, idx = bid >> 3;
    const int pm = (idx >> 2) * 2 + (xcd & 1), pn = (xcd >> 1) * 4 + (idx & 3);
    const bool local = (pm >> 1) == rank;
    const int jwg = local ? 224 + (idx & 3) * 8 + xcd : (((idx >> 2) < rank ? (idx >> 2) : (idx >> 2) - 1) * 4 + (idx & 3)) * 8 + xcd;
    const int unit = (jwg >> 2) + 64 * wid, uq = jwg & 3;
    const int ukh = unit >> 1, upl = unit & 1, upm = rank * 2 + upl;
    const bool burst = ukh < KBURST;

    const unsigned voff0 = lane * 16, voff1 = lane * 16 + 4096;
    unsigned ticket = 0;
    if (lane == 0) asm volatile("global_atomic_add %0, %1, %2, %3 sc0" : "=v"(ticket) : "v"(0u), "v"(1u), "s"(wsl + WS_CTL + CW_TICKET * 4 + wid * 128) : "memory");
    const char* xb = (const char*)uptr(p.x) + ((size_t)(upl * 256 + uq * 64) * GK + ukh * 32) * 4;
    const unsigned xv = (unsigned)(fr * GK + fq * 8) * 4u;
    sfor<4>([&](auto I) __attribute__((always_inline)) { constexpr int i = I; gload16<R_S + 16 + 4 * i, 0>((unsigned)((lane * 8 + i * 2 + 1) * GK + ((lane * 68 + i * 1028) & (GK - 1))) * 4u, (const char*)uptr(p.x)); });
    sfor<4>([&](auto I) __attribute__((always_inline)) { constexpr int i = I; gload16<R_X + 8 * i, 0>(xv, xb + (size_t)i * 16 * GK * 4); gload16<R_X + 8 * i + 4, 16>(xv, xb + (size_t)i * 16 * GK * 4); });
    const unsigned wvoff = (unsigned)(fq * 8 * GN + fr * 4) * 4u;
    const char* wbase = (const char*)uptr(p.w) + ((size_t)pm * 32 * GN + pn * 256 + wid * 64) * 4;
    sfor<4>([&](auto I) __attribute__((always_inline)) { constexpr int i = I; gload16<R_S + 4 * i, 0>((unsigned)((lane * 64 + i * 16 + 5) * GN + ((lane * 68 + i * 1028) & (GN - 1))) * 4u, (const char*)uptr(p.w)); });
    sfor<8>([&](auto J) __attribute__((always_inline)) { constexpr int j = J; gload16<R_W + 4 * j, 0>(wvoff, wbase + (size_t)j * GN * 4); });
    {
        unsigned long long pc; unsigned sz;
        asm volatile("s_getpc_b64 %0\n.Lagk_begin:\n\ts_mov_b32 %1, .Lagk_end-.Lagk_begin" : "=s"(pc), "=s"(sz));
        gload16<R_B + 124, 0>(min((unsigned)(idx * 4 + wid) * 1024u, sz - 1024u) + lane * 16, (const char*)pc);
    }
    const int r0 = (unit * 4 + uq * 2) % 7;
    char* fdst = wsl;
#pragma unroll
    for (int i = 1; i < WORLD; ++i) { char* q = p.ws[i]; fdst = (lane & 7) == i ? q : fdst; }
    STAMP(15);
    asm volatile("s_waitcnt vmcnt(13)" : "+v"(ticket) :: "memory");
    STAMP(14);
    const unsigned call = __builtin_amdgcn_readfirstlane(ticket) / GRID + 1u;
    const size_t abuf = WS_A0 + (size_t)(call & 1u) * OPER_B;
    const size_t faoff = WS_FLAGA + (size_t)(call & 1u) * 65536;
    int dead = 0;
    const int ex = scale_exp(wave_umax(amax_regs<R_S + 16, 16>()), 4);
    const unsigned tx0 = (unsigned)rtclock();
    STAMP(3);
    u32x4 xq[4];
    bool xbad;
    {
        const float sx = exp2i(ex);
        sfor<4>([&](auto I) __attribute__((always_inline)) { constexpr int i = I; xq[i] = cvt8r<R_X + 8 * i, 1>(sx); });
        const float a = __uint_as_float(amax_regs<R_X, 32>()) * sx;
        xbad = __ballot(!(a < 60000.0f)) != 0ull || (__ballot(a >= 0.000244140625f) == 0ull && __ballot(a != 0.0f) != 0ull);
    }
    const unsigned xoff = (unsigned)(abuf + (size_t)upm * PANEL_B + (size_t)ukh * 16384 + uq * 4096 + lane * 16);
    char* fladdr = fdst + faoff + (((size_t)upm * 128 + ukh) * 4 + uq) * 8;
    const u32x2 flval = {call, (unsigned)(ex & 0xffff) | (xbad ? 0x10000u : 0u)};
    const unsigned plo = (unsigned)(unsigned long long)fdst, phi = (unsigned)((unsigned long long)fdst >> 32);
    const unsigned xlds = (unsigned)(uintptr_t)lds + LDS_X + wid * 4096 + lane * 16;
    sfor<4>([&](auto I) __attribute__((always_inline)) { constexpr int i = I; lds_write16<i * 1024>(xlds, xq[i]); });
    int pst = 0;
    auto push_all = [&]() __attribute__((always_inline)) {
        STAMP(11);
        const u32x4 v0 = lds_read16_wait<0>(xlds), v1 = lds_read16_wait<1024>(xlds), v2 = lds_read16_wait<2048>(xlds), v3 = lds_read16_wait<3072>(xlds);
        int q = r0;
#pragma unroll 1
        for (int d = 0; d < 7; ++d) {
            gstore64_sys_lane(xoff, v0, v1, v2, v3, plo, phi, q + 1);
            q = q == 6 ? 0 : q + 1;
        }
        pst = 1;
    };
    auto push_flag = [&]() __attribute__((always_inline)) { if (lane >= 1 && lane < 8) gstore8_sys(fladdr, flval); pst = 5; STAMP(12); };
    const unsigned due = tx0 + (unsigned)(ukh < KNOW ? 0 : ukh * PACE);
    if (__builtin_expect(xbad, 0)) {
        const unsigned so = (unsigned)(WS_X32 + (size_t)(call & 1u) * 2 * OPER_B + ((size_t)(upm * 256 + uq * 64) * GK + ukh * 32) * 4) + xv;
#pragma unroll 1
        for (int d = 0; d < WORLD; ++d) {
            const unsigned long long bq = ((unsigned long long)(unsigned)__builtin_amdgcn_readlane((int)phi, d) << 32) | (unsigned)__builtin_amdgcn_readlane((int)plo, d);
            sfor<8>([&](auto I) __attribute__((always_inline)) { constexpr int i = I; storeown_sys<R_X + 4 * i, (i & 1) * 16>(so + (unsigned)(i >> 1) * 16u * GK * 4u, (char*)bq); });
        }
        wait_vm<0>();
    }
    if (ukh < KNOW) push_all();
    sfor<4>([&](auto I) __attribute__((always_inline)) { constexpr int i = I; gstore16_sys<i * 1024>(xoff, xq[i], wsl); });
    sfor<8>([&](auto J) __attribute__((always_inline)) { constexpr int j = J; gload16<R_W2 + 4 * j, 0>(wvoff, wbase + (size_t)(512 + j) * GN * 4); });
    if (ukh < KNOW) wait_vm<41>(); else wait_vm<13>();
    const int ew = scale_exp(wave_umax(amax_regs<R_S, 16>()), 0);
    const float sw = exp2i(ew);
    char* bdst = wsl + WS_B + (size_t)pn * PANEL_B + (size_t)(pm >> 1) * TB + wid * 8192 + (pm & 1) * 4096;
    char* fbdst = wsl + WS_FLAGB + ((size_t)(pn * 4 + wid) * 128 + pm) * 4;
    auto wbits = [&]() __attribute__((always_inline)) -> unsigned {
        const float a = __uint_as_float(amax_regs<R_W, 32>()) * sw;
        return call * 4u + ((__ballot(!(a < 60000.0f)) ? 1u : 0u) | (__ballot(a >= 0.015625f) ? 2u : 0u));
    };
    auto wstore = [&](auto C, char* dst) __attribute__((always_inline)) { constexpr int c = C; gstore16_agent<c * 1024>(voff0, cvt8r<R_W + c, 4>(sw), dst); };
    sfor<4>([&](auto C) __attribute__((always_inline)) { wstore(C, bdst); });
    const unsigned fb0 = wbits();
    wait_vm<0>();
    gstore4_agent(0u, fb0, fbdst);
    if (lane == 0) gstore8_sys(fladdr, flval);
    if (pst == 1) push_flag();
    if (pst == 0 && ukh < KEARLY) {
        while ((int)((unsigned)rtclock() - due) < 0) __builtin_amdgcn_s_sleep(1);
        push_all(); wait_vm<0>(); push_flag();
    }
    {
        sfor<4>([&](auto C) __attribute__((always_inline)) { constexpr int c = C; gstore16_agent<c * 1024>(voff0, cvt8r<R_W2 + c, 4>(sw), bdst + 8 * TB); });
        const float a = __uint_as_float(amax_regs<R_W2, 32>()) * sw;
        const unsigned f1 = call * 4u + ((__ballot(!(a < 60000.0f)) ? 1u : 0u) | (__ballot(a >= 0.015625f) ? 2u : 0u));
        wait_vm<0>();
        gstore4_agent(64u, f1, fbdst);
    }
    STAMP(4);

    const char* gA = wsl + abuf + (size_t)pm * PANEL_B + wid * 8192;
    const char* gB = wsl + WS_B + (size_t)pn * PANEL_B + wid * 8192;
    const char* fA = wsl + faoff + (size_t)pm * 4096;
    const char* fB = wsl + WS_FLAGB + (size_t)(pn * 4 + wid) * 512;
    int RA = 0, RB = 0, exa = 0;
    int plim = 0;
    int qa = 0, qb = 0;
    int tsend = 0;
    int evt = 0;
    int nslow = 0;
    const unsigned call4 = call * 4u;
    auto pollA = [&](int last) __attribute__((always_inline)) { qa = RA; last = min(last, qa + PWIN - 1);
        gload16_sys<R_PA>((unsigned)min(qa + (lane >> 2), last) * 64u + (lane & 3) * 16u, fA);
    };
    auto pollB = [&](int last) __attribute__((always_inline)) { qb = RB; gload8_agent<R_PB>((unsigned)min(qb + fr, last) * 8u, fB); };
    auto digestA = [&](int lim) __attribute__((always_inline)) { const unsigned long long nm = ~ready_mask<R_PA, R_PA + 2>(call); RA = max(RA, min(qa + min(nm ? (int)__builtin_ctzll(nm) >> 2 : 16, PWIN), lim)); };
    auto digestB = [&](int lim) __attribute__((always_inline)) { RB = max(RB, min(qb + (int)__builtin_ctz(~(unsigned)ready_mask<R_PB, R_PB + 1>(call4) | 0x10000u), lim)); };
    auto limits = [&]() __attribute__((always_inline)) { nslow = min(min(RA >= NT ? 1000 : RA - 3, RB >= NT ? 1000 : min(RB - 2, NT - 4)), evt); };
    auto retime = [&](int tcur) __attribute__((always_inline)) {
        const int rem = (int)(due - (unsigned)rtclock());
        tsend = rem <= TILE_TICKS / 2 ? tcur : tcur + max(1, CLOSED ? (rem * 7) / (8 * TILE_TICKS) : (rem + TILE_TICKS / 2) / TILE_TICKS);
    };
    auto spin = [&](int na, int nb, int tcur) __attribute__((always_inline)) {
        const unsigned long long t0 = rtclock();
        wait_vm<0>();
        if (pst == 1 || pst == 2) push_flag();
        bool wide = true;
        for (int it = 1; !dead; ++it) {
            if (pst == 0 && (int)((unsigned)rtclock() - due) >= 0) { push_all(); wait_vm<0>(); push_flag(); }
            const int la = wide ? NT - 1 : min(max(na, RA + 1) + 2, NT) - 1, lb = wide ? NT - 1 : min(max(nb, RB + 1) + 4, NT) - 1;
            pollA(la); pollB(lb);
            wait_vm<0>();
            digestA(la + 1); digestB(lb + 1);
            plim = 0;
            if (__builtin_amdgcn_readfirstlane(rdreg<R_PA>()) == call) exa = (int)(short)__builtin_amdgcn_readfirstlane((int)rdreg<R_PA + 1>());
            if (RA >= na && RB >= nb) break;
            wide = false;
            const unsigned long long now = rtclock();
            if ((it & 15) == 0 || now - t0 > DEADLINE_TICKS) dead = give_up(tmo, call, t0);
            __builtin_amdgcn_s_sleep(SPINSLEEP);
        }
        if (pst == 0) retime(tcur);
    };
    acc_zero_all();
    const unsigned ldsb = (unsigned)(uintptr_t)lds + lane * 16, ldsh = ldsb + 65536;
#define GLDS_A(tile, st, i) __builtin_amdgcn_global_load_lds((const unsigned*)(gA + (size_t)(tile) * TB + ((i) >> 2) * 4096 + voff0), (LAS unsigned*)(lds + (st) * TB + wid * 8192 + ((i) >> 2) * 4096), 16, ((i) & 3) * 1024, 0)
#define LOAD_B(buf, tile, i) gload16<R_B + (buf) * 32 + (i) * 4, ((i) & 3) * 1024>(((i) >> 2) ? voff1 : voff0, gB + (size_t)(tile) * TB)
    sfor<8>([&](auto J) __attribute__((always_inline)) { constexpr int j = J; gload16<R_W + 4 * j, 0>(wvoff, wbase + (size_t)(1024 + j) * GN * 4); });
    spin(3, 3, 0);
    sfor<8>([&](auto I) __attribute__((always_inline)) { constexpr int i = I; GLDS_A(0, 0, i); });
    sfor<8>([&](auto I) __attribute__((always_inline)) { constexpr int i = I; LOAD_B(0, 0, i); });
    sfor<8>([&](auto I) __attribute__((always_inline)) { constexpr int i = I; GLDS_A(1, 1, i); });
    sfor<8>([&](auto I) __attribute__((always_inline)) { constexpr int i = I; LOAD_B(1, 1, i); });
    sfor<8>([&](auto I) __attribute__((always_inline)) { constexpr int i = I; GLDS_A(2, 2, i); });
    wait_vm<16>();
    __builtin_amdgcn_s_barrier();
    STAMP(5);
    sfor<APF>([&](auto G) __attribute__((always_inline)) { constexpr int g = G; ldsrd<R_AF + 4 * g, g * 1024>(ldsb, ldsh); });
    evt = pst == 5 ? 1000 : tsend;
    limits();
    float wmax = 0.f; unsigned wfl = 0;
    auto tile = [&](auto PH_, auto LASTP_, const int t) __attribute__((always_inline)) {
        constexpr int PH = PH_; constexpr bool LASTQ = LASTP_, LASTP = LASTQ || ABL_NOW;
        constexpr int cur = PH & 3, PP = (PH + 7) & 7;
        constexpr bool PREV_W = !ABL_NOW && !(LASTP && PH > 0);
        constexpr int e1p = (PREV_W && PP == 6 ? 8 : 0) + (PREV_W && PP == 2 ? 1 : 0), e2p = (PREV_W && PP == 0 ? 4 : 0) + ((PP & 1) ? 2 : 0);
        constexpr int base = 16 + e1p + e2p;
        constexpr bool FIN = LASTQ && PH == 4;
        if (__builtin_expect(t >= nslow, 0)) {
            if (!FLAG1 && pst == 1) { wait_vm<base + 28>(); pst = 2; } else { wait_vm<base>(); if (pst == 1 || pst == 2) push_flag(); }
            const int na = t + 4 < NT ? t + 4 : NT, nb = FIN ? NT : t + 3;
            if (RA < na || RB < nb) spin(na, nb, t);
            if (pst == 0 && t >= tsend) { if (CLOSED) retime(t); if (t >= tsend) push_all(); }
            evt = pst == 5 ? 1000 : (pst == 0 ? tsend : t + 1);
            limits();
        } else wait_vm<base>();
        const int slab = min((t >> 3) + 2, 7);
        sfor<32>([&](auto G) __attribute__((always_inline)) { constexpr int g = G;
            constexpr int RC = (g & 15) * 16, RAF = R_AF + 4 * (g & 7), RBF = R_B + cur * 32 + (g >> 4) * 16;
            if constexpr (g == 16) __builtin_amdgcn_s_barrier();
            constexpr int N0 = (SNAKE && (g & 1)) ? 12 : 0, DN = (SNAKE && (g & 1)) ? -4 : 4;
            mfma1w<RC + N0, (FIN && g + APF > 32) ? (32 - g - 1) : APF - 1, RAF, RBF + N0>();
            if constexpr (g + APF < 32) ldsrd<R_AF + 4 * ((g + APF) & 7), (PH & 3) * TB + (g + APF) * 1024>(ldsb, ldsh);
            else if constexpr (!FIN) ldsrd<R_AF + 4 * ((g + APF) & 7), ((PH + 1) & 3) * TB + (g + APF - 32) * 1024>(ldsb, ldsh);
            mfma1<RC + N0 + DN, RAF, RBF + N0 + DN>();
            if constexpr ((PH & 1) == 1 && g == 8) digestA(plim);
            if constexpr ((PH & 1) == 1 && g == 10) digestB(plim);
            if constexpr ((PH & 1) == 1 && g == 12) limits();
            if constexpr (!LASTP && PH == 0 && g >= 8 && g < 16) { if constexpr (g == 8) wmax = 0.f; amax4r<R_W + 4 * (g - 8)>(wmax); }
            if constexpr (!LASTP && PH == 0 && g == 17) { const float a = wmax * sw; wfl = call4 + ((__ballot(!(a < 60000.0f)) ? 1u : 0u) | (__ballot(a >= 0.015625f) ? 2u : 0u)); }
            u32x4 wq;
            if constexpr (!LASTP && PH == 0 && g >= 25 && g < 29) wq = cvt8r<R_W + (g - 25), 4>(sw);
            mfma1<RC + N0 + 2 * DN, RAF, RBF + N0 + 2 * DN>();
            if constexpr (g < 8) LOAD_B((cur + 2) & 3, t + 2, g);
            if constexpr (FIN && g >= 8 && g < 16) LOAD_B(3, t + 3, g - 8);
            if constexpr (FIN && g >= 26 && g < 30) gload16_sys<R_W + 4 * (g - 26)>(lane * 16 + (g - 26) * 1024, fA);
            if constexpr (FIN && g == 16) gload8_agent<R_PB>(lane * 8, fB);
            if constexpr (!LASTP && PH == 6 && g >= 8 && g < 16) gload16<R_W + 4 * (g - 8), 0>(wvoff, wbase + ((size_t)min(slab + 1, 7) * 512 + (g - 8)) * GN * 4);
            if constexpr (!LASTP && PH == 2 && g == 9) gstore4_agent(0u, wfl, fbdst + slab * 64);
            if constexpr (g >= 17 && g < 25) GLDS_A(t + 3, (PH + 3) & 3, g - 17);
            if constexpr (!LASTP && PH == 0 && g >= 25 && g < 29) gstore16_agent<(g - 25) * 1024>(voff0, wq, bdst + (size_t)slab * 8 * TB);
            if constexpr ((PH & 1) == 1 && g == 29) { pollA(NT - 1); plim = NT; }
            if constexpr ((PH & 1) == 1 && g == 30) pollB(NT - 1);
            mfma1<RC + N0 + 3 * DN, RAF, RBF + N0 + 3 * DN>();
        });
    };
    for (int s = 0; s < 7; ++s) {
        sfor<8>([&](auto PH) __attribute__((always_inline)) { tile(PH, std::false_type{}, s * 8 + PH); });
    }
    sfor<5>([&](auto PH) __attribute__((always_inline)) { tile(PH, std::true_type{}, 56 + PH); });
    if (pst == 0) push_all();
    wait_vm<0>();
    if (pst != 5) push_flag();
    __builtin_amdgcn_s_barrier();
    STAMP(8);
    bool wbad;
    { const unsigned o = rdreg<R_PB>() | rdreg<R_PB + 1>(); wbad = __ballot(o & 1u) != 0ull || __ballot(o & 2u) == 0ull;
      unsigned xo = 0; sfor<8>([&](auto I) __attribute__((always_inline)) { xo |= rdreg<R_W + 2 * I + 1>(); }); wbad = wbad || __ballot(xo & 0x10000u) != 0ull; }
    float fx = exp2i(-exa), fw = exp2i(-ew);
    bool bad = false;
    float* cp = p.out + (size_t)(pm * 256 + fq * 4) * GN + pn * 256 + wid * 64 + fr * 4;
    auto store_rows = [&](auto M_) __attribute__((always_inline)) { constexpr int m = M_;
        sfor<4>([&](auto J) __attribute__((always_inline)) { constexpr int j = J;
            f32x4 v = acc_read_row<m * 16 + j>();
#pragma unroll
            for (int e = 0; e < 4; ++e) { float y = v[e]; y = y < 0.f ? 0.f : y; v[e] = bad ? __builtin_nanf("") : (y * fx) * fw; }
            __builtin_nontemporal_store(v, (f32x4*)(cp + (size_t)(m * 16 + j) * GN));
        });
    };
    if (!wbad) {
        bad = dead || load_word(tmo) == call;
        sfor<APF>([&](auto I) __attribute__((always_inline)) { constexpr int i = I, m = i / 6, q = i % 6; ldsrd<R_AF + 4 * (i & 7), (1 + (q >> 1)) * TB + ((q & 1) * 16 + m) * 1024>(ldsb, ldsh); });
        sfor<96>([&](auto I) __attribute__((always_inline)) { constexpr int i = I, m = i / 6, q = i % 6, i2 = i + APF, m2 = i2 / 6, q2 = i2 % 6;
            if constexpr (i2 < 96) ldsrd<R_AF + 4 * (i2 & 7), (1 + (q2 >> 1)) * TB + ((q2 & 1) * 16 + m2) * 1024>(ldsb, ldsh);
            mfma_group<m * 16, (i2 < 96) ? APF : (95 - i), R_AF + 4 * (i & 7), R_B + (1 + (q >> 1)) * 32 + (q & 1) * 16>();
            if constexpr (q == 1 && m > 0) store_rows(std::integral_constant<int, m - 1>{});
        });
        asm volatile("s_nop 15\n\ts_nop 15" ::: "memory");
        store_rows(std::integral_constant<int, 15>{});
    } else {
        bad = dead || load_word(tmo) == call;
        asm volatile("s_nop 15\n\ts_nop 15" ::: "memory");
        acc_zero_all();
        const _Float16* ax = (const _Float16*)(wsl + abuf + (size_t)pm * PANEL_B);
        const float* x32 = (const float*)(wsl + WS_X32 + (size_t)(call & 1u) * 2 * OPER_B) + (size_t)(pm * 256 + fr) * GK;
        const unsigned* fl = (const unsigned*)fA;
        const float* wq = p.w + pn * 256 + wid * 64 + fr * 4;
        for (int k0 = 0; k0 < GK; k0 += 4) {
            const int k = k0 + fq;
            const f32x4 b = *(const f32x4*)(wq + (size_t)k * GN);
            const _Float16* ap = ax + (size_t)(k >> 5) * 8192 + (size_t)(((k >> 3) & 3) * 16 + fr) * 8 + (k & 7);
            sfor<4>([&](auto Q_) __attribute__((always_inline)) { constexpr int q = Q_;
                const bool f32src = (__hip_atomic_load(fl + ((k0 >> 5) * 4 + q) * 2 + 1, __ATOMIC_RELAXED, __HIP_MEMORY_SCOPE_SYSTEM) & 0x10000u) != 0u;
                sfor<4>([&](auto M_) __attribute__((always_inline)) { constexpr int m = q * 4 + M_;
                    const float a = f32src ? __hip_atomic_load(x32 + (size_t)m * 16 * GK + k, __ATOMIC_RELAXED, __HIP_MEMORY_SCOPE_SYSTEM) : (float)ap[m * 512] * fx;
                    mfma_f32<m * 16>(a, b[0]); mfma_f32<m * 16 + 4>(a, b[1]); mfma_f32<m * 16 + 8>(a, b[2]); mfma_f32<m * 16 + 12>(a, b[3]);
                });
            });
        }
        fx = 1.0f; fw = 1.0f;
        asm volatile("s_nop 15\n\ts_nop 15" ::: "memory");
        sfor<16>([&](auto M_) __attribute__((always_inline)) { store_rows(M_); });
    }
    asm volatile(".Lagk_end:" ::: "memory");
    STAMP(9);
}

}

extern "C" void kernel_launch(int rank, int nranks, ncclComm_t comm,
                              hipStream_t stream,
                              void* const* d_in, const int* in_sizes, int n_in,
                              void* d_out, int out_size,
                              const int* in_dtypes, int out_dtype,
                              void* const* d_out_all,
                              void* const* d_ws_all, size_t ws_bytes) {
    if (nranks != WORLD || rank < 0 || rank >= WORLD || ws_bytes < WS_END || n_in < 2) { fprintf(stderr, "unsupported configuration\n"); return; }
    static bool init[WORLD] = {};
    if (!init[rank]) {
        HIP_CHECK(hipFuncSetAttribute((const void*)fused_kernel, hipFuncAttributeMaxDynamicSharedMemorySize, LDS_BYTES));
        init[rank] = true;
    }
    Params p;
    p.x = (const float*)d_in[0]; p.w = (const float*)d_in[1]; p.out = (float*)d_out; p.rank = rank;
    for (int i = 0; i < WORLD; ++i) p.ws[i] = (char*)d_ws_all[(rank + i) & 7];
    hipLaunchKernelGGL(fused_kernel, dim3(GRID), dim3(NTHREADS), LDS_BYTES, stream, p);
    HIP_CHECK(hipGetLastError());
}
```

```cpp
#include <hip/hip_runtime.h>
#include <rccl/rccl.h>
#include <cstdio>
#include <cstdint>
#include <utility>

#define HIP_CHECK(expr) do { hipError_t _e = (expr); if (_e != hipSuccess) { \
    fprintf(stderr, "HIP error %s at %s:%d\n", hipGetErrorString(_e), __FILE__, __LINE__); \
    return; } } while (0)

namespace {

template <int... Is, class F> __device__ __forceinline__ void sfor_impl(std::integer_sequence<int, Is...>, F&& f) { (f(std::integral_constant<int, Is>{}), ...); }
template <int N, class F> __device__ __forceinline__ void sfor(F&& f) { sfor_impl(std::make_integer_sequence<int, N>{}, f); }

typedef _Float16 half8 __attribute__((ext_vector_type(8)));
typedef float f32x4 __attribute__((ext_vector_type(4)));
typedef unsigned u32x4 __attribute__((ext_vector_type(4)));
typedef unsigned u32x2 __attribute__((ext_vector_type(2)));
#define LAS __attribute__((address_space(3)))

constexpr int WORLD = 8;
constexpr int GM = 4096, GK = 4096, GN = 4096;
constexpr int MS = GM / WORLD;
constexpr int NT = GK / 64;
constexpr int TB = 32768;
constexpr size_t PANEL_B = (size_t)NT * TB;
constexpr size_t OPER_B = 16 * PANEL_B;
constexpr size_t WS_CTL = 0;
constexpr size_t WS_SLOT = 4096;
constexpr size_t WS_FLAGA = 65536;
constexpr size_t WS_FLAGB = 262144;
constexpr size_t WS_DBG = 512u << 10;
constexpr size_t WS_A0 = 1u << 20;
constexpr size_t WS_B = WS_A0 + 2 * OPER_B;
constexpr size_t WS_X32 = WS_B + OPER_B;
constexpr size_t WS_END = WS_X32 + 4 * OPER_B;
constexpr int CW_TICKET = 0, CW_TMO = 256;
constexpr unsigned long long DEADLINE_TICKS = 50000000ull;
constexpr int LDS_X = 4 * TB;
constexpr int LDS_BYTES = 4 * TB + 16384;
constexpr int GRID = 256;
#define NTHREADS 256
#define PACE 68
#define PWIN 16
#define KNOW 4
#define K1 24
#define P1 56
#define POFF 0
#define SNAKE 1
#define CLOSED 0
#define FLAG1 1
#define KEARLY 8
#define KBURST 128
#define TILE_TICKS 145
#define SLOPE 43
#define OFFL 230
#define OFFR 440
#define ABL_NOPOLL 0
#define ABL_NOW 0
#define ABL_NOPUSH 0
#define SPINSLEEP 8
#define POLLP 1
#define PAHEAD 10
#define PBHEAD 7
#define APF 5

struct Params { const float* x; const float* w; float* out; char* ws[WORLD]; int rank; };

template <class T> __device__ __forceinline__ T* uptr(T* q) {
    unsigned long long v = (unsigned long long)q;
    unsigned lo = __builtin_amdgcn_readfirstlane((unsigned)v), hi = __builtin_amdgcn_readfirstlane((unsigned)(v >> 32));
    return (T*)(((unsigned long long)hi << 32) | lo);
}
__device__ __forceinline__ u32x4 cvt8(const f32x4 a, const f32x4 b, float s) {
    half8 h;
    h[0] = (_Float16)(a[0] * s); h[1] = (_Float16)(a[1] * s); h[2] = (_Float16)(a[2] * s); h[3] = (_Float16)(a[3] * s);
    h[4] = (_Float16)(b[0] * s); h[5] = (_Float16)(b[1] * s); h[6] = (_Float16)(b[2] * s); h[7] = (_Float16)(b[3] * s);
    return __builtin_bit_cast(u32x4, h);
}
__device__ __forceinline__ unsigned absbits(float v) { return __float_as_uint(v) & 0x7fffffffu; }
__device__ __forceinline__ unsigned amax4(const f32x4 a) { return max(max(absbits(a[0]), absbits(a[1])), max(absbits(a[2]), absbits(a[3]))); }
__device__ __forceinline__ unsigned wave_umax(unsigned v) {
    v = max(v, (unsigned)__builtin_amdgcn_update_dpp(0, (int)v, 0x111, 0xf, 0xf, false));
    v = max(v, (unsigned)__builtin_amdgcn_update_dpp(0, (int)v, 0x112, 0xf, 0xf, false));
    v = max(v, (unsigned)__builtin_amdgcn_update_dpp(0, (int)v, 0x114, 0xf, 0xf, false));
    v = max(v, (unsigned)__builtin_amdgcn_update_dpp(0, (int)v, 0x118, 0xf, 0xf, false));
    v = max(v, (unsigned)__builtin_amdgcn_update_dpp(0, (int)v, 0x142, 0xa, 0xf, false));
    v = max(v, (unsigned)__builtin_amdgcn_update_dpp(0, (int)v, 0x143, 0xc, 0xf, false));
    return (unsigned)__builtin_amdgcn_readlane((int)v, 63);
}
__device__ __forceinline__ int scale_exp(unsigned maxbits, int target) {
    const int ef = (int)(maxbits >> 23);
    if (ef == 0 || ef == 255) return 0;
    int e = target - (ef - 127);
    return e > 126 ? 126 : (e < -126 ? -126 : e);
}
__device__ __forceinline__ float exp2i(int e) { return __uint_as_float((unsigned)(e + 127) << 23); }

constexpr int R_PB = 58, R_PA = 60, R_W = 64, R_AF = 96, R_B = 128, R_X = 128, R_S = 96, R_SL = 112, R_W2 = 160;
#define OWNED_REGS "a0","a1","a2","a3","a4","a5","a6","a7","a8","a9","a10","a11","a12","a13","a14","a15","a16","a17","a18","a19","a20","a21","a22","a23","a24","a25","a26","a27","a28","a29","a30","a31","a32","a33","a34","a35","a36","a37","a38","a39","a40","a41","a42","a43","a44","a45","a46","a47","a48","a49","a50","a51","a52","a53","a54","a55","a56","a57","a58","a59","a60","a61","a62","a63","a64","a65","a66","a67","a68","a69","a70","a71","a72","a73","a74","a75","a76","a77","a78","a79","a80","a81","a82","a83","a84","a85","a86","a87","a88","a89","a90","a91","a92","a93","a94","a95","a96","a97","a98","a99","a100","a101","a102","a103","a104","a105","a106","a107","a108","a109","a110","a111","a112","a113","a114","a115","a116","a117","a118","a119","a120","a121","a122","a123","a124","a125","a126","a127","a128","a129","a130","a131","a132","a133","a134","a135","a136","a137","a138","a139","a140","a141","a142","a143","a144","a145","a146","a147","a148","a149","a150","a151","a152","a153","a154","a155","a156","a157","a158","a159","a160","a161","a162","a163","a164","a165","a166","a167","a168","a169","a170","a171","a172","a173","a174","a175","a176","a177","a178","a179","a180","a181","a182","a183","a184","a185","a186","a187","a188","a189","a190","a191","a192","a193","a194","a195","a196","a197","a198","a199","a200","a201","a202","a203","a204","a205","a206","a207","a208","a209","a210","a211","a212","a213","a214","a215","a216","a217","a218","a219","a220","a221","a222","a223","a224","a225","a226","a227","a228","a229","a230","a231","a232","a233","a234","a235","a236","a237","a238","a239","a240","a241","a242","a243","a244","a245","a246","a247","a248","a249","a250","a251","a252","a253","a254","a255","v58","v59","v60","v61","v62","v63","v64","v65","v66","v67","v68","v69","v70","v71","v72","v73","v74","v75","v76","v77","v78","v79","v80","v81","v82","v83","v84","v85","v86","v87","v88","v89","v90","v91","v92","v93","v94","v95","v96","v97","v98","v99","v100","v101","v102","v103","v104","v105","v106","v107","v108","v109","v110","v111","v112","v113","v114","v115","v116","v117","v118","v119","v120","v121","v122","v123","v124","v125","v126","v127","v128","v129","v130","v131","v132","v133","v134","v135","v136","v137","v138","v139","v140","v141","v142","v143","v144","v145","v146","v147","v148","v149","v150","v151","v152","v153","v154","v155","v156","v157","v158","v159","v160","v161","v162","v163","v164","v165","v166","v167","v168","v169","v170","v171","v172","v173","v174","v175","v176","v177","v178","v179","v180","v181","v182","v183","v184","v185","v186","v187","v188","v189","v190","v191","v192","v193","v194","v195","v196","v197","v198","v199","v200","v201","v202","v203","v204","v205","v206","v207","v208","v209","v210","v211","v212","v213","v214","v215","v216","v217","v218","v219","v220","v221","v222","v223","v224","v225","v226","v227","v228","v229","v230","v231","v232","v233","v234","v235","v236","v237","v238","v239","v240","v241","v242","v243","v244","v245","v246","v247","v248","v249","v250","v251","v252","v253","v254","v255"
#define ALL_AGPRS "a0","a1","a2","a3","a4","a5","a6","a7","a8","a9","a10","a11","a12","a13","a14","a15","a16","a17","a18","a19","a20","a21","a22","a23","a24","a25","a26","a27","a28","a29","a30","a31","a32","a33","a34","a35","a36","a37","a38","a39","a40","a41","a42","a43","a44","a45","a46","a47","a48","a49","a50","a51","a52","a53","a54","a55","a56","a57","a58","a59","a60","a61","a62","a63","a64","a65","a66","a67","a68","a69","a70","a71","a72","a73","a74","a75","a76","a77","a78","a79","a80","a81","a82","a83","a84","a85","a86","a87","a88","a89","a90","a91","a92","a93","a94","a95","a96","a97","a98","a99","a100","a101","a102","a103","a104","a105","a106","a107","a108","a109","a110","a111","a112","a113","a114","a115","a116","a117","a118","a119","a120","a121","a122","a123","a124","a125","a126","a127","a128","a129","a130","a131","a132","a133","a134","a135","a136","a137","a138","a139","a140","a141","a142","a143","a144","a145","a146","a147","a148","a149","a150","a151","a152","a153","a154","a155","a156","a157","a158","a159","a160","a161","a162","a163","a164","a165","a166","a167","a168","a169","a170","a171","a172","a173","a174","a175","a176","a177","a178","a179","a180","a181","a182","a183","a184","a185","a186","a187","a188","a189","a190","a191","a192","a193","a194","a195","a196","a197","a198","a199","a200","a201","a202","a203","a204","a205","a206","a207","a208","a209","a210","a211","a212","a213","a214","a215","a216","a217","a218","a219","a220","a221","a222","a223","a224","a225","a226","a227","a228","a229","a230","a231","a232","a233","a234","a235","a236","a237","a238","a239","a240","a241","a242","a243","a244","a245","a246","a247","a248","a249","a250","a251","a252","a253","a254","a255"
template <int R, int OFF> __device__ __forceinline__ void gload16(unsigned voff, const char* sbase) {
    asm volatile("global_load_dwordx4 v[%c2:%c2+3], %0, %1 offset:%c3" :: "v"(voff), "s"(sbase), "i"(R), "i"(OFF) : "memory");
}
template <int R> __device__ __forceinline__ void gload16_sys(unsigned voff, const char* sbase) {
    asm volatile("global_load_dwordx4 v[%c2:%c2+3], %0, %1 sc0 sc1" :: "v"(voff), "s"(sbase), "i"(R) : "memory");
}
template <int R> __device__ __forceinline__ void gload16_agent(unsigned voff, const char* sbase) {
    asm volatile("global_load_dwordx4 v[%c2:%c2+3], %0, %1 sc1" :: "v"(voff), "s"(sbase), "i"(R) : "memory");
}
template <int R> __device__ __forceinline__ void gload8_agent(unsigned voff, const char* sbase) {
    asm volatile("global_load_dwordx2 v[%c2:%c2+1], %0, %1 sc1" :: "v"(voff), "s"(sbase), "i"(R) : "memory");
}
template <int R> __device__ __forceinline__ unsigned rdreg() { unsigned v; asm volatile("v_mov_b32 %0, v%c1" : "=v"(v) : "i"(R)); return v; }
template <int OFF> __device__ __forceinline__ void gstore16_sys(unsigned voff, const u32x4& v, char* sbase) {
    asm volatile("global_store_dwordx4 %0, %1, %2 offset:%3 sc0 sc1\n\ts_nop 1" :: "v"(voff), "v"(v), "s"(sbase), "i"(OFF) : "memory");
}
template <int OFF> __device__ __forceinline__ void gstore16_agent(unsigned voff, const u32x4& v, char* sbase) {
    asm volatile("global_store_dwordx4 %0, %1, %2 offset:%3 sc1\n\ts_nop 1" :: "v"(voff), "v"(v), "s"(sbase), "i"(OFF) : "memory");
}
__device__ __forceinline__ void gstore64_sys_lane(unsigned voff, const u32x4& v0, const u32x4& v1, const u32x4& v2, const u32x4& v3, unsigned plo, unsigned phi, int ln) {
    const unsigned long long b = ((unsigned long long)(unsigned)__builtin_amdgcn_readlane((int)phi, ln) << 32) | (unsigned)__builtin_amdgcn_readlane((int)plo, ln);
    asm volatile("s_nop 4\n\tglobal_store_dwordx4 %0, %1, %5 sc0 sc1\n\tglobal_store_dwordx4 %0, %2, %5 offset:1024 sc0 sc1\n\t"
                 "global_store_dwordx4 %0, %3, %5 offset:2048 sc0 sc1\n\tglobal_store_dwordx4 %0, %4, %5 offset:3072 sc0 sc1\n\ts_nop 1"
                 :: "v"(voff), "v"(v0), "v"(v1), "v"(v2), "v"(v3), "s"(b) : "memory");
}
template <int R, int OFF> __device__ __forceinline__ void storeown_sys(unsigned voff, char* sbase) {
    asm volatile("s_nop 4\n\tglobal_store_dwordx4 %0, v[%c2:%c2+3], %1 offset:%c3 sc0 sc1\n\ts_nop 1" :: "v"(voff), "s"(sbase), "i"(R), "i"(OFF) : "memory");
}
__device__ __forceinline__ void gstore8_sys(char* addr, const u32x2& v) {
    asm volatile("global_store_dwordx2 %0, %1, off sc0 sc1\n\ts_nop 1" :: "v"(addr), "v"(v) : "memory");
}
__device__ __forceinline__ void gstore8_agent(unsigned voff, const u32x2& v, char* sbase) {
    asm volatile("global_store_dwordx2 %0, %1, %2 sc1\n\ts_nop 1" :: "v"(voff), "v"(v), "s"(sbase) : "memory");
}
__device__ __forceinline__ void gstore4_agent(unsigned voff, unsigned v, char* sbase) {
    asm volatile("global_store_dword %0, %1, %2 sc1\n\ts_nop 1" :: "v"(voff), "v"(v), "s"(sbase) : "memory");
}
__device__ __forceinline__ unsigned long long rtclock() {
    unsigned long long t; asm volatile("s_memrealtime %0\n\ts_waitcnt lgkmcnt(0)" : "=s"(t) :: "memory"); return t;
}
__device__ __forceinline__ unsigned load_word(const char* sbase) {
    unsigned v; asm volatile("global_load_dword %0, %1, %2 sc1\n\ts_waitcnt vmcnt(0)" : "=&v"(v) : "v"(0u), "s"(sbase) : "memory");
    return __builtin_amdgcn_readfirstlane(v);
}
__device__ __forceinline__ int give_up(char* tmo, unsigned call, unsigned long long t0) {
    if (load_word(tmo) == call) return 1;
    if (rtclock() - t0 > DEADLINE_TICKS) { gstore4_agent(0u, call, tmo); return 1; }
    __builtin_amdgcn_s_sleep(1);
    return 0;
}
template <int R, int OFF> __device__ __forceinline__ void ldsrd(unsigned lo, unsigned hi) {
    if constexpr (OFF < 65536) asm volatile("ds_read_b128 v[%c1:%c1+3], %0 offset:%c2" :: "v"(lo), "i"(R), "i"(OFF) : "memory");
    else asm volatile("ds_read_b128 v[%c1:%c1+3], %0 offset:%c2" :: "v"(hi), "i"(R), "i"(OFF - 65536) : "memory");
}
template <int OFF> __device__ __forceinline__ void lds_write16(unsigned a, const u32x4& v) { asm volatile("ds_write_b128 %0, %1 offset:%2" :: "v"(a), "v"(v), "i"(OFF) : "memory"); }
template <int OFF> __device__ __forceinline__ u32x4 lds_read16_wait(unsigned a) { u32x4 v; asm volatile("ds_read_b128 %0, %1 offset:%2\n\ts_waitcnt lgkmcnt(0)" : "=&v"(v) : "v"(a), "i"(OFF) : "memory"); return v; }
template <int N> __device__ __forceinline__ void wait_vm() { asm volatile("s_waitcnt vmcnt(%0)" :: "i"(N) : "memory"); }
template <int R> __device__ __forceinline__ void amax4r(float& m) {
    asm volatile("v_max3_f32 %0, |v%c1|, |v[%c1+1]|, %0\n\tv_max3_f32 %0, |v[%c1+2]|, |v[%c1+3]|, %0" : "+v"(m) : "i"(R));
}
template <int R, int N> __device__ __forceinline__ unsigned amax_regs() {
    float m = 0.f;
    sfor<N / 4>([&](auto I) __attribute__((always_inline)) { amax4r<R + 4 * I>(m); });
    return __float_as_uint(m);
}
template <int R, int S> __device__ __forceinline__ u32x4 cvt8r(float s) {
    unsigned a, b, c, d;
    asm volatile("v_fma_mixlo_f16 %0, v%c5, %4, 0\n\tv_fma_mixlo_f16 %1, v[%c5+2*%c6], %4, 0\n\tv_fma_mixlo_f16 %2, v[%c5+4*%c6], %4, 0\n\tv_fma_mixlo_f16 %3, v[%c5+6*%c6], %4, 0\n\t"
                 "v_fma_mixhi_f16 %0, v[%c5+%c6], %4, 0\n\tv_fma_mixhi_f16 %1, v[%c5+3*%c6], %4, 0\n\tv_fma_mixhi_f16 %2, v[%c5+5*%c6], %4, 0\n\tv_fma_mixhi_f16 %3, v[%c5+7*%c6], %4, 0"
                 : "=&v"(a), "=&v"(b), "=&v"(c), "=&v"(d) : "v"(s), "i"(R), "i"(S));
    return (u32x4){a, b, c, d};
}
template <int R, int W, int A, int B> __device__ __forceinline__ void mfma_group() {
    asm volatile("s_waitcnt lgkmcnt(%c0)\n\t"
                 "v_mfma_f32_16x16x32_f16 a[%c1:%c1+3], v[%c2:%c2+3], v[%c3:%c3+3], a[%c1:%c1+3]\n\t"
                 "v_mfma_f32_16x16x32_f16 a[%c1+4:%c1+7], v[%c2:%c2+3], v[%c3+4:%c3+7], a[%c1+4:%c1+7]\n\t"
                 "v_mfma_f32_16x16x32_f16 a[%c1+8:%c1+11], v[%c2:%c2+3], v[%c3+8:%c3+11], a[%c1+8:%c1+11]\n\t"
                 "v_mfma_f32_16x16x32_f16 a[%c1+12:%c1+15], v[%c2:%c2+3], v[%c3+12:%c3+15], a[%c1+12:%c1+15]"
                 :: "i"(W), "i"(R), "i"(A), "i"(B) : "memory", ALL_AGPRS);
}
template <int R, int A, int B> __device__ __forceinline__ void mfma1() {
    asm volatile("v_mfma_f32_16x16x32_f16 a[%c0:%c0+3], v[%c1:%c1+3], v[%c2:%c2+3], a[%c0:%c0+3]" :: "i"(R), "i"(A), "i"(B) : "memory");
}
template <int R, int W, int A, int B> __device__ __forceinline__ void mfma1w() {
    asm volatile("s_waitcnt lgkmcnt(%c0)\n\tv_mfma_f32_16x16x32_f16 a[%c1:%c1+3], v[%c2:%c2+3], v[%c3:%c3+3], a[%c1:%c1+3]" :: "i"(W), "i"(R), "i"(A), "i"(B) : "memory", ALL_AGPRS);
}
template <int R0, int R1> __device__ __forceinline__ unsigned long long ready_mask(unsigned thr) {
    unsigned long long m; unsigned t;
    asm volatile("v_min_u32 %1, v%c3, v%c4\n\tv_cmp_le_u32 %0, %2, %1" : "=s"(m), "=&v"(t) : "s"(thr), "i"(R0), "i"(R1));
    return m;
}
template <int R> __device__ __forceinline__ void acc_zero4() {
    asm volatile("v_accvgpr_write_b32 a%c0, 0\n\tv_accvgpr_write_b32 a[%c0+1], 0\n\tv_accvgpr_write_b32 a[%c0+2], 0\n\tv_accvgpr_write_b32 a[%c0+3], 0" :: "i"(R) : ALL_AGPRS);
}
template <int R> __device__ __forceinline__ void acc_zero16(const u32x4& z) {
    asm volatile("v_mfma_f32_32x32x16_f16 a[%c1:%c1+15], %0, %0, 0" :: "v"(z), "i"(R) : ALL_AGPRS);
}
__device__ __forceinline__ void acc_zero_all() {
    u32x4 z = {0u, 0u, 0u, 0u}; asm volatile("s_nop 1" : "+v"(z));
    sfor<16>([&](auto I) __attribute__((always_inline)) { acc_zero16<I * 16>(z); });
    asm volatile("s_nop 15\n\ts_nop 3" ::: "memory");
}
template <int R> __device__ __forceinline__ f32x4 acc_read_row() {
    float a, b, c, d;
    asm volatile("v_accvgpr_read_b32 %0, a%c4\n\tv_accvgpr_read_b32 %1, a[%c4+4]\n\tv_accvgpr_read_b32 %2, a[%c4+8]\n\tv_accvgpr_read_b32 %3, a[%c4+12]"
                 : "=v"(a), "=v"(b), "=v"(c), "=v"(d) : "i"(R));
    return (f32x4){a, b, c, d};
}
template <int R> __device__ __forceinline__ void mfma_f32(float a, float b) {
    asm volatile("s_nop 1\n\tv_mfma_f32_16x16x4_f32 a[%c2:%c2+3], %0, %1, a[%c2:%c2+3]" :: "v"(a), "v"(b), "i"(R) : ALL_AGPRS);
}

#define STAMP(i) do {} while (0)

__global__ __launch_bounds__(256, 1) __attribute__((amdgpu_num_vgpr(58))) void fused_kernel(Params p) {
    extern __shared__ __attribute__((aligned(16))) unsigned char smem[];
    LAS unsigned char* lds = (LAS unsigned char*)smem;
    const int tid = threadIdx.x, wid = __builtin_amdgcn_readfirstlane(tid >> 6), lane = tid & 63;
    const int fr = lane & 15, fq = lane >> 4;
    const int rank = p.rank;
    char* wsl = uptr(p.ws[0]);
    unsigned* ctl = (unsigned*)(wsl + WS_CTL);
    char* tmo = wsl + WS_CTL + CW_TMO * 4;
    asm volatile("" ::: OWNED_REGS);
    STAMP(0);
    const int bid = blockIdx.x, xcd = bid & 7, idx = bid >> 3;
    const int pm = (idx >> 2) * 2 + (xcd & 1), pn = (xcd >> 1) * 4 + (idx & 3);
    const bool local = (pm >> 1) == rank;
    const int jwg = local ? 224 + (idx & 3) * 8 + xcd : (((idx >> 2) < rank ? (idx >> 2) : (idx >> 2) - 1) * 4 + (idx & 3)) * 8 + xcd;
    const int unit = (jwg >> 2) + 64 * wid, uq = jwg & 3;
    const int ukh = unit >> 1, upl = unit & 1, upm = rank * 2 + upl;
    const bool burst = ukh < KBURST;

    const unsigned voff0 = lane * 16, voff1 = lane * 16 + 4096;
    unsigned ticket = 0;
    if (lane == 0) asm volatile("global_atomic_add %0, %1, %2, %3 sc0" : "=v"(ticket) : "v"(0u), "v"(1u), "s"(wsl + WS_CTL + CW_TICKET * 4 + wid * 128) : "memory");
    const char* xb = (const char*)uptr(p.x) + ((size_t)(upl * 256 + uq * 64) * GK + ukh * 32) * 4;
    const unsigned xv = (unsigned)(fr * GK + fq * 8) * 4u;
    sfor<4>([&](auto I) __attribute__((always_inline)) { constexpr int i = I; gload16<R_S + 16 + 4 * i, 0>((unsigned)((lane * 8 + i * 2 + 1) * GK + ((lane * 68 + i * 1028) & (GK - 1))) * 4u, (const char*)uptr(p.x)); });
    sfor<4>([&](auto I) __attribute__((always_inline)) { constexpr int i = I; gload16<R_X + 8 * i, 0>(xv, xb + (size_t)i * 16 * GK * 4); gload16<R_X + 8 * i + 4, 16>(xv, xb + (size_t)i * 16 * GK * 4); });
    const unsigned wvoff = (unsigned)(fq * 8 * GN + fr * 4) * 4u;
    const char* wbase = (const char*)uptr(p.w) + ((size_t)pm * 32 * GN + pn * 256 + wid * 64) * 4;
    sfor<4>([&](auto I) __attribute__((always_inline)) { constexpr int i = I; gload16<R_S + 4 * i, 0>((unsigned)((lane * 64 + i * 16 + 5) * GN + ((lane * 68 + i * 1028) & (GN - 1))) * 4u, (const char*)uptr(p.w)); });
    sfor<8>([&](auto J) __attribute__((always_inline)) { constexpr int j = J; gload16<R_W + 4 * j, 0>(wvoff, wbase + (size_t)j * GN * 4); });
    {
        unsigned long long pc; unsigned sz;
        asm volatile("s_getpc_b64 %0\n.Lagk_begin:\n\ts_mov_b32 %1, .Lagk_end-.Lagk_begin" : "=s"(pc), "=s"(sz));
        gload16<R_B + 124, 0>(min((unsigned)(idx * 4 + wid) * 1024u, sz - 1024u) + lane * 16, (const char*)pc);
    }
    const int r0 = (unit * 4 + uq * 2) % 7;
    char* fdst = wsl;
#pragma unroll
    for (int i = 1; i < WORLD; ++i) { char* q = p.ws[i]; fdst = (lane & 7) == i ? q : fdst; }
    STAMP(15);
    asm volatile("s_waitcnt vmcnt(13)" : "+v"(ticket) :: "memory");
    STAMP(14);
    const unsigned call = __builtin_amdgcn_readfirstlane(ticket) / GRID + 1u;
    const size_t abuf = WS_A0 + (size_t)(call & 1u) * OPER_B;
    const size_t faoff = WS_FLAGA + (size_t)(call & 1u) * 65536;
    int dead = 0;
    const int ex = scale_exp(wave_umax(amax_regs<R_S + 16, 16>()), 4);
    const unsigned tx0 = (unsigned)rtclock();
    STAMP(3);
    u32x4 xq[4];
    bool xbad;
    {
        const float sx = exp2i(ex);
        sfor<4>([&](auto I) __attribute__((always_inline)) { constexpr int i = I; xq[i] = cvt8r<R_X + 8 * i, 1>(sx); });
        const float a = __uint_as_float(amax_regs<R_X, 32>()) * sx;
        xbad = __ballot(!(a < 60000.0f)) != 0ull || (__ballot(a >= 0.000244140625f) == 0ull && __ballot(a != 0.0f) != 0ull);
    }
    const unsigned xoff = (unsigned)(abuf + (size_t)upm * PANEL_B + (size_t)ukh * 16384 + uq * 4096 + lane * 16);
    char* fladdr = fdst + faoff + (((size_t)upm * 128 + ukh) * 4 + uq) * 8;
    const u32x2 flval = {call, (unsigned)(ex & 0xffff) | (xbad ? 0x10000u : 0u)};
    const unsigned plo = (unsigned)(unsigned long long)fdst, phi = (unsigned)((unsigned long long)fdst >> 32);
    const unsigned xlds = (unsigned)(uintptr_t)lds + LDS_X + wid * 4096 + lane * 16;
    sfor<4>([&](auto I) __attribute__((always_inline)) { constexpr int i = I; lds_write16<i * 1024>(xlds, xq[i]); });
    int pst = 0;
    auto push_all = [&]() __attribute__((always_inline)) {
        STAMP(11);
        const u32x4 v0 = lds_read16_wait<0>(xlds), v1 = lds_read16_wait<1024>(xlds), v2 = lds_read16_wait<2048>(xlds), v3 = lds_read16_wait<3072>(xlds);
        int q = r0;
#pragma unroll 1
        for (int d = 0; d < 7; ++d) {
            gstore64_sys_lane(xoff, v0, v1, v2, v3, plo, phi, q + 1);
            q = q == 6 ? 0 : q + 1;
        }
        pst = 1;
    };
    auto push_flag = [&]() __attribute__((always_inline)) { if (lane >= 1 && lane < 8) gstore8_sys(fladdr, flval); pst = 5; STAMP(12); };
    const unsigned due = tx0 + (unsigned)(ukh < KNOW ? 0 : ukh * PACE);
    if (__builtin_expect(xbad, 0)) {
        const unsigned so = (unsigned)(WS_X32 + (size_t)(call & 1u) * 2 * OPER_B + ((size_t)(upm * 256 + uq * 64) * GK + ukh * 32) * 4) + xv;
#pragma unroll 1
        for (int d = 0; d < WORLD; ++d) {
            const unsigned long long bq = ((unsigned long long)(unsigned)__builtin_amdgcn_readlane((int)phi, d) << 32) | (unsigned)__builtin_amdgcn_readlane((int)plo, d);
            sfor<8>([&](auto I) __attribute__((always_inline)) { constexpr int i = I; storeown_sys<R_X + 4 * i, (i & 1) * 16>(so + (unsigned)(i >> 1) * 16u * GK * 4u, (char*)bq); });
        }
        wait_vm<0>();
    }
    if (ukh < KNOW) push_all();
    sfor<4>([&](auto I) __attribute__((always_inline)) { constexpr int i = I; gstore16_sys<i * 1024>(xoff, xq[i], wsl); });
    sfor<8>([&](auto J) __attribute__((always_inline)) { constexpr int j = J; gload16<R_W2 + 4 * j, 0>(wvoff, wbase + (size_t)(512 + j) * GN * 4); });
    if (ukh < KNOW) wait_vm<41>(); else wait_vm<13>();
    const int ew = scale_exp(wave_umax(amax_regs<R_S, 16>()), 0);
    const float sw = exp2i(ew);
    char* bdst = wsl + WS_B + (size_t)pn * PANEL_B + (size_t)(pm >> 1) * TB + wid * 8192 + (pm & 1) * 4096;
    char* fbdst = wsl + WS_FLAGB + ((size_t)(pn * 4 + wid) * 128 + pm) * 4;
    auto wbits = [&]() __attribute__((always_inline)) -> unsigned {
        const float a = __uint_as_float(amax_regs<R_W, 32>()) * sw;
        return call * 4u + ((__ballot(!(a < 60000.0f)) ? 1u : 0u) | (__ballot(a >= 0.015625f) ? 2u : 0u));
    };
    auto wstore = [&](auto C, char* dst) __attribute__((always_inline)) { constexpr int c = C; gstore16_agent<c * 1024>(voff0, cvt8r<R_W + c, 4>(sw), dst); };
    sfor<4>([&](auto C) __attribute__((always_inline)) { wstore(C, bdst); });
    const unsigned fb0 = wbits();
    wait_vm<0>();
    gstore4_agent(0u, fb0, fbdst);
    if (lane == 0) gstore8_sys(fladdr, flval);
    if (pst == 1) push_flag();
    if (pst == 0 && ukh < KEARLY) {
        while ((int)((unsigned)rtclock() - due) < 0) __builtin_amdgcn_s_sleep(1);
        push_all(); wait_vm<0>(); push_flag();
    }
    {
        sfor<4>([&](auto C) __attribute__((always_inline)) { constexpr int c = C; gstore16_agent<c * 1024>(voff0, cvt8r<R_W2 + c, 4>(sw), bdst + 8 * TB); });
        const float a = __uint_as_float(amax_regs<R_W2, 32>()) * sw;
        const unsigned f1 = call * 4u + ((__ballot(!(a < 60000.0f)) ? 1u : 0u) | (__ballot(a >= 0.015625f) ? 2u : 0u));
        wait_vm<0>();
        gstore4_agent(64u, f1, fbdst);
    }
    STAMP(4);

    const char* gA = wsl + abuf + (size_t)pm * PANEL_B + wid * 8192;
    const char* gB = wsl + WS_B + (size_t)pn * PANEL_B + wid * 8192;
    const char* fA = wsl + faoff + (size_t)pm * 4096;
    const char* fB = wsl + WS_FLAGB + (size_t)(pn * 4 + wid) * 512;
    int RA = 0, RB = 0, exa = 0;
    int plim = 0;
    int qa = 0, qb = 0;
    int tsend = 0;
    int evt = 0;
    int nslow = 0;
    const unsigned call4 = call * 4u;
    auto pollA = [&](int last) __attribute__((always_inline)) { qa = RA; last = min(last, qa + PWIN - 1);
        gload16_sys<R_PA>((unsigned)min(qa + (lane >> 2), last) * 64u + (lane & 3) * 16u, fA);
    };
    auto pollB = [&](int last) __attribute__((always_inline)) { qb = RB; gload8_agent<R_PB>((unsigned)min(qb + fr, last) * 8u, fB); };
    auto digestA = [&](int lim) __attribute__((always_inline)) { const unsigned long long nm = ~ready_mask<R_PA, R_PA + 2>(call); RA = max(RA, min(qa + min(nm ? (int)__builtin_ctzll(nm) >> 2 : 16, PWIN), lim)); };
    auto digestB = [&](int lim) __attribute__((always_inline)) { RB = max(RB, min(qb + (int)__builtin_ctz(~(unsigned)ready_mask<R_PB, R_PB + 1>(call4) | 0x10000u), lim)); };
    auto limits = [&]() __attribute__((always_inline)) { nslow = min(min(RA >= NT ? 1000 : min(RA - 2, NT - 4), RB >= NT ? 1000 : min(RB - 2, NT - 4)), evt); };
    auto retime = [&](int tcur) __attribute__((always_inline)) {
        const int rem = (int)(due - (unsigned)rtclock());
        tsend = rem <= TILE_TICKS / 2 ? tcur : tcur + max(1, CLOSED ? (rem * 7) / (8 * TILE_TICKS) : (rem + TILE_TICKS / 2) / TILE_TICKS);
    };
    auto spin = [&](int na, int nb, int tcur) __attribute__((always_inline)) {
        const unsigned long long t0 = rtclock();
        wait_vm<0>();
        if (pst == 1 || pst == 2) push_flag();
        bool wide = true;
        for (int it = 1; !dead; ++it) {
            if (pst == 0 && (int)((unsigned)rtclock() - due) >= 0) { push_all(); wait_vm<0>(); push_flag(); }
            const int la = wide ? NT - 1 : min(max(na, RA + 1) + 2, NT) - 1, lb = wide ? NT - 1 : min(max(nb, RB + 1) + 4, NT) - 1;
            pollA(la); pollB(lb);
            wait_vm<0>();
            digestA(la + 1); digestB(lb + 1);
            plim = 0;
            if (__builtin_amdgcn_readfirstlane(rdreg<R_PA>()) == call) exa = (int)(short)__builtin_amdgcn_readfirstlane((int)rdreg<R_PA + 1>());
            if (RA >= na && RB >= nb) break;
            wide = false;
            const unsigned long long now = rtclock();
            if ((it & 15) == 0 || now - t0 > DEADLINE_TICKS) dead = give_up(tmo, call, t0);
            __builtin_amdgcn_s_sleep(SPINSLEEP);
        }
        if (pst == 0) retime(tcur);
    };
    acc_zero_all();
    const unsigned ldsb = (unsigned)(uintptr_t)lds + lane * 16, ldsh = ldsb + 65536;
#define GLDS_A(tile, st, i) __builtin_amdgcn_global_load_lds((const unsigned*)(gA + (size_t)(tile) * TB + ((i) >> 2) * 4096 + voff0), (LAS unsigned*)(lds + (st) * TB + wid * 8192 + ((i) >> 2) * 4096), 16, ((i) & 3) * 1024, 0)
#define LOAD_B(buf, tile, i) gload16<R_B + (buf) * 32 + (i) * 4, ((i) & 3) * 1024>(((i) >> 2) ? voff1 : voff0, gB + (size_t)(tile) * TB)
    sfor<8>([&](auto J) __attribute__((always_inline)) { constexpr int j = J; gload16<R_W + 4 * j, 0>(wvoff, wbase + (size_t)(1024 + j) * GN * 4); });
    spin(2, 2, 0);
    sfor<8>([&](auto I) __attribute__((always_inline)) { constexpr int i = I; GLDS_A(0, 0, i); });
    sfor<8>([&](auto I) __attribute__((always_inline)) { constexpr int i = I; LOAD_B(0, 0, i); });
    sfor<8>([&](auto I) __attribute__((always_inline)) { constexpr int i = I; GLDS_A(1, 1, i); });
    sfor<8>([&](auto I) __attribute__((always_inline)) { constexpr int i = I; LOAD_B(1, 1, i); });
    pollA(NT - 1); pollB(NT - 1); plim = NT;
    wait_vm<18>();
    __builtin_amdgcn_s_barrier();
    STAMP(5);
    sfor<APF>([&](auto G) __attribute__((always_inline)) { constexpr int g = G; ldsrd<R_AF + 4 * g, g * 1024>(ldsb, ldsh); });
    evt = pst == 5 ? 1000 : tsend;
    limits();
    float wmax = 0.f; unsigned wfl = 0;
    auto tile = [&](auto PH_, auto LASTP_, const int t) __attribute__((always_inline)) {
        constexpr int PH = PH_; constexpr bool LASTP = LASTP_;
        constexpr int cur = PH & 3, Q1 = (PH + 7) & 7, Q2 = (PH + 6) & 7;
        constexpr bool W1 = !(LASTP && PH > 0), W2 = !(LASTP && PH > 1);
        constexpr bool FIN = LASTP && PH == 4;
        constexpr int e1 = FIN ? 8 : (!LASTP && PH == 6 ? 8 : 0);
        constexpr int e1p = (W1 && Q1 == 6 ? 8 : 0), e2p = (W1 && Q1 == 0 ? 4 : 0) + (W1 && Q1 == 2 ? 1 : 0) + ((Q1 & 1) ? 2 : 0);
        constexpr int e2pp = (W2 && Q2 == 0 ? 4 : 0) + (W2 && Q2 == 2 ? 1 : 0) + ((Q2 & 1) ? 2 : 0);
        constexpr int base = e2pp + 16 + e1p + e2p;
        if (__builtin_expect(t >= nslow, 0)) {
            if (pst == 1) { wait_vm<16 + e1p + e2p>(); push_flag(); } else wait_vm<base>();
            const int nn = FIN ? NT : t + 3;
            if (RA < nn || RB < nn) spin(nn, nn, t);
            if (pst == 0 && t >= tsend) push_all();
            evt = pst == 5 ? 1000 : (pst == 0 ? tsend : t + 1);
            limits();
        } else wait_vm<base>();
        const int slab = min((t >> 3) + 2, 7);
        sfor<32>([&](auto G) __attribute__((always_inline)) { constexpr int g = G;
            constexpr int RC = (g & 15) * 16, RAF = R_AF + 4 * (g & 7), RBF = R_B + cur * 32 + (g >> 4) * 16;
            if constexpr (g == 16) { wait_vm<e1p + 8 + e2p + 8 + e1>(); __builtin_amdgcn_s_barrier(); }
            constexpr int N0 = (SNAKE && (g & 1)) ? 12 : 0, DN = (SNAKE && (g & 1)) ? -4 : 4;
            mfma1w<RC + N0, (FIN && g + APF > 32) ? (32 - g - 1) : APF - 1, RAF, RBF + N0>();
            if constexpr (g + APF < 32) ldsrd<R_AF + 4 * ((g + APF) & 7), (PH & 3) * TB + (g + APF) * 1024>(ldsb, ldsh);
            else if constexpr (!FIN) ldsrd<R_AF + 4 * ((g + APF) & 7), ((PH + 1) & 3) * TB + (g + APF - 32) * 1024>(ldsb, ldsh);
            mfma1<RC + N0 + DN, RAF, RBF + N0 + DN>();
            if constexpr ((PH & 1) == 1 && g == 25) digestA(plim);
            if constexpr ((PH & 1) == 1 && g == 26) digestB(plim);
            if constexpr ((PH & 1) == 1 && g == 27) limits();
            if constexpr (!LASTP && PH == 0 && g >= 8 && g < 16) { if constexpr (g == 8) wmax = 0.f; amax4r<R_W + 4 * (g - 8)>(wmax); }
            if constexpr (!LASTP && PH == 0 && g == 17) { const float a = wmax * sw; wfl = call4 + ((__ballot(!(a < 60000.0f)) ? 1u : 0u) | (__ballot(a >= 0.015625f) ? 2u : 0u)); }
            u32x4 wq;
            if constexpr (!LASTP && PH == 0 && g >= 25 && g < 29) wq = cvt8r<R_W + (g - 25), 4>(sw);
            mfma1<RC + N0 + 2 * DN, RAF, RBF + N0 + 2 * DN>();
            if constexpr (g < 8) GLDS_A(t + 2, (PH + 2) & 3, g);
                        if constexpr (!LASTP && PH == 6 && g >= 8 && g < 16) gload16<R_W + 4 * (g - 8), 0>(wvoff, wbase + ((size_t)min(slab + 1, 7) * 512 + (g - 8)) * GN * 4);
            if constexpr (FIN && g >= 8 && g < 16) LOAD_B(3, t + 3, g - 8);
            if constexpr (g >= 17 && g < 25) LOAD_B((cur + 2) & 3, t + 2, g - 17);
            if constexpr (FIN && g >= 24) GLDS_A(t + 3, 3, g - 24);
            if constexpr (FIN && g >= 26 && g < 30) gload16_sys<R_W + 4 * (g - 26)>(lane * 16 + (g - 26) * 1024, fA);
            if constexpr (FIN && g == 25) gload8_agent<R_PB>(lane * 8, fB);
            if constexpr (!LASTP && PH == 0 && g >= 25 && g < 29) gstore16_agent<(g - 25) * 1024>(voff0, wq, bdst + (size_t)slab * 8 * TB);
            if constexpr (!LASTP && PH == 2 && g == 25) gstore4_agent(0u, wfl, fbdst + slab * 64);
            if constexpr ((PH & 1) == 1 && g == 29) { pollA(NT - 1); plim = NT; }
            if constexpr ((PH & 1) == 1 && g == 30) pollB(NT - 1);
            mfma1<RC + N0 + 3 * DN, RAF, RBF + N0 + 3 * DN>();
        });
    };
    for (int s = 0; s < 7; ++s) {
        sfor<8>([&](auto PH) __attribute__((always_inline)) { tile(PH, std::false_type{}, s * 8 + PH); });
    }
    sfor<5>([&](auto PH) __attribute__((always_inline)) { tile(PH, std::true_type{}, 56 + PH); });
    if (pst == 0) push_all();
    wait_vm<0>();
    if (pst != 5) push_flag();
    __builtin_amdgcn_s_barrier();
    STAMP(8);
    bool wbad;
    { const unsigned o = rdreg<R_PB>() | rdreg<R_PB + 1>(); wbad = __ballot(o & 1u) != 0ull || __ballot(o & 2u) == 0ull;
      unsigned xo = 0; sfor<8>([&](auto I) __attribute__((always_inline)) { xo |= rdreg<R_W + 2 * I + 1>(); }); wbad = wbad || __ballot(xo & 0x10000u) != 0ull; }
    float fx = exp2i(-exa), fw = exp2i(-ew);
    bool bad = false;
    float* cp = p.out + (size_t)(pm * 256 + fq * 4) * GN + pn * 256 + wid * 64 + fr * 4;
    auto store_rows = [&](auto M_) __attribute__((always_inline)) { constexpr int m = M_;
        sfor<4>([&](auto J) __attribute__((always_inline)) { constexpr int j = J;
            f32x4 v = acc_read_row<m * 16 + j>();
#pragma unroll
            for (int e = 0; e < 4; ++e) { float y = v[e]; y = y < 0.f ? 0.f : y; v[e] = bad ? __builtin_nanf("") : (y * fx) * fw; }
            __builtin_nontemporal_store(v, (f32x4*)(cp + (size_t)(m * 16 + j) * GN));
        });
    };
    if (!wbad) {
        bad = dead || load_word(tmo) == call;
        sfor<APF>([&](auto I) __attribute__((always_inline)) { constexpr int i = I, m = i / 6, q = i % 6; ldsrd<R_AF + 4 * (i & 7), (1 + (q >> 1)) * TB + ((q & 1) * 16 + m) * 1024>(ldsb, ldsh); });
        sfor<96>([&](auto I) __attribute__((always_inline)) { constexpr int i = I, m = i / 6, q = i % 6, i2 = i + APF, m2 = i2 / 6, q2 = i2 % 6;
            if constexpr (i2 < 96) ldsrd<R_AF + 4 * (i2 & 7), (1 + (q2 >> 1)) * TB + ((q2 & 1) * 16 + m2) * 1024>(ldsb, ldsh);
            mfma_group<m * 16, (i2 < 96) ? APF : (95 - i), R_AF + 4 * (i & 7), R_B + (1 + (q >> 1)) * 32 + (q & 1) * 16>();
            if constexpr (q == 1 && m > 0) store_rows(std::integral_constant<int, m - 1>{});
        });
        asm volatile("s_nop 15\n\ts_nop 15" ::: "memory");
        store_rows(std::integral_constant<int, 15>{});
    } else {
        bad = dead || load_word(tmo) == call;
        asm volatile("s_nop 15\n\ts_nop 15" ::: "memory");
        acc_zero_all();
        const _Float16* ax = (const _Float16*)(wsl + abuf + (size_t)pm * PANEL_B);
        const float* x32 = (const float*)(wsl + WS_X32 + (size_t)(call & 1u) * 2 * OPER_B) + (size_t)(pm * 256 + fr) * GK;
        const unsigned* fl = (const unsigned*)fA;
        const float* wq = p.w + pn * 256 + wid * 64 + fr * 4;
        for (int k0 = 0; k0 < GK; k0 += 4) {
            const int k = k0 + fq;
            const f32x4 b = *(const f32x4*)(wq + (size_t)k * GN);
            const _Float16* ap = ax + (size_t)(k >> 5) * 8192 + (size_t)(((k >> 3) & 3) * 16 + fr) * 8 + (k & 7);
            sfor<4>([&](auto Q_) __attribute__((always_inline)) { constexpr int q = Q_;
                const bool f32src = (__hip_atomic_load(fl + ((k0 >> 5) * 4 + q) * 2 + 1, __ATOMIC_RELAXED, __HIP_MEMORY_SCOPE_SYSTEM) & 0x10000u) != 0u;
                sfor<4>([&](auto M_) __attribute__((always_inline)) { constexpr int m = q * 4 + M_;
                    const float a = f32src ? __hip_atomic_load(x32 + (size_t)m * 16 * GK + k, __ATOMIC_RELAXED, __HIP_MEMORY_SCOPE_SYSTEM) : (float)ap[m * 512] * fx;
                    mfma_f32<m * 16>(a, b[0]); mfma_f32<m * 16 + 4>(a, b[1]); mfma_f32<m * 16 + 8>(a, b[2]); mfma_f32<m * 16 + 12>(a, b[3]);
                });
            });
        }
        fx = 1.0f; fw = 1.0f;
        asm volatile("s_nop 15\n\ts_nop 15" ::: "memory");
        sfor<16>([&](auto M_) __attribute__((always_inline)) { store_rows(M_); });
    }
    asm volatile(".Lagk_end:" ::: "memory");
    STAMP(9);
}

}

extern "C" void kernel_launch(int rank, int nranks, ncclComm_t comm,
                              hipStream_t stream,
                              void* const* d_in, const int* in_sizes, int n_in,
                              void* d_out, int out_size,
                              const int* in_dtypes, int out_dtype,
                              void* const* d_out_all,
                              void* const* d_ws_all, size_t ws_bytes) {
    if (nranks != WORLD || rank < 0 || rank >= WORLD || ws_bytes < WS_END || n_in < 2) { fprintf(stderr, "unsupported configuration\n"); return; }
    static bool init[WORLD] = {};
    if (!init[rank]) {
        HIP_CHECK(hipFuncSetAttribute((const void*)fused_kernel, hipFuncAttributeMaxDynamicSharedMemorySize, LDS_BYTES));
        init[rank] = true;
    }
    Params p;
    p.x = (const float*)d_in[0]; p.w = (const float*)d_in[1]; p.out = (float*)d_out; p.rank = rank;
    for (int i = 0; i < WORLD; ++i) p.ws[i] = (char*)d_ws_all[(rank + i) & 7];
    hipLaunchKernelGGL(fused_kernel, dim3(GRID), dim3(NTHREADS), LDS_BYTES, stream, p);
    HIP_CHECK(hipGetLastError());
}
```

```cpp
#include <hip/hip_runtime.h>
#include <rccl/rccl.h>
#include <cstdio>
#include <cstdint>
#include <utility>

#define HIP_CHECK(expr) do { hipError_t _e = (expr); if (_e != hipSuccess) { \
    fprintf(stderr, "HIP error %s at %s:%d\n", hipGetErrorString(_e), __FILE__, __LINE__); \
    return; } } while (0)

namespace {

template <int... Is, class F> __device__ __forceinline__ void sfor_impl(std::integer_sequence<int, Is...>, F&& f) { (f(std::integral_constant<int, Is>{}), ...); }
template <int N, class F> __device__ __forceinline__ void sfor(F&& f) { sfor_impl(std::make_integer_sequence<int, N>{}, f); }

typedef _Float16 half8 __attribute__((ext_vector_type(8)));
typedef float f32x4 __attribute__((ext_vector_type(4)));
typedef unsigned u32x4 __attribute__((ext_vector_type(4)));
typedef unsigned u32x2 __attribute__((ext_vector_type(2)));
#define LAS __attribute__((address_space(3)))

constexpr int WORLD = 8;
constexpr int GM = 4096, GK = 4096, GN = 4096;
constexpr int MS = GM / WORLD;
constexpr int NT = GK / 64;
constexpr int TB = 32768;
constexpr size_t PANEL_B = (size_t)NT * TB;
constexpr size_t OPER_B = 16 * PANEL_B;
constexpr size_t WS_CTL = 0;
constexpr size_t WS_SLOT = 4096;
constexpr size_t WS_FLAGA = 65536;
constexpr size_t WS_FLAGB = 262144;
constexpr size_t WS_DBG = 512u << 10;
constexpr size_t WS_A0 = 1u << 20;
constexpr size_t WS_B = WS_A0 + 2 * OPER_B;
constexpr size_t WS_X32 = WS_B + OPER_B;
constexpr size_t WS_END = WS_X32 + 4 * OPER_B;
constexpr int CW_TICKET = 0, CW_TMO = 256;
constexpr unsigned long long DEADLINE_TICKS = 50000000ull;
constexpr int LDS_X = 4 * TB;
constexpr int LDS_BYTES = 4 * TB + 16384;
constexpr int GRID = 256;
#define NTHREADS 256
#define PACE 68
#define PWIN 16
#define KNOW 6
#define K1 24
#define P1 56
#define POFF 0
#define SNAKE 1
#define CLOSED 0
#define FLAG1 1
#define KEARLY 8
#define KBURST 128
#define TILE_TICKS 145
#define SLOPE 43
#define OFFL 230
#define OFFR 440
#define ABL_NOPOLL 0
#define ABL_NOW 0
#define ABL_NOPUSH 0
#define SPINSLEEP 8
#define POLLP 1
#define PAHEAD 10
#define PBHEAD 7
#define APF 5

struct Params { const float* x; const float* w; float* out; char* ws[WORLD]; int rank; };

template <class T> __device__ __forceinline__ T* uptr(T* q) {
    unsigned long long v = (unsigned long long)q;
    unsigned lo = __builtin_amdgcn_readfirstlane((unsigned)v), hi = __builtin_amdgcn_readfirstlane((unsigned)(v >> 32));
    return (T*)(((unsigned long long)hi << 32) | lo);
}
__device__ __forceinline__ u32x4 cvt8(const f32x4 a, const f32x4 b, float s) {
    half8 h;
    h[0] = (_Float16)(a[0] * s); h[1] = (_Float16)(a[1] * s); h[2] = (_Float16)(a[2] * s); h[3] = (_Float16)(a[3] * s);
    h[4] = (_Float16)(b[0] * s); h[5] = (_Float16)(b[1] * s); h[6] = (_Float16)(b[2] * s); h[7] = (_Float16)(b[3] * s);
    return __builtin_bit_cast(u32x4, h);
}
__device__ __forceinline__ unsigned absbits(float v) { return __float_as_uint(v) & 0x7fffffffu; }
__device__ __forceinline__ unsigned amax4(const f32x4 a) { return max(max(absbits(a[0]), absbits(a[1])), max(absbits(a[2]), absbits(a[3]))); }
__device__ __forceinline__ unsigned wave_umax(unsigned v) {
    v = max(v, (unsigned)__builtin_amdgcn_update_dpp(0, (int)v, 0x111, 0xf, 0xf, false));
    v = max(v, (unsigned)__builtin_amdgcn_update_dpp(0, (int)v, 0x112, 0xf, 0xf, false));
    v = max(v, (unsigned)__builtin_amdgcn_update_dpp(0, (int)v, 0x114, 0xf, 0xf, false));
    v = max(v, (unsigned)__builtin_amdgcn_update_dpp(0, (int)v, 0x118, 0xf, 0xf, false));
    v = max(v, (unsigned)__builtin_amdgcn_update_dpp(0, (int)v, 0x142, 0xa, 0xf, false));
    v = max(v, (unsigned)__builtin_amdgcn_update_dpp(0, (int)v, 0x143, 0xc, 0xf, false));
    return (unsigned)__builtin_amdgcn_readlane((int)v, 63);
}
__device__ __forceinline__ int scale_exp(unsigned maxbits, int target) {
    const int ef = (int)(maxbits >> 23);
    if (ef == 0 || ef == 255) return 0;
    int e = target - (ef - 127);
    return e > 126 ? 126 : (e < -126 ? -126 : e);
}
__device__ __forceinline__ float exp2i(int e) { return __uint_as_float((unsigned)(e + 127) << 23); }

constexpr int R_PB = 58, R_PA = 60, R_W = 64, R_AF = 96, R_B = 128, R_X = 128, R_S = 96, R_SL = 112, R_W2 = 160;
#define OWNED_REGS "a0","a1","a2","a3","a4","a5","a6","a7","a8","a9","a10","a11","a12","a13","a14","a15","a16","a17","a18","a19","a20","a21","a22","a23","a24","a25","a26","a27","a28","a29","a30","a31","a32","a33","a34","a35","a36","a37","a38","a39","a40","a41","a42","a43","a44","a45","a46","a47","a48","a49","a50","a51","a52","a53","a54","a55","a56","a57","a58","a59","a60","a61","a62","a63","a64","a65","a66","a67","a68","a69","a70","a71","a72","a73","a74","a75","a76","a77","a78","a79","a80","a81","a82","a83","a84","a85","a86","a87","a88","a89","a90","a91","a92","a93","a94","a95","a96","a97","a98","a99","a100","a101","a102","a103","a104","a105","a106","a107","a108","a109","a110","a111","a112","a113","a114","a115","a116","a117","a118","a119","a120","a121","a122","a123","a124","a125","a126","a127","a128","a129","a130","a131","a132","a133","a134","a135","a136","a137","a138","a139","a140","a141","a142","a143","a144","a145","a146","a147","a148","a149","a150","a151","a152","a153","a154","a155","a156","a157","a158","a159","a160","a161","a162","a163","a164","a165","a166","a167","a168","a169","a170","a171","a172","a173","a174","a175","a176","a177","a178","a179","a180","a181","a182","a183","a184","a185","a186","a187","a188","a189","a190","a191","a192","a193","a194","a195","a196","a197","a198","a199","a200","a201","a202","a203","a204","a205","a206","a207","a208","a209","a210","a211","a212","a213","a214","a215","a216","a217","a218","a219","a220","a221","a222","a223","a224","a225","a226","a227","a228","a229","a230","a231","a232","a233","a234","a235","a236","a237","a238","a239","a240","a241","a242","a243","a244","a245","a246","a247","a248","a249","a250","a251","a252","a253","a254","a255","v58","v59","v60","v61","v62","v63","v64","v65","v66","v67","v68","v69","v70","v71","v72","v73","v74","v75","v76","v77","v78","v79","v80","v81","v82","v83","v84","v85","v86","v87","v88","v89","v90","v91","v92","v93","v94","v95","v96","v97","v98","v99","v100","v101","v102","v103","v104","v105","v106","v107","v108","v109","v110","v111","v112","v113","v114","v115","v116","v117","v118","v119","v120","v121","v122","v123","v124","v125","v126","v127","v128","v129","v130","v131","v132","v133","v134","v135","v136","v137","v138","v139","v140","v141","v142","v143","v144","v145","v146","v147","v148","v149","v150","v151","v152","v153","v154","v155","v156","v157","v158","v159","v160","v161","v162","v163","v164","v165","v166","v167","v168","v169","v170","v171","v172","v173","v174","v175","v176","v177","v178","v179","v180","v181","v182","v183","v184","v185","v186","v187","v188","v189","v190","v191","v192","v193","v194","v195","v196","v197","v198","v199","v200","v201","v202","v203","v204","v205","v206","v207","v208","v209","v210","v211","v212","v213","v214","v215","v216","v217","v218","v219","v220","v221","v222","v223","v224","v225","v226","v227","v228","v229","v230","v231","v232","v233","v234","v235","v236","v237","v238","v239","v240","v241","v242","v243","v244","v245","v246","v247","v248","v249","v250","v251","v252","v253","v254","v255"
#define ALL_AGPRS "a0","a1","a2","a3","a4","a5","a6","a7","a8","a9","a10","a11","a12","a13","a14","a15","a16","a17","a18","a19","a20","a21","a22","a23","a24","a25","a26","a27","a28","a29","a30","a31","a32","a33","a34","a35","a36","a37","a38","a39","a40","a41","a42","a43","a44","a45","a46","a47","a48","a49","a50","a51","a52","a53","a54","a55","a56","a57","a58","a59","a60","a61","a62","a63","a64","a65","a66","a67","a68","a69","a70","a71","a72","a73","a74","a75","a76","a77","a78","a79","a80","a81","a82","a83","a84","a85","a86","a87","a88","a89","a90","a91","a92","a93","a94","a95","a96","a97","a98","a99","a100","a101","a102","a103","a104","a105","a106","a107","a108","a109","a110","a111","a112","a113","a114","a115","a116","a117","a118","a119","a120","a121","a122","a123","a124","a125","a126","a127","a128","a129","a130","a131","a132","a133","a134","a135","a136","a137","a138","a139","a140","a141","a142","a143","a144","a145","a146","a147","a148","a149","a150","a151","a152","a153","a154","a155","a156","a157","a158","a159","a160","a161","a162","a163","a164","a165","a166","a167","a168","a169","a170","a171","a172","a173","a174","a175","a176","a177","a178","a179","a180","a181","a182","a183","a184","a185","a186","a187","a188","a189","a190","a191","a192","a193","a194","a195","a196","a197","a198","a199","a200","a201","a202","a203","a204","a205","a206","a207","a208","a209","a210","a211","a212","a213","a214","a215","a216","a217","a218","a219","a220","a221","a222","a223","a224","a225","a226","a227","a228","a229","a230","a231","a232","a233","a234","a235","a236","a237","a238","a239","a240","a241","a242","a243","a244","a245","a246","a247","a248","a249","a250","a251","a252","a253","a254","a255"
template <int R, int OFF> __device__ __forceinline__ void gload16(unsigned voff, const char* sbase) {
    asm volatile("global_load_dwordx4 v[%c2:%c2+3], %0, %1 offset:%c3" :: "v"(voff), "s"(sbase), "i"(R), "i"(OFF) : "memory");
}
template <int R> __device__ __forceinline__ void gload16_sys(unsigned voff, const char* sbase) {
    asm volatile("global_load_dwordx4 v[%c2:%c2+3], %0, %1 sc0 sc1" :: "v"(voff), "s"(sbase), "i"(R) : "memory");
}
template <int R> __device__ __forceinline__ void gload16_agent(unsigned voff, const char* sbase) {
    asm volatile("global_load_dwordx4 v[%c2:%c2+3], %0, %1 sc1" :: "v"(voff), "s"(sbase), "i"(R) : "memory");
}
template <int R> __device__ __forceinline__ void gload8_agent(unsigned voff, const char* sbase) {
    asm volatile("global_load_dwordx2 v[%c2:%c2+1], %0, %1 sc1" :: "v"(voff), "s"(sbase), "i"(R) : "memory");
}
template <int R> __device__ __forceinline__ unsigned rdreg() { unsigned v; asm volatile("v_mov_b32 %0, v%c1" : "=v"(v) : "i"(R)); return v; }
template <int OFF> __device__ __forceinline__ void gstore16_sys(unsigned voff, const u32x4& v, char* sbase) {
    asm volatile("global_store_dwordx4 %0, %1, %2 offset:%3 sc0 sc1\n\ts_nop 1" :: "v"(voff), "v"(v), "s"(sbase), "i"(OFF) : "memory");
}
template <int OFF> __device__ __forceinline__ void gstore16_agent(unsigned voff, const u32x4& v, char* sbase) {
    asm volatile("global_store_dwordx4 %0, %1, %2 offset:%3 sc1\n\ts_nop 1" :: "v"(voff), "v"(v), "s"(sbase), "i"(OFF) : "memory");
}
__device__ __forceinline__ void gstore64_sys_lane(unsigned voff, const u32x4& v0, const u32x4& v1, const u32x4& v2, const u32x4& v3, unsigned plo, unsigned phi, int ln) {
    const unsigned long long b = ((unsigned long long)(unsigned)__builtin_amdgcn_readlane((int)phi, ln) << 32) | (unsigned)__builtin_amdgcn_readlane((int)plo, ln);
    asm volatile("s_nop 4\n\tglobal_store_dwordx4 %0, %1, %5 sc0 sc1\n\tglobal_store_dwordx4 %0, %2, %5 offset:1024 sc0 sc1\n\t"
                 "global_store_dwordx4 %0, %3, %5 offset:2048 sc0 sc1\n\tglobal_store_dwordx4 %0, %4, %5 offset:3072 sc0 sc1\n\ts_nop 1"
                 :: "v"(voff), "v"(v0), "v"(v1), "v"(v2), "v"(v3), "s"(b) : "memory");
}
template <int R, int OFF> __device__ __forceinline__ void storeown_sys(unsigned voff, char* sbase) {
    asm volatile("s_nop 4\n\tglobal_store_dwordx4 %0, v[%c2:%c2+3], %1 offset:%c3 sc0 sc1\n\ts_nop 1" :: "v"(voff), "s"(sbase), "i"(R), "i"(OFF) : "memory");
}
__device__ __forceinline__ void gstore8_sys(char* addr, const u32x2& v) {
    asm volatile("global_store_dwordx2 %0, %1, off sc0 sc1\n\ts_nop 1" :: "v"(addr), "v"(v) : "memory");
}
__device__ __forceinline__ void gstore8_agent(unsigned voff, const u32x2& v, char* sbase) {
    asm volatile("global_store_dwordx2 %0, %1, %2 sc1\n\ts_nop 1" :: "v"(voff), "v"(v), "s"(sbase) : "memory");
}
__device__ __forceinline__ void gstore4_agent(unsigned voff, unsigned v, char* sbase) {
    asm volatile("global_store_dword %0, %1, %2 sc1\n\ts_nop 1" :: "v"(voff), "v"(v), "s"(sbase) : "memory");
}
__device__ __forceinline__ unsigned long long rtclock() {
    unsigned long long t; asm volatile("s_memrealtime %0\n\ts_waitcnt lgkmcnt(0)" : "=s"(t) :: "memory"); return t;
}
__device__ __forceinline__ unsigned load_word(const char* sbase) {
    unsigned v; asm volatile("global_load_dword %0, %1, %2 sc1\n\ts_waitcnt vmcnt(0)" : "=&v"(v) : "v"(0u), "s"(sbase) : "memory");
    return __builtin_amdgcn_readfirstlane(v);
}
__device__ __forceinline__ int give_up(char* tmo, unsigned call, unsigned long long t0) {
    if (load_word(tmo) == call) return 1;
    if (rtclock() - t0 > DEADLINE_TICKS) { gstore4_agent(0u, call, tmo); return 1; }
    __builtin_amdgcn_s_sleep(1);
    return 0;
}
template <int R, int OFF> __device__ __forceinline__ void ldsrd(unsigned lo, unsigned hi) {
    if constexpr (OFF < 65536) asm volatile("ds_read_b128 v[%c1:%c1+3], %0 offset:%c2" :: "v"(lo), "i"(R), "i"(OFF) : "memory");
    else asm volatile("ds_read_b128 v[%c1:%c1+3], %0 offset:%c2" :: "v"(hi), "i"(R), "i"(OFF - 65536) : "memory");
}
template <int OFF> __device__ __forceinline__ void lds_write16(unsigned a, const u32x4& v) { asm volatile("ds_write_b128 %0, %1 offset:%2" :: "v"(a), "v"(v), "i"(OFF) : "memory"); }
template <int OFF> __device__ __forceinline__ u32x4 lds_read16_wait(unsigned a) { u32x4 v; asm volatile("ds_read_b128 %0, %1 offset:%2\n\ts_waitcnt lgkmcnt(0)" : "=&v"(v) : "v"(a), "i"(OFF) : "memory"); return v; }
template <int N> __device__ __forceinline__ void wait_vm() { asm volatile("s_waitcnt vmcnt(%0)" :: "i"(N) : "memory"); }
template <int R> __device__ __forceinline__ void amax4r(float& m) {
    asm volatile("v_max3_f32 %0, |v%c1|, |v[%c1+1]|, %0\n\tv_max3_f32 %0, |v[%c1+2]|, |v[%c1+3]|, %0" : "+v"(m) : "i"(R));
}
template <int R, int N> __device__ __forceinline__ unsigned amax_regs() {
    float m = 0.f;
    sfor<N / 4>([&](auto I) __attribute__((always_inline)) { amax4r<R + 4 * I>(m); });
    return __float_as_uint(m);
}
template <int R, int S> __device__ __forceinline__ u32x4 cvt8r(float s) {
    unsigned a, b, c, d;
    asm volatile("v_fma_mixlo_f16 %0, v%c5, %4, 0\n\tv_fma_mixlo_f16 %1, v[%c5+2*%c6], %4, 0\n\tv_fma_mixlo_f16 %2, v[%c5+4*%c6], %4, 0\n\tv_fma_mixlo_f16 %3, v[%c5+6*%c6], %4, 0\n\t"
                 "v_fma_mixhi_f16 %0, v[%c5+%c6], %4, 0\n\tv_fma_mixhi_f16 %1, v[%c5+3*%c6], %4, 0\n\tv_fma_mixhi_f16 %2, v[%c5+5*%c6], %4, 0\n\tv_fma_mixhi_f16 %3, v[%c5+7*%c6], %4, 0"
                 : "=&v"(a), "=&v"(b), "=&v"(c), "=&v"(d) : "v"(s), "i"(R), "i"(S));
    return (u32x4){a, b, c, d};
}
template <int R, int W, int A, int B> __device__ __forceinline__ void mfma_group() {
    asm volatile("s_waitcnt lgkmcnt(%c0)\n\t"
                 "v_mfma_f32_16x16x32_f16 a[%c1:%c1+3], v[%c2:%c2+3], v[%c3:%c3+3], a[%c1:%c1+3]\n\t"
                 "v_mfma_f32_16x16x32_f16 a[%c1+4:%c1+7], v[%c2:%c2+3], v[%c3+4:%c3+7], a[%c1+4:%c1+7]\n\t"
                 "v_mfma_f32_16x16x32_f16 a[%c1+8:%c1+11], v[%c2:%c2+3], v[%c3+8:%c3+11], a[%c1+8:%c1+11]\n\t"
                 "v_mfma_f32_16x16x32_f16 a[%c1+12:%c1+15], v[%c2:%c2+3], v[%c3+12:%c3+15], a[%c1+12:%c1+15]"
                 :: "i"(W), "i"(R), "i"(A), "i"(B) : "memory", ALL_AGPRS);
}
template <int R, int A, int B> __device__ __forceinline__ void mfma1() {
    asm volatile("v_mfma_f32_16x16x32_f16 a[%c0:%c0+3], v[%c1:%c1+3], v[%c2:%c2+3], a[%c0:%c0+3]" :: "i"(R), "i"(A), "i"(B) : "memory");
}
template <int R, int W, int A, int B> __device__ __forceinline__ void mfma1w() {
    asm volatile("s_waitcnt lgkmcnt(%c0)\n\tv_mfma_f32_16x16x32_f16 a[%c1:%c1+3], v[%c2:%c2+3], v[%c3:%c3+3], a[%c1:%c1+3]" :: "i"(W), "i"(R), "i"(A), "i"(B) : "memory", ALL_AGPRS);
}
template <int R0, int R1> __device__ __forceinline__ unsigned long long ready_mask(unsigned thr) {
    unsigned long long m; unsigned t;
    asm volatile("v_min_u32 %1, v%c3, v%c4\n\tv_cmp_le_u32 %0, %2, %1" : "=s"(m), "=&v"(t) : "s"(thr), "i"(R0), "i"(R1));
    return m;
}
template <int R> __device__ __forceinline__ void acc_zero4() {
    asm volatile("v_accvgpr_write_b32 a%c0, 0\n\tv_accvgpr_write_b32 a[%c0+1], 0\n\tv_accvgpr_write_b32 a[%c0+2], 0\n\tv_accvgpr_write_b32 a[%c0+3], 0" :: "i"(R) : ALL_AGPRS);
}
template <int R> __device__ __forceinline__ void acc_zero16(const u32x4& z) {
    asm volatile("v_mfma_f32_32x32x16_f16 a[%c1:%c1+15], %0, %0, 0" :: "v"(z), "i"(R) : ALL_AGPRS);
}
__device__ __forceinline__ void acc_zero_all() {
    u32x4 z = {0u, 0u, 0u, 0u}; asm volatile("s_nop 1" : "+v"(z));
    sfor<16>([&](auto I) __attribute__((always_inline)) { acc_zero16<I * 16>(z); });
    asm volatile("s_nop 15\n\ts_nop 3" ::: "memory");
}
template <int R> __device__ __forceinline__ f32x4 acc_read_row() {
    float a, b, c, d;
    asm volatile("v_accvgpr_read_b32 %0, a%c4\n\tv_accvgpr_read_b32 %1, a[%c4+4]\n\tv_accvgpr_read_b32 %2, a[%c4+8]\n\tv_accvgpr_read_b32 %3, a[%c4+12]"
                 : "=v"(a), "=v"(b), "=v"(c), "=v"(d) : "i"(R));
    return (f32x4){a, b, c, d};
}
template <int R> __device__ __forceinline__ void mfma_f32(float a, float b) {
    asm volatile("s_nop 1\n\tv_mfma_f32_16x16x4_f32 a[%c2:%c2+3], %0, %1, a[%c2:%c2+3]" :: "v"(a), "v"(b), "i"(R) : ALL_AGPRS);
}

#define STAMP(i) do {} while (0)

__global__ __launch_bounds__(256, 1) __attribute__((amdgpu_num_vgpr(58))) void fused_kernel(Params p) {
    extern __shared__ __attribute__((aligned(16))) unsigned char smem[];
    LAS unsigned char* lds = (LAS unsigned char*)smem;
    const int tid = threadIdx.x, wid = __builtin_amdgcn_readfirstlane(tid >> 6), lane = tid & 63;
    const int fr = lane & 15, fq = lane >> 4;
    const int rank = p.rank;
    char* wsl = uptr(p.ws[0]);
    unsigned* ctl = (unsigned*)(wsl + WS_CTL);
    char* tmo = wsl + WS_CTL + CW_TMO * 4;
    asm volatile("" ::: OWNED_REGS);
    STAMP(0);
    const int bid = blockIdx.x, xcd = bid & 7, idx = bid >> 3;
    const int pm = (idx >> 2) * 2 + (xcd & 1), pn = (xcd >> 1) * 4 + (idx & 3);
    const bool local = (pm >> 1) == rank;
    const int jwg = local ? 224 + (idx & 3) * 8 + xcd : (((idx >> 2) < rank ? (idx >> 2) : (idx >> 2) - 1) * 4 + (idx & 3)) * 8 + xcd;
    const int unit = (jwg >> 2) + 64 * wid, uq = jwg & 3;
    const int ukh = unit >> 1, upl = unit & 1, upm = rank * 2 + upl;
    const bool burst = ukh < KBURST;

    const unsigned voff0 = lane * 16, voff1 = lane * 16 + 4096;
    unsigned ticket = 0;
    if (lane == 0) asm volatile("global_atomic_add %0, %1, %2, %3 sc0" : "=v"(ticket) : "v"(0u), "v"(1u), "s"(wsl + WS_CTL + CW_TICKET * 4 + wid * 128) : "memory");
    const char* xb = (const char*)uptr(p.x) + ((size_t)(upl * 256 + uq * 64) * GK + ukh * 32) * 4;
    const unsigned xv = (unsigned)(fr * GK + fq * 8) * 4u;
    sfor<4>([&](auto I) __attribute__((always_inline)) { constexpr int i = I; gload16<R_S + 16 + 4 * i, 0>((unsigned)((lane * 8 + i * 2 + 1) * GK + ((lane * 68 + i * 1028) & (GK - 1))) * 4u, (const char*)uptr(p.x)); });
    sfor<4>([&](auto I) __attribute__((always_inline)) { constexpr int i = I; gload16<R_X + 8 * i, 0>(xv, xb + (size_t)i * 16 * GK * 4); gload16<R_X + 8 * i + 4, 16>(xv, xb + (size_t)i * 16 * GK * 4); });
    const unsigned wvoff = (unsigned)(fq * 8 * GN + fr * 4) * 4u;
    const char* wbase = (const char*)uptr(p.w) + ((size_t)pm * 32 * GN + pn * 256 + wid * 64) * 4;
    sfor<4>([&](auto I) __attribute__((always_inline)) { constexpr int i = I; gload16<R_S + 4 * i, 0>((unsigned)((lane * 64 + i * 16 + 5) * GN + ((lane * 68 + i * 1028) & (GN - 1))) * 4u, (const char*)uptr(p.w)); });
    sfor<8>([&](auto J) __attribute__((always_inline)) { constexpr int j = J; gload16<R_W + 4 * j, 0>(wvoff, wbase + (size_t)j * GN * 4); });
    {
        unsigned long long pc; unsigned sz;
        asm volatile("s_getpc_b64 %0\n.Lagk_begin:\n\ts_mov_b32 %1, .Lagk_end-.Lagk_begin" : "=s"(pc), "=s"(sz));
        gload16<R_B + 124, 0>(min((unsigned)(idx * 4 + wid) * 1024u, sz - 1024u) + lane * 16, (const char*)pc);
    }
    const int r0 = (unit * 4 + uq * 2) % 7;
    char* fdst = wsl;
#pragma unroll
    for (int i = 1; i < WORLD; ++i) { char* q = p.ws[i]; fdst = (lane & 7) == i ? q : fdst; }
    STAMP(15);
    asm volatile("s_waitcnt vmcnt(13)" : "+v"(ticket) :: "memory");
    STAMP(14);
    const unsigned call = __builtin_amdgcn_readfirstlane(ticket) / GRID + 1u;
    const size_t abuf = WS_A0 + (size_t)(call & 1u) * OPER_B;
    const size_t faoff = WS_FLAGA + (size_t)(call & 1u) * 65536;
    int dead = 0;
    const int ex = scale_exp(wave_umax(amax_regs<R_S + 16, 16>()), 4);
    const unsigned tx0 = (unsigned)rtclock();
    STAMP(3);
    u32x4 xq[4];
    bool xbad;
    {
        const float sx = exp2i(ex);
        sfor<4>([&](auto I) __attribute__((always_inline)) { constexpr int i = I; xq[i] = cvt8r<R_X + 8 * i, 1>(sx); });
        const float a = __uint_as_float(amax_regs<R_X, 32>()) * sx;
        xbad = __ballot(!(a < 60000.0f)) != 0ull || (__ballot(a >= 0.000244140625f) == 0ull && __ballot(a != 0.0f) != 0ull);
    }
    const unsigned xoff = (unsigned)(abuf + (size_t)upm * PANEL_B + (size_t)ukh * 16384 + uq * 4096 + lane * 16);
    char* fladdr = fdst + faoff + (((size_t)upm * 128 + ukh) * 4 + uq) * 8;
    const u32x2 flval = {call, (unsigned)(ex & 0xffff) | (xbad ? 0x10000u : 0u)};
    const unsigned plo = (unsigned)(unsigned long long)fdst, phi = (unsigned)((unsigned long long)fdst >> 32);
    const unsigned xlds = (unsigned)(uintptr_t)lds + LDS_X + wid * 4096 + lane * 16;
    sfor<4>([&](auto I) __attribute__((always_inline)) { constexpr int i = I; lds_write16<i * 1024>(xlds, xq[i]); });
    int pst = 0;
    auto push_all = [&]() __attribute__((always_inline)) {
        STAMP(11);
        const u32x4 v0 = lds_read16_wait<0>(xlds), v1 = lds_read16_wait<1024>(xlds), v2 = lds_read16_wait<2048>(xlds), v3 = lds_read16_wait<3072>(xlds);
        int q = r0;
#pragma unroll 1
        for (int d = 0; d < 7; ++d) {
            gstore64_sys_lane(xoff, v0, v1, v2, v3, plo, phi, q + 1);
            q = q == 6 ? 0 : q + 1;
        }
        pst = 1;
    };
    auto push_flag = [&]() __attribute__((always_inline)) { if (lane >= 1 && lane < 8) gstore8_sys(fladdr, flval); pst = 5; STAMP(12); };
    const unsigned due = tx0 + (unsigned)(ukh < KNOW ? 0 : ukh * PACE);
    if (__builtin_expect(xbad, 0)) {
        const unsigned so = (unsigned)(WS_X32 + (size_t)(call & 1u) * 2 * OPER_B + ((size_t)(upm * 256 + uq * 64) * GK + ukh * 32) * 4) + xv;
#pragma unroll 1
        for (int d = 0; d < WORLD; ++d) {
            const unsigned long long bq = ((unsigned long long)(unsigned)__builtin_amdgcn_readlane((int)phi, d) << 32) | (unsigned)__builtin_amdgcn_readlane((int)plo, d);
            sfor<8>([&](auto I) __attribute__((always_inline)) { constexpr int i = I; storeown_sys<R_X + 4 * i, (i & 1) * 16>(so + (unsigned)(i >> 1) * 16u * GK * 4u, (char*)bq); });
        }
        wait_vm<0>();
    }
    if (ukh < KNOW) push_all();
    sfor<4>([&](auto I) __attribute__((always_inline)) { constexpr int i = I; gstore16_sys<i * 1024>(xoff, xq[i], wsl); });
    sfor<8>([&](auto J) __attribute__((always_inline)) { constexpr int j = J; gload16<R_W2 + 4 * j, 0>(wvoff, wbase + (size_t)(512 + j) * GN * 4); });
    if (ukh < KNOW) wait_vm<41>(); else wait_vm<13>();
    const int ew = scale_exp(wave_umax(amax_regs<R_S, 16>()), 0);
    const float sw = exp2i(ew);
    char* bdst = wsl + WS_B + (size_t)pn * PANEL_B + (size_t)(pm >> 1) * TB + wid * 8192 + (pm & 1) * 4096;
    char* fbdst = wsl + WS_FLAGB + ((size_t)(pn * 4 + wid) * 128 + pm) * 4;
    auto wbits = [&]() __attribute__((always_inline)) -> unsigned {
        const float a = __uint_as_float(amax_regs<R_W, 32>()) * sw;
        return call * 4u + ((__ballot(!(a < 60000.0f)) ? 1u : 0u) | (__ballot(a >= 0.015625f) ? 2u : 0u));
    };
    auto wstore = [&](auto C, char* dst) __attribute__((always_inline)) { constexpr int c = C; gstore16_agent<c * 1024>(voff0, cvt8r<R_W + c, 4>(sw), dst); };
    sfor<4>([&](auto C) __attribute__((always_inline)) { wstore(C, bdst); });
    const unsigned fb0 = wbits();
    wait_vm<0>();
    gstore4_agent(0u, fb0, fbdst);
    if (lane == 0) gstore8_sys(fladdr, flval);
    if (pst == 1) push_flag();
    if (pst == 0 && ukh < KEARLY) {
        while ((int)((unsigned)rtclock() - due) < 0) __builtin_amdgcn_s_sleep(1);
        push_all(); wait_vm<0>(); push_flag();
    }
    {
        sfor<4>([&](auto C) __attribute__((always_inline)) { constexpr int c = C; gstore16_agent<c * 1024>(voff0, cvt8r<R_W2 + c, 4>(sw), bdst + 8 * TB); });
        const float a = __uint_as_float(amax_regs<R_W2, 32>()) * sw;
        const unsigned f1 = call * 4u + ((__ballot(!(a < 60000.0f)) ? 1u : 0u) | (__ballot(a >= 0.015625f) ? 2u : 0u));
        wait_vm<0>();
        gstore4_agent(64u, f1, fbdst);
    }
    STAMP(4);

    const char* gA = wsl + abuf + (size_t)pm * PANEL_B + wid * 8192;
    const char* gB = wsl + WS_B + (size_t)pn * PANEL_B + wid * 8192;
    const char* fA = wsl + faoff + (size_t)pm * 4096;
    const char* fB = wsl + WS_FLAGB + (size_t)(pn * 4 + wid) * 512;
    int RA = 0, RB = 0, exa = 0;
    int plim = 0;
    int qa = 0, qb = 0;
    int tsend = 0;
    int evt = 0;
    int nslow = 0;
    const unsigned call4 = call * 4u;
    auto pollA = [&](int last) __attribute__((always_inline)) { qa = RA; last = min(last, qa + PWIN - 1);
        gload16_sys<R_PA>((unsigned)min(qa + (lane >> 2), last) * 64u + (lane & 3) * 16u, fA);
    };
    auto pollB = [&](int last) __attribute__((always_inline)) { qb = RB; gload8_agent<R_PB>((unsigned)min(qb + fr, last) * 8u, fB); };
    auto digestA = [&](int lim) __attribute__((always_inline)) { const unsigned long long nm = ~ready_mask<R_PA, R_PA + 2>(call); RA = max(RA, min(qa + min(nm ? (int)__builtin_ctzll(nm) >> 2 : 16, PWIN), lim)); };
    auto digestB = [&](int lim) __attribute__((always_inline)) { RB = max(RB, min(qb + (int)__builtin_ctz(~(unsigned)ready_mask<R_PB, R_PB + 1>(call4) | 0x10000u), lim)); };
    auto limits = [&]() __attribute__((always_inline)) { nslow = min(min(RA >= NT ? 1000 : RA - 3, RB >= NT ? 1000 : min(RB - 2, NT - 4)), evt); };
    auto retime = [&](int tcur) __attribute__((always_inline)) {
        const int rem = (int)(due - (unsigned)rtclock());
        tsend = rem <= TILE_TICKS / 2 ? tcur : tcur + max(1, CLOSED ? (rem * 7) / (8 * TILE_TICKS) : (rem + TILE_TICKS / 2) / TILE_TICKS);
    };
    auto spin = [&](int na, int nb, int tcur) __attribute__((always_inline)) {
        const unsigned long long t0 = rtclock();
        wait_vm<0>();
        if (pst == 1 || pst == 2) push_flag();
        bool wide = true;
        for (int it = 1; !dead; ++it) {
            if (pst == 0 && (int)((unsigned)rtclock() - due) >= 0) { push_all(); wait_vm<0>(); push_flag(); }
            const int la = wide ? NT - 1 : min(max(na, RA + 1) + 2, NT) - 1, lb = wide ? NT - 1 : min(max(nb, RB + 1) + 4, NT) - 1;
            pollA(la); pollB(lb);
            wait_vm<0>();
            digestA(la + 1); digestB(lb + 1);
            plim = 0;
            if (__builtin_amdgcn_readfirstlane(rdreg<R_PA>()) == call) exa = (int)(short)__builtin_amdgcn_readfirstlane((int)rdreg<R_PA + 1>());
            if (RA >= na && RB >= nb) break;
            wide = false;
            const unsigned long long now = rtclock();
            if ((it & 15) == 0 || now - t0 > DEADLINE_TICKS) dead = give_up(tmo, call, t0);
            __builtin_amdgcn_s_sleep(SPINSLEEP);
        }
        if (pst == 0) retime(tcur);
    };
    acc_zero_all();
    const unsigned ldsb = (unsigned)(uintptr_t)lds + lane * 16, ldsh = ldsb + 65536;
#define GLDS_A(tile, st, i) __builtin_amdgcn_global_load_lds((const unsigned*)(gA + (size_t)(tile) * TB + ((i) >> 2) * 4096 + voff0), (LAS unsigned*)(lds + (st) * TB + wid * 8192 + ((i) >> 2) * 4096), 16, ((i) & 3) * 1024, 0)
#define LOAD_B(buf, tile, i) gload16<R_B + (buf) * 32 + (i) * 4, ((i) & 3) * 1024>(((i) >> 2) ? voff1 : voff0, gB + (size_t)(tile) * TB)
    sfor<8>([&](auto J) __attribute__((always_inline)) { constexpr int j = J; gload16<R_W + 4 * j, 0>(wvoff, wbase + (size_t)(1024 + j) * GN * 4); });
    spin(3, 3, 0);
    sfor<8>([&](auto I) __attribute__((always_inline)) { constexpr int i = I; GLDS_A(0, 0, i); });
    sfor<8>([&](auto I) __attribute__((always_inline)) { constexpr int i = I; LOAD_B(0, 0, i); });
    sfor<8>([&](auto I) __attribute__((always_inline)) { constexpr int i = I; GLDS_A(1, 1, i); });
    sfor<8>([&](auto I) __attribute__((always_inline)) { constexpr int i = I; LOAD_B(1, 1, i); });
    sfor<8>([&](auto I) __attribute__((always_inline)) { constexpr int i = I; GLDS_A(2, 2, i); });
    wait_vm<16>();
    __builtin_amdgcn_s_barrier();
    STAMP(5);
    sfor<APF>([&](auto G) __attribute__((always_inline)) { constexpr int g = G; ldsrd<R_AF + 4 * g, g * 1024>(ldsb, ldsh); });
    evt = pst == 5 ? 1000 : tsend;
    limits();
    float wmax = 0.f; unsigned wfl = 0;
    auto tile = [&](auto PH_, auto LASTP_, const int t) __attribute__((always_inline)) {
        constexpr int PH = PH_; constexpr bool LASTQ = LASTP_, LASTP = LASTQ || ABL_NOW;
        constexpr int cur = PH & 3, PP = (PH + 7) & 7;
        constexpr bool PREV_W = true;
        constexpr int e1p = (PREV_W && PP == 6 ? 8 : 0) + (PREV_W && PP == 2 ? 1 : 0), e2p = (PREV_W && PP == 0 ? 4 : 0) + ((PP & 1) ? 2 : 0);
        constexpr int base = 16 + e1p + e2p;
        constexpr bool FIN = LASTQ && PH == 4;
        if (__builtin_expect(t >= nslow, 0)) {
            if (!FLAG1 && pst == 1) { wait_vm<base + 28>(); pst = 2; } else { wait_vm<base>(); if (pst == 1 || pst == 2) push_flag(); }
            const int na = t + 4 < NT ? t + 4 : NT, nb = FIN ? NT : t + 3;
            if (RA < na || RB < nb) spin(na, nb, t);
            if (pst == 0 && t >= tsend) { if (CLOSED) retime(t); if (t >= tsend) push_all(); }
            evt = pst == 5 ? 1000 : (pst == 0 ? tsend : t + 1);
            limits();
        } else wait_vm<base>();
        const int slab = min((t >> 3) + 2, 7);
        sfor<32>([&](auto G) __attribute__((always_inline)) { constexpr int g = G;
            constexpr int RC = (g & 15) * 16, RAF = R_AF + 4 * (g & 7), RBF = R_B + cur * 32 + (g >> 4) * 16;
            if constexpr (g == 16) __builtin_amdgcn_s_barrier();
            constexpr int N0 = (SNAKE && (g & 1)) ? 12 : 0, DN = (SNAKE && (g & 1)) ? -4 : 4;
            mfma1w<RC + N0, (FIN && g + APF > 32) ? (32 - g - 1) : APF - 1, RAF, RBF + N0>();
            if constexpr (g + APF < 32) ldsrd<R_AF + 4 * ((g + APF) & 7), (PH & 3) * TB + (g + APF) * 1024>(ldsb, ldsh);
            else if constexpr (!FIN) ldsrd<R_AF + 4 * ((g + APF) & 7), ((PH + 1) & 3) * TB + (g + APF - 32) * 1024>(ldsb, ldsh);
            mfma1<RC + N0 + DN, RAF, RBF + N0 + DN>();
            if constexpr ((PH & 1) == 1 && g == 8) digestA(plim);
            if constexpr ((PH & 1) == 1 && g == 10) digestB(plim);
            if constexpr ((PH & 1) == 1 && g == 12) limits();
            if constexpr (!LASTP && PH == 0 && g >= 8 && g < 16) { if constexpr (g == 8) wmax = 0.f; amax4r<R_W + 4 * (g - 8)>(wmax); }
            if constexpr (!LASTP && PH == 0 && g == 17) { const float a = wmax * sw; wfl = call4 + ((__ballot(!(a < 60000.0f)) ? 1u : 0u) | (__ballot(a >= 0.015625f) ? 2u : 0u)); }
            u32x4 wq;
            if constexpr (!LASTP && PH == 0 && g >= 25 && g < 29) wq = cvt8r<R_W + (g - 25), 4>(sw);
            mfma1<RC + N0 + 2 * DN, RAF, RBF + N0 + 2 * DN>();
            if constexpr (g < 8) LOAD_B((cur + 2) & 3, t + 2, g);
            if constexpr (FIN && g >= 8 && g < 16) LOAD_B(3, t + 3, g - 8);
            if constexpr (FIN && g >= 26 && g < 30) gload16_sys<R_W + 4 * (g - 26)>(lane * 16 + (g - 26) * 1024, fA);
            if constexpr (FIN && g == 16) gload8_agent<R_PB>(lane * 8, fB);
            if constexpr (!LASTP && PH == 6 && g >= 8 && g < 16) gload16<R_W + 4 * (g - 8), 0>(wvoff, wbase + ((size_t)min(slab + 1, 7) * 512 + (g - 8)) * GN * 4);
            if constexpr (!LASTP && PH == 2 && g == 9) gstore4_agent(0u, wfl, fbdst + slab * 64);
            if constexpr (g >= 17 && g < 25) GLDS_A(t + 3, (PH + 3) & 3, g - 17);
            if constexpr (!LASTP && PH == 0 && g >= 25 && g < 29) gstore16_agent<(g - 25) * 1024>(voff0, wq, bdst + (size_t)slab * 8 * TB);
            if constexpr ((PH & 1) == 1 && g == 29) { pollA(NT - 1); plim = NT; }
            if constexpr ((PH & 1) == 1 && g == 30) pollB(NT - 1);
            mfma1<RC + N0 + 3 * DN, RAF, RBF + N0 + 3 * DN>();
        });
    };
    for (int s = 0;; ++s) {
        sfor<4>([&](auto PH) __attribute__((always_inline)) { tile(PH, std::false_type{}, s * 8 + PH); });
        if (__builtin_expect(s == 7, 0)) break;
        sfor<4>([&](auto PH) __attribute__((always_inline)) { tile(std::integral_constant<int, PH + 4>{}, std::false_type{}, s * 8 + 4 + PH); });
    }
    tile(std::integral_constant<int, 4>{}, std::true_type{}, 60);
    if (pst == 0) push_all();
    wait_vm<0>();
    if (pst != 5) push_flag();
    __builtin_amdgcn_s_barrier();
    STAMP(8);
    bool wbad;
    { const unsigned o = rdreg<R_PB>() | rdreg<R_PB + 1>(); wbad = __ballot(o & 1u) != 0ull || __ballot(o & 2u) == 0ull;
      unsigned xo = 0; sfor<8>([&](auto I) __attribute__((always_inline)) { xo |= rdreg<R_W + 2 * I + 1>(); }); wbad = wbad || __ballot(xo & 0x10000u) != 0ull; }
    float fx = exp2i(-exa), fw = exp2i(-ew);
    bool bad = false;
    float* cp = p.out + (size_t)(pm * 256 + fq * 4) * GN + pn * 256 + wid * 64 + fr * 4;
    auto store_rows = [&](auto M_) __attribute__((always_inline)) { constexpr int m = M_;
        sfor<4>([&](auto J) __attribute__((always_inline)) { constexpr int j = J;
            f32x4 v = acc_read_row<m * 16 + j>();
#pragma unroll
            for (int e = 0; e < 4; ++e) { float y = v[e]; y = y < 0.f ? 0.f : y; v[e] = (y * fx) * fw; }
            __builtin_nontemporal_store(v, (f32x4*)(cp + (size_t)(m * 16 + j) * GN));
        });
    };
    if (!wbad) {
        bad = dead || load_word(tmo) == call;
        if (bad) fw = __builtin_nanf("");
        sfor<APF>([&](auto I) __attribute__((always_inline)) { constexpr int i = I, m = i / 6, q = i % 6; ldsrd<R_AF + 4 * (i & 7), (1 + (q >> 1)) * TB + ((q & 1) * 16 + m) * 1024>(ldsb, ldsh); });
        sfor<96>([&](auto I) __attribute__((always_inline)) { constexpr int i = I, m = i / 6, q = i % 6, i2 = i + APF, m2 = i2 / 6, q2 = i2 % 6;
            if constexpr (i2 < 96) ldsrd<R_AF + 4 * (i2 & 7), (1 + (q2 >> 1)) * TB + ((q2 & 1) * 16 + m2) * 1024>(ldsb, ldsh);
            mfma_group<m * 16, (i2 < 96) ? APF : (95 - i), R_AF + 4 * (i & 7), R_B + (1 + (q >> 1)) * 32 + (q & 1) * 16>();
            if constexpr (q == 1 && m > 0) store_rows(std::integral_constant<int, m - 1>{});
        });
        asm volatile("s_nop 15\n\ts_nop 15" ::: "memory");
        store_rows(std::integral_constant<int, 15>{});
    } else {
        bad = dead || load_word(tmo) == call;
        asm volatile("s_nop 15\n\ts_nop 15" ::: "memory");
        acc_zero_all();
        const _Float16* ax = (const _Float16*)(wsl + abuf + (size_t)pm * PANEL_B);
        const float* x32 = (const float*)(wsl + WS_X32 + (size_t)(call & 1u) * 2 * OPER_B) + (size_t)(pm * 256 + fr) * GK;
        const unsigned* fl = (const unsigned*)fA;
        const float* wq = p.w + pn * 256 + wid * 64 + fr * 4;
        for (int k0 = 0; k0 < GK; k0 += 4) {
            const int k = k0 + fq;
            const f32x4 b = *(const f32x4*)(wq + (size_t)k * GN);
            const _Float16* ap = ax + (size_t)(k >> 5) * 8192 + (size_t)(((k >> 3) & 3) * 16 + fr) * 8 + (k & 7);
            sfor<4>([&](auto Q_) __attribute__((always_inline)) { constexpr int q = Q_;
                const bool f32src = (__hip_atomic_load(fl + ((k0 >> 5) * 4 + q) * 2 + 1, __ATOMIC_RELAXED, __HIP_MEMORY_SCOPE_SYSTEM) & 0x10000u) != 0u;
                sfor<4>([&](auto M_) __attribute__((always_inline)) { constexpr int m = q * 4 + M_;
                    const float a = f32src ? __hip_atomic_load(x32 + (size_t)m * 16 * GK + k, __ATOMIC_RELAXED, __HIP_MEMORY_SCOPE_SYSTEM) : (float)ap[m * 512] * fx;
                    mfma_f32<m * 16>(a, b[0]); mfma_f32<m * 16 + 4>(a, b[1]); mfma_f32<m * 16 + 8>(a, b[2]); mfma_f32<m * 16 + 12>(a, b[3]);
                });
            });
        }
        fx = 1.0f; fw = bad ? __builtin_nanf("") : 1.0f;
        asm volatile("s_nop 15\n\ts_nop 15" ::: "memory");
        sfor<16>([&](auto M_) __attribute__((always_inline)) { store_rows(M_); });
    }
    asm volatile(".Lagk_end:" ::: "memory");
    STAMP(9);
}

}

extern "C" void kernel_launch(int rank, int nranks, ncclComm_t comm,
                              hipStream_t stream,
                              void* const* d_in, const int* in_sizes, int n_in,
                              void* d_out, int out_size,
                              const int* in_dtypes, int out_dtype,
                              void* const* d_out_all,
                              void* const* d_ws_all, size_t ws_bytes) {
    if (nranks != WORLD || rank < 0 || rank >= WORLD || ws_bytes < WS_END || n_in < 2) { fprintf(stderr, "unsupported configuration\n"); return; }
    static bool init[WORLD] = {};
    if (!init[rank]) {
        HIP_CHECK(hipFuncSetAttribute((const void*)fused_kernel, hipFuncAttributeMaxDynamicSharedMemorySize, LDS_BYTES));
        init[rank] = true;
    }
    Params p;
    p.x = (const float*)d_in[0]; p.w = (const float*)d_in[1]; p.out = (float*)d_out; p.rank = rank;
    for (int i = 0; i < WORLD; ++i) p.ws[i] = (char*)d_ws_all[(rank + i) & 7];
    hipLaunchKernelGGL(fused_kernel, dim3(GRID), dim3(NTHREADS), LDS_BYTES, stream, p);
    HIP_CHECK(hipGetLastError());
}
```

```cpp
#include <hip/hip_runtime.h>
#include <rccl/rccl.h>
#include <cstdio>
#include <cstdint>
#include <utility>

#define HIP_CHECK(expr) do { hipError_t _e = (expr); if (_e != hipSuccess) { \
    fprintf(stderr, "HIP error %s at %s:%d\n", hipGetErrorString(_e), __FILE__, __LINE__); \
    return; } } while (0)

namespace {

template <int... Is, class F> __device__ __forceinline__ void sfor_impl(std::integer_sequence<int, Is...>, F&& f) { (f(std::integral_constant<int, Is>{}), ...); }
template <int N, class F> __device__ __forceinline__ void sfor(F&& f) { sfor_impl(std::make_integer_sequence<int, N>{}, f); }

typedef _Float16 half8 __attribute__((ext_vector_type(8)));
typedef float f32x4 __attribute__((ext_vector_type(4)));
typedef unsigned u32x4 __attribute__((ext_vector_type(4)));
typedef unsigned u32x2 __attribute__((ext_vector_type(2)));
#define LAS __attribute__((address_space(3)))

constexpr int WORLD = 8;
constexpr int GM = 4096, GK = 4096, GN = 4096;
constexpr int NT = GK / 64;
constexpr int TB = 32768;
constexpr size_t PANEL_B = (size_t)NT * TB;
constexpr size_t OPER_B = 16 * PANEL_B;
constexpr size_t WS_CTL = 0;
constexpr size_t WS_FLAGA = 65536;
constexpr size_t WS_FLAGB = 262144;
constexpr size_t WS_DBG = 512u << 10;
constexpr size_t WS_A0 = 1u << 20;
constexpr size_t WS_B = WS_A0 + 2 * OPER_B;
constexpr size_t WS_X32 = WS_B + OPER_B;
constexpr size_t WS_END = WS_X32 + 4 * OPER_B;
constexpr int CW_TICKET = 0, CW_TMO = 256;
constexpr unsigned long long DEADLINE_TICKS = 50000000ull;
constexpr int LDS_X = 4 * TB;
constexpr int LDS_BYTES = 4 * TB + 16384;
constexpr int GRID = 256;
#define NTHREADS 256
constexpr int PACE = 72;
constexpr int KNOW = 6;
constexpr int KEARLY = 8;
constexpr int TILE_TICKS = 145;
constexpr int SPINSLEEP = 8;
constexpr int APF = 5;

struct Params { const float* x; const float* w; float* out; char* ws[WORLD]; int rank; };

template <class T> __device__ __forceinline__ T* uptr(T* q) {
    unsigned long long v = (unsigned long long)q;
    unsigned lo = __builtin_amdgcn_readfirstlane((unsigned)v), hi = __builtin_amdgcn_readfirstlane((unsigned)(v >> 32));
    return (T*)(((unsigned long long)hi << 32) | lo);
}
__device__ __forceinline__ u32x4 cvt8(const f32x4 a, const f32x4 b, float s) {
    half8 h;
    h[0] = (_Float16)(a[0] * s); h[1] = (_Float16)(a[1] * s); h[2] = (_Float16)(a[2] * s); h[3] = (_Float16)(a[3] * s);
    h[4] = (_Float16)(b[0] * s); h[5] = (_Float16)(b[1] * s); h[6] = (_Float16)(b[2] * s); h[7] = (_Float16)(b[3] * s);
    return __builtin_bit_cast(u32x4, h);
}
__device__ __forceinline__ unsigned absbits(float v) { return __float_as_uint(v) & 0x7fffffffu; }
__device__ __forceinline__ unsigned amax4(const f32x4 a) { return max(max(absbits(a[0]), absbits(a[1])), max(absbits(a[2]), absbits(a[3]))); }
__device__ __forceinline__ unsigned wave_umax(unsigned v) {
    v = max(v, (unsigned)__builtin_amdgcn_update_dpp(0, (int)v, 0x111, 0xf, 0xf, false));
    v = max(v, (unsigned)__builtin_amdgcn_update_dpp(0, (int)v, 0x112, 0xf, 0xf, false));
    v = max(v, (unsigned)__builtin_amdgcn_update_dpp(0, (int)v, 0x114, 0xf, 0xf, false));
    v = max(v, (unsigned)__builtin_amdgcn_update_dpp(0, (int)v, 0x118, 0xf, 0xf, false));
    v = max(v, (unsigned)__builtin_amdgcn_update_dpp(0, (int)v, 0x142, 0xa, 0xf, false));
    v = max(v, (unsigned)__builtin_amdgcn_update_dpp(0, (int)v, 0x143, 0xc, 0xf, false));
    return (unsigned)__builtin_amdgcn_readlane((int)v, 63);
}
__device__ __forceinline__ int scale_exp(unsigned maxbits, int target) {
    const int ef = (int)(maxbits >> 23);
    if (ef == 0 || ef == 255) return 0;
    int e = target - (ef - 127);
    return e > 126 ? 126 : (e < -126 ? -126 : e);
}
__device__ __forceinline__ float exp2i(int e) { return __uint_as_float((unsigned)(e + 127) << 23); }

constexpr int R_PB = 58, R_PA = 60, R_W = 64, R_AF = 96, R_B = 128, R_X = 128, R_S = 96, R_SL = 112, R_W2 = 160;
#define OWNED_REGS "a0","a1","a2","a3","a4","a5","a6","a7","a8","a9","a10","a11","a12","a13","a14","a15","a16","a17","a18","a19","a20","a21","a22","a23","a24","a25","a26","a27","a28","a29","a30","a31","a32","a33","a34","a35","a36","a37","a38","a39","a40","a41","a42","a43","a44","a45","a46","a47","a48","a49","a50","a51","a52","a53","a54","a55","a56","a57","a58","a59","a60","a61","a62","a63","a64","a65","a66","a67","a68","a69","a70","a71","a72","a73","a74","a75","a76","a77","a78","a79","a80","a81","a82","a83","a84","a85","a86","a87","a88","a89","a90","a91","a92","a93","a94","a95","a96","a97","a98","a99","a100","a101","a102","a103","a104","a105","a106","a107","a108","a109","a110","a111","a112","a113","a114","a115","a116","a117","a118","a119","a120","a121","a122","a123","a124","a125","a126","a127","a128","a129","a130","a131","a132","a133","a134","a135","a136","a137","a138","a139","a140","a141","a142","a143","a144","a145","a146","a147","a148","a149","a150","a151","a152","a153","a154","a155","a156","a157","a158","a159","a160","a161","a162","a163","a164","a165","a166","a167","a168","a169","a170","a171","a172","a173","a174","a175","a176","a177","a178","a179","a180","a181","a182","a183","a184","a185","a186","a187","a188","a189","a190","a191","a192","a193","a194","a195","a196","a197","a198","a199","a200","a201","a202","a203","a204","a205","a206","a207","a208","a209","a210","a211","a212","a213","a214","a215","a216","a217","a218","a219","a220","a221","a222","a223","a224","a225","a226","a227","a228","a229","a230","a231","a232","a233","a234","a235","a236","a237","a238","a239","a240","a241","a242","a243","a244","a245","a246","a247","a248","a249","a250","a251","a252","a253","a254","a255","v58","v59","v60","v61","v62","v63","v64","v65","v66","v67","v68","v69","v70","v71","v72","v73","v74","v75","v76","v77","v78","v79","v80","v81","v82","v83","v84","v85","v86","v87","v88","v89","v90","v91","v92","v93","v94","v95","v96","v97","v98","v99","v100","v101","v102","v103","v104","v105","v106","v107","v108","v109","v110","v111","v112","v113","v114","v115","v116","v117","v118","v119","v120","v121","v122","v123","v124","v125","v126","v127","v128","v129","v130","v131","v132","v133","v134","v135","v136","v137","v138","v139","v140","v141","v142","v143","v144","v145","v146","v147","v148","v149","v150","v151","v152","v153","v154","v155","v156","v157","v158","v159","v160","v161","v162","v163","v164","v165","v166","v167","v168","v169","v170","v171","v172","v173","v174","v175","v176","v177","v178","v179","v180","v181","v182","v183","v184","v185","v186","v187","v188","v189","v190","v191","v192","v193","v194","v195","v196","v197","v198","v199","v200","v201","v202","v203","v204","v205","v206","v207","v208","v209","v210","v211","v212","v213","v214","v215","v216","v217","v218","v219","v220","v221","v222","v223","v224","v225","v226","v227","v228","v229","v230","v231","v232","v233","v234","v235","v236","v237","v238","v239","v240","v241","v242","v243","v244","v245","v246","v247","v248","v249","v250","v251","v252","v253","v254","v255"
#define ALL_AGPRS "a0","a1","a2","a3","a4","a5","a6","a7","a8","a9","a10","a11","a12","a13","a14","a15","a16","a17","a18","a19","a20","a21","a22","a23","a24","a25","a26","a27","a28","a29","a30","a31","a32","a33","a34","a35","a36","a37","a38","a39","a40","a41","a42","a43","a44","a45","a46","a47","a48","a49","a50","a51","a52","a53","a54","a55","a56","a57","a58","a59","a60","a61","a62","a63","a64","a65","a66","a67","a68","a69","a70","a71","a72","a73","a74","a75","a76","a77","a78","a79","a80","a81","a82","a83","a84","a85","a86","a87","a88","a89","a90","a91","a92","a93","a94","a95","a96","a97","a98","a99","a100","a101","a102","a103","a104","a105","a106","a107","a108","a109","a110","a111","a112","a113","a114","a115","a116","a117","a118","a119","a120","a121","a122","a123","a124","a125","a126","a127","a128","a129","a130","a131","a132","a133","a134","a135","a136","a137","a138","a139","a140","a141","a142","a143","a144","a145","a146","a147","a148","a149","a150","a151","a152","a153","a154","a155","a156","a157","a158","a159","a160","a161","a162","a163","a164","a165","a166","a167","a168","a169","a170","a171","a172","a173","a174","a175","a176","a177","a178","a179","a180","a181","a182","a183","a184","a185","a186","a187","a188","a189","a190","a191","a192","a193","a194","a195","a196","a197","a198","a199","a200","a201","a202","a203","a204","a205","a206","a207","a208","a209","a210","a211","a212","a213","a214","a215","a216","a217","a218","a219","a220","a221","a222","a223","a224","a225","a226","a227","a228","a229","a230","a231","a232","a233","a234","a235","a236","a237","a238","a239","a240","a241","a242","a243","a244","a245","a246","a247","a248","a249","a250","a251","a252","a253","a254","a255"
template <int R, int OFF> __device__ __forceinline__ void gload16(unsigned voff, const char* sbase) {
    asm volatile("global_load_dwordx4 v[%c2:%c2+3], %0, %1 offset:%c3" :: "v"(voff), "s"(sbase), "i"(R), "i"(OFF) : "memory");
}
template <int R> __device__ __forceinline__ void gload16_sys(unsigned voff, const char* sbase) {
    asm volatile("global_load_dwordx4 v[%c2:%c2+3], %0, %1 sc0 sc1" :: "v"(voff), "s"(sbase), "i"(R) : "memory");
}
template <int R> __device__ __forceinline__ void gload16_agent(unsigned voff, const char* sbase) {
    asm volatile("global_load_dwordx4 v[%c2:%c2+3], %0, %1 sc1" :: "v"(voff), "s"(sbase), "i"(R) : "memory");
}
template <int R> __device__ __forceinline__ void gload8_agent(unsigned voff, const char* sbase) {
    asm volatile("global_load_dwordx2 v[%c2:%c2+1], %0, %1 sc1" :: "v"(voff), "s"(sbase), "i"(R) : "memory");
}
template <int R> __device__ __forceinline__ unsigned rdreg() { unsigned v; asm volatile("v_mov_b32 %0, v%c1" : "=v"(v) : "i"(R)); return v; }
template <int OFF> __device__ __forceinline__ void gstore16_sys(unsigned voff, const u32x4& v, char* sbase) {
    asm volatile("global_store_dwordx4 %0, %1, %2 offset:%3 sc0 sc1\n\ts_nop 1" :: "v"(voff), "v"(v), "s"(sbase), "i"(OFF) : "memory");
}
template <int OFF> __device__ __forceinline__ void gstore16_agent(unsigned voff, const u32x4& v, char* sbase) {
    asm volatile("global_store_dwordx4 %0, %1, %2 offset:%3 sc1\n\ts_nop 1" :: "v"(voff), "v"(v), "s"(sbase), "i"(OFF) : "memory");
}
__device__ __forceinline__ void gstore64_sys_lane(unsigned voff, const u32x4& v0, const u32x4& v1, const u32x4& v2, const u32x4& v3, unsigned plo, unsigned phi, int ln) {
    const unsigned long long b = ((unsigned long long)(unsigned)__builtin_amdgcn_readlane((int)phi, ln) << 32) | (unsigned)__builtin_amdgcn_readlane((int)plo, ln);
    asm volatile("s_nop 4\n\tglobal_store_dwordx4 %0, %1, %5 sc0 sc1\n\tglobal_store_dwordx4 %0, %2, %5 offset:1024 sc0 sc1\n\t"
                 "global_store_dwordx4 %0, %3, %5 offset:2048 sc0 sc1\n\tglobal_store_dwordx4 %0, %4, %5 offset:3072 sc0 sc1\n\ts_nop 1"
                 :: "v"(voff), "v"(v0), "v"(v1), "v"(v2), "v"(v3), "s"(b) : "memory");
}
template <int R, int OFF> __device__ __forceinline__ void storeown_sys(unsigned voff, char* sbase) {
    asm volatile("s_nop 4\n\tglobal_store_dwordx4 %0, v[%c2:%c2+3], %1 offset:%c3 sc0 sc1\n\ts_nop 1" :: "v"(voff), "s"(sbase), "i"(R), "i"(OFF) : "memory");
}
__device__ __forceinline__ void gstore8_sys(char* addr, const u32x2& v) {
    asm volatile("global_store_dwordx2 %0, %1, off sc0 sc1\n\ts_nop 1" :: "v"(addr), "v"(v) : "memory");
}
__device__ __forceinline__ void gstore8_agent(unsigned voff, const u32x2& v, char* sbase) {
    asm volatile("global_store_dwordx2 %0, %1, %2 sc1\n\ts_nop 1" :: "v"(voff), "v"(v), "s"(sbase) : "memory");
}
__device__ __forceinline__ void gstore4_agent(unsigned voff, unsigned v, char* sbase) {
    asm volatile("global_store_dword %0, %1, %2 sc1\n\ts_nop 1" :: "v"(voff), "v"(v), "s"(sbase) : "memory");
}
__device__ __forceinline__ unsigned long long rtclock() {
    unsigned long long t; asm volatile("s_memrealtime %0\n\ts_waitcnt lgkmcnt(0)" : "=s"(t) :: "memory"); return t;
}
__device__ __forceinline__ unsigned load_word(const char* sbase) {
    unsigned v; asm volatile("global_load_dword %0, %1, %2 sc1\n\ts_waitcnt vmcnt(0)" : "=&v"(v) : "v"(0u), "s"(sbase) : "memory");
    return __builtin_amdgcn_readfirstlane(v);
}
__device__ __forceinline__ int give_up(char* tmo, unsigned call, unsigned long long t0) {
    if (load_word(tmo) == call) return 1;
    if (rtclock() - t0 > DEADLINE_TICKS) { gstore4_agent(0u, call, tmo); return 1; }
    __builtin_amdgcn_s_sleep(1);
    return 0;
}
template <int R, int OFF> __device__ __forceinline__ void ldsrd(unsigned lo, unsigned hi) {
    if constexpr (OFF < 65536) asm volatile("ds_read_b128 v[%c1:%c1+3], %0 offset:%c2" :: "v"(lo), "i"(R), "i"(OFF) : "memory");
    else asm volatile("ds_read_b128 v[%c1:%c1+3], %0 offset:%c2" :: "v"(hi), "i"(R), "i"(OFF - 65536) : "memory");
}
template <int OFF> __device__ __forceinline__ void lds_write16(unsigned a, const u32x4& v) { asm volatile("ds_write_b128 %0, %1 offset:%2" :: "v"(a), "v"(v), "i"(OFF) : "memory"); }
template <int OFF> __device__ __forceinline__ u32x4 lds_read16_wait(unsigned a) { u32x4 v; asm volatile("ds_read_b128 %0, %1 offset:%2\n\ts_waitcnt lgkmcnt(0)" : "=&v"(v) : "v"(a), "i"(OFF) : "memory"); return v; }
template <int N> __device__ __forceinline__ void wait_vm() { asm volatile("s_waitcnt vmcnt(%0)" :: "i"(N) : "memory"); }
template <int R> __device__ __forceinline__ void amax4r(float& m) {
    asm volatile("v_max3_f32 %0, |v%c1|, |v[%c1+1]|, %0\n\tv_max3_f32 %0, |v[%c1+2]|, |v[%c1+3]|, %0" : "+v"(m) : "i"(R));
}
template <int R, int N> __device__ __forceinline__ unsigned amax_regs() {
    float m = 0.f;
    sfor<N / 4>([&](auto I) __attribute__((always_inline)) { amax4r<R + 4 * I>(m); });
    return __float_as_uint(m);
}
template <int R, int S> __device__ __forceinline__ u32x4 cvt8r(float s) {
    unsigned a, b, c, d;
    asm volatile("v_fma_mixlo_f16 %0, v%c5, %4, 0\n\tv_fma_mixlo_f16 %1, v[%c5+2*%c6], %4, 0\n\tv_fma_mixlo_f16 %2, v[%c5+4*%c6], %4, 0\n\tv_fma_mixlo_f16 %3, v[%c5+6*%c6], %4, 0\n\t"
                 "v_fma_mixhi_f16 %0, v[%c5+%c6], %4, 0\n\tv_fma_mixhi_f16 %1, v[%c5+3*%c6], %4, 0\n\tv_fma_mixhi_f16 %2, v[%c5+5*%c6], %4, 0\n\tv_fma_mixhi_f16 %3, v[%c5+7*%c6], %4, 0"
                 : "=&v"(a), "=&v"(b), "=&v"(c), "=&v"(d) : "v"(s), "i"(R), "i"(S));
    return (u32x4){a, b, c, d};
}
template <int R, int W, int A, int B> __device__ __forceinline__ void mfma_group() {
    asm volatile("s_waitcnt lgkmcnt(%c0)\n\t"
                 "v_mfma_f32_16x16x32_f16 a[%c1:%c1+3], v[%c2:%c2+3], v[%c3:%c3+3], a[%c1:%c1+3]\n\t"
                 "v_mfma_f32_16x16x32_f16 a[%c1+4:%c1+7], v[%c2:%c2+3], v[%c3+4:%c3+7], a[%c1+4:%c1+7]\n\t"
                 "v_mfma_f32_16x16x32_f16 a[%c1+8:%c1+11], v[%c2:%c2+3], v[%c3+8:%c3+11], a[%c1+8:%c1+11]\n\t"
                 "v_mfma_f32_16x16x32_f16 a[%c1+12:%c1+15], v[%c2:%c2+3], v[%c3+12:%c3+15], a[%c1+12:%c1+15]"
                 :: "i"(W), "i"(R), "i"(A), "i"(B) : "memory", ALL_AGPRS);
}
template <int R, int A, int B> __device__ __forceinline__ void mfma1() {
    asm volatile("v_mfma_f32_16x16x32_f16 a[%c0:%c0+3], v[%c1:%c1+3], v[%c2:%c2+3], a[%c0:%c0+3]" :: "i"(R), "i"(A), "i"(B) : "memory");
}
template <int R, int W, int A, int B> __device__ __forceinline__ void mfma1w() {
    asm volatile("s_waitcnt lgkmcnt(%c0)\n\tv_mfma_f32_16x16x32_f16 a[%c1:%c1+3], v[%c2:%c2+3], v[%c3:%c3+3], a[%c1:%c1+3]" :: "i"(W), "i"(R), "i"(A), "i"(B) : "memory", ALL_AGPRS);
}
template <int R0, int R1> __device__ __forceinline__ unsigned long long ready_mask(unsigned thr) {
    unsigned long long m; unsigned t;
    asm volatile("v_min_u32 %1, v%c3, v%c4\n\tv_cmp_le_u32 %0, %2, %1" : "=s"(m), "=&v"(t) : "s"(thr), "i"(R0), "i"(R1));
    return m;
}
template <int R> __device__ __forceinline__ void acc_zero4() {
    asm volatile("v_accvgpr_write_b32 a%c0, 0\n\tv_accvgpr_write_b32 a[%c0+1], 0\n\tv_accvgpr_write_b32 a[%c0+2], 0\n\tv_accvgpr_write_b32 a[%c0+3], 0" :: "i"(R) : ALL_AGPRS);
}
template <int R> __device__ __forceinline__ void acc_zero16(const u32x4& z) {
    asm volatile("v_mfma_f32_32x32x16_f16 a[%c1:%c1+15], %0, %0, 0" :: "v"(z), "i"(R) : ALL_AGPRS);
}
__device__ __forceinline__ void acc_zero_all() {
    u32x4 z = {0u, 0u, 0u, 0u}; asm volatile("s_nop 1" : "+v"(z));
    sfor<16>([&](auto I) __attribute__((always_inline)) { acc_zero16<I * 16>(z); });
    asm volatile("s_nop 15\n\ts_nop 3" ::: "memory");
}
template <int R> __device__ __forceinline__ f32x4 acc_read_row() {
    float a, b, c, d;
    asm volatile("v_accvgpr_read_b32 %0, a%c4\n\tv_accvgpr_read_b32 %1, a[%c4+4]\n\tv_accvgpr_read_b32 %2, a[%c4+8]\n\tv_accvgpr_read_b32 %3, a[%c4+12]"
                 : "=v"(a), "=v"(b), "=v"(c), "=v"(d) : "i"(R));
    return (f32x4){a, b, c, d};
}
template <int R> __device__ __forceinline__ void mfma_f32(float a, float b) {
    asm volatile("s_nop 1\n\tv_mfma_f32_16x16x4_f32 a[%c2:%c2+3], %0, %1, a[%c2:%c2+3]" :: "v"(a), "v"(b), "i"(R) : ALL_AGPRS);
}

#define STAMP(i) do {} while (0)

__global__ __launch_bounds__(256, 1) __attribute__((amdgpu_num_vgpr(58))) void fused_kernel(Params p) {
    extern __shared__ __attribute__((aligned(16))) unsigned char smem[];
    LAS unsigned char* lds = (LAS unsigned char*)smem;
    const int tid = threadIdx.x, wid = __builtin_amdgcn_readfirstlane(tid >> 6), lane = tid & 63;
    const int fr = lane & 15, fq = lane >> 4;
    const int rank = p.rank;
    char* wsl = uptr(p.ws[0]);
    char* tmo = wsl + WS_CTL + CW_TMO * 4;
    asm volatile("" ::: OWNED_REGS);
    STAMP(0);
    const int bid = blockIdx.x, xcd = bid & 7, idx = bid >> 3;
    const int pm = (idx >> 2) * 2 + (xcd & 1), pn = (xcd >> 1) * 4 + (idx & 3);
    const bool local = (pm >> 1) == rank;
    const int jwg = local ? 224 + (idx & 3) * 8 + xcd : (((idx >> 2) < rank ? (idx >> 2) : (idx >> 2) - 1) * 4 + (idx & 3)) * 8 + xcd;
    const int unit = (jwg >> 2) + 64 * wid, uq = jwg & 3;
    const int ukh = unit >> 1, upl = unit & 1, upm = rank * 2 + upl;

    const unsigned voff0 = lane * 16, voff1 = lane * 16 + 4096;
    if (lane == 0) asm volatile("global_atomic_add v%c3, %0, %1, %2 sc0" :: "v"(0u), "v"(1u), "s"(wsl + WS_CTL + CW_TICKET * 4 + wid * 128), "i"(R_PB) : "memory");
    const char* xb = (const char*)uptr(p.x) + ((size_t)(upl * 256 + uq * 64) * GK + ukh * 32) * 4;
    const unsigned xv = (unsigned)(fr * GK + fq * 8) * 4u;
    sfor<4>([&](auto I) __attribute__((always_inline)) { constexpr int i = I; gload16<R_S + 16 + 4 * i, 0>((unsigned)((lane * 8 + i * 2 + 1) * GK + ((lane * 68 + i * 1028) & (GK - 1))) * 4u, (const char*)uptr(p.x)); });
    sfor<4>([&](auto I) __attribute__((always_inline)) { constexpr int i = I; gload16<R_X + 8 * i, 0>(xv, xb + (size_t)i * 16 * GK * 4); gload16<R_X + 8 * i + 4, 16>(xv, xb + (size_t)i * 16 * GK * 4); });
    const unsigned wvoff = (unsigned)(fq * 8 * GN + fr * 4) * 4u;
    const char* wbase = (const char*)uptr(p.w) + ((size_t)pm * 32 * GN + pn * 256 + wid * 64) * 4;
    sfor<4>([&](auto I) __attribute__((always_inline)) { constexpr int i = I; gload16<R_S + 4 * i, 0>((unsigned)((lane * 64 + i * 16 + 5) * GN + ((lane * 68 + i * 1028) & (GN - 1))) * 4u, (const char*)uptr(p.w)); });
    sfor<8>([&](auto J) __attribute__((always_inline)) { constexpr int j = J; gload16<R_W + 4 * j, 0>(wvoff, wbase + (size_t)j * GN * 4); });
    {
        unsigned long long pc; unsigned sz;
        asm volatile("s_getpc_b64 %0\n.Lagk_begin:\n\ts_mov_b32 %1, .Lagk_end-.Lagk_begin" : "=s"(pc), "=s"(sz));
        gload16<R_B + 124, 0>(min((unsigned)(idx * 4 + wid) * 1024u, sz - 1024u) + lane * 16, (const char*)pc);
    }
    const int r0 = (unit * 4 + uq * 2) % 7;
    char* fdst = wsl;
#pragma unroll
    for (int i = 1; i < WORLD; ++i) { char* q = p.ws[i]; fdst = (lane & 7) == i ? q : fdst; }
    STAMP(15);
    wait_vm<13>();
    STAMP(14);
    const unsigned call = __builtin_amdgcn_readfirstlane(rdreg<R_PB>()) / GRID + 1u;
    const size_t abuf = WS_A0 + (size_t)(call & 1u) * OPER_B;
    const size_t faoff = WS_FLAGA + (size_t)(call & 1u) * 65536;
    int dead = 0;
    const int ex = scale_exp(wave_umax(amax_regs<R_S + 16, 16>()), 4);
    const unsigned tx0 = (unsigned)rtclock();
    STAMP(3);
    u32x4 xq[4];
    bool xbad;
    {
        const float sx = exp2i(ex);
        sfor<4>([&](auto I) __attribute__((always_inline)) { constexpr int i = I; xq[i] = cvt8r<R_X + 8 * i, 1>(sx); });
        const float a = __uint_as_float(amax_regs<R_X, 32>()) * sx;
        xbad = __ballot(!(a < 60000.0f)) != 0ull || (__ballot(a >= 0.000244140625f) == 0ull && __ballot(a != 0.0f) != 0ull);
    }
    const unsigned xoff = (unsigned)(abuf + (size_t)upm * PANEL_B + (size_t)ukh * 16384 + uq * 4096 + lane * 16);
    char* fladdr = fdst + faoff + (((size_t)upm * 128 + ukh) * 4 + uq) * 8;
    const u32x2 flval = {call, (unsigned)(ex & 0xffff) | (xbad ? 0x10000u : 0u)};
    const unsigned plo = (unsigned)(unsigned long long)fdst, phi = (unsigned)((unsigned long long)fdst >> 32);
    const unsigned xlds = (unsigned)(uintptr_t)lds + LDS_X + wid * 4096 + lane * 16;
    sfor<4>([&](auto I) __attribute__((always_inline)) { constexpr int i = I; lds_write16<i * 1024>(xlds, xq[i]); });
    int pst = 0;
    auto push_all = [&]() __attribute__((always_inline)) {
        STAMP(11);
        const u32x4 v0 = lds_read16_wait<0>(xlds), v1 = lds_read16_wait<1024>(xlds), v2 = lds_read16_wait<2048>(xlds), v3 = lds_read16_wait<3072>(xlds);
        int q = r0;
#pragma unroll 1
        for (int d = 0; d < 7; ++d) {
            gstore64_sys_lane(xoff, v0, v1, v2, v3, plo, phi, q + 1);
            q = q == 6 ? 0 : q + 1;
        }
        pst = 1;
    };
    auto push_flag = [&]() __attribute__((always_inline)) { if (lane >= 1 && lane < 8) gstore8_sys(fladdr, flval); pst = 5; STAMP(12); };
    const unsigned due = tx0 + (unsigned)(ukh < KNOW ? 0 : ukh * PACE);
    if (__builtin_expect(xbad, 0)) {
        const unsigned so = (unsigned)(WS_X32 + (size_t)(call & 1u) * 2 * OPER_B + ((size_t)(upm * 256 + uq * 64) * GK + ukh * 32) * 4) + xv;
#pragma unroll 1
        for (int d = 0; d < WORLD; ++d) {
            const unsigned long long bq = ((unsigned long long)(unsigned)__builtin_amdgcn_readlane((int)phi, d) << 32) | (unsigned)__builtin_amdgcn_readlane((int)plo, d);
            sfor<8>([&](auto I) __attribute__((always_inline)) { constexpr int i = I; storeown_sys<R_X + 4 * i, (i & 1) * 16>(so + (unsigned)(i >> 1) * 16u * GK * 4u, (char*)bq); });
        }
        wait_vm<0>();
    }
    if (ukh < KNOW) push_all();
    sfor<4>([&](auto I) __attribute__((always_inline)) { constexpr int i = I; gstore16_sys<i * 1024>(xoff, xq[i], wsl); });
    sfor<8>([&](auto J) __attribute__((always_inline)) { constexpr int j = J; gload16<R_W2 + 4 * j, 0>(wvoff, wbase + (size_t)(512 + j) * GN * 4); });
    if (ukh < KNOW) wait_vm<41>(); else wait_vm<13>();
    const int ew = scale_exp(wave_umax(amax_regs<R_S, 16>()), 0);
    const float sw = exp2i(ew);
    char* bdst = wsl + WS_B + (size_t)pn * PANEL_B + (size_t)(pm >> 1) * TB + wid * 8192 + (pm & 1) * 4096;
    char* fbdst = wsl + WS_FLAGB + ((size_t)(pn * 4 + wid) * 128 + pm) * 4;
    auto wbits = [&]() __attribute__((always_inline)) -> unsigned {
        const float a = __uint_as_float(amax_regs<R_W, 32>()) * sw;
        return call * 4u + ((__ballot(!(a < 60000.0f)) ? 1u : 0u) | (__ballot(a >= 0.015625f) ? 2u : 0u));
    };
    auto wstore = [&](auto C, char* dst) __attribute__((always_inline)) { constexpr int c = C; gstore16_agent<c * 1024>(voff0, cvt8r<R_W + c, 4>(sw), dst); };
    sfor<4>([&](auto C) __attribute__((always_inline)) { wstore(C, bdst); });
    const unsigned fb0 = wbits();
    wait_vm<0>();
    gstore4_agent(0u, fb0, fbdst);
    if (lane == 0) gstore8_sys(fladdr, flval);
    if (pst == 1) push_flag();
    if (pst == 0 && ukh < KEARLY) {
        while ((int)((unsigned)rtclock() - due) < 0) __builtin_amdgcn_s_sleep(1);
        push_all(); wait_vm<0>(); push_flag();
    }
    {
        sfor<4>([&](auto C) __attribute__((always_inline)) { constexpr int c = C; gstore16_agent<c * 1024>(voff0, cvt8r<R_W2 + c, 4>(sw), bdst + 8 * TB); });
        const float a = __uint_as_float(amax_regs<R_W2, 32>()) * sw;
        const unsigned f1 = call * 4u + ((__ballot(!(a < 60000.0f)) ? 1u : 0u) | (__ballot(a >= 0.015625f) ? 2u : 0u));
        wait_vm<0>();
        gstore4_agent(64u, f1, fbdst);
    }
    STAMP(4);

    const char* gA = wsl + abuf + (size_t)pm * PANEL_B + wid * 8192;
    const char* gB = wsl + WS_B + (size_t)pn * PANEL_B + wid * 8192;
    const char* fA = wsl + faoff + (size_t)pm * 4096;
    const char* fB = wsl + WS_FLAGB + (size_t)(pn * 4 + wid) * 512;
    int RA = 0, RB = 0, exa = 0;
    int plim = 0;
    int qa = 0, qb = 0;
    int tsend = 0;
    int evt = 0;
    int nslow = 0;
    const unsigned call4 = call * 4u;
    auto pollA = [&](int last) __attribute__((always_inline)) { qa = RA;
        gload16_sys<R_PA>((unsigned)min(qa + (lane >> 2), last) * 64u + (lane & 3) * 16u, fA);
    };
    auto pollB = [&](int last) __attribute__((always_inline)) { qb = RB; gload8_agent<R_PB>((unsigned)min(qb + fr, last) * 8u, fB); };
    auto digestA = [&](int lim) __attribute__((always_inline)) { const unsigned long long nm = ~ready_mask<R_PA, R_PA + 2>(call); RA = max(RA, min(qa + (nm ? (int)__builtin_ctzll(nm) >> 2 : 16), lim)); };
    auto digestB = [&](int lim) __attribute__((always_inline)) { RB = max(RB, min(qb + (int)__builtin_ctz(~(unsigned)ready_mask<R_PB, R_PB + 1>(call4) | 0x10000u), lim)); };
    auto limits = [&]() __attribute__((always_inline)) { nslow = min(min(RA >= NT ? 1000 : RA - 3, RB >= NT ? 1000 : min(RB - 2, NT - 4)), evt); };
    auto retime = [&](int tcur) __attribute__((always_inline)) {
        const int rem = (int)(due - (unsigned)rtclock());
        tsend = rem <= TILE_TICKS / 2 ? tcur : tcur + max(1, (rem + TILE_TICKS / 2) / TILE_TICKS);
    };
    auto spin = [&](int na, int nb, int tcur) __attribute__((always_inline)) {
        const unsigned long long t0 = rtclock();
        wait_vm<0>();
        if (pst == 1) push_flag();
        bool wide = true;
        for (int it = 1; !dead; ++it) {
            if (pst == 0 && (int)((unsigned)rtclock() - due) >= 0) { push_all(); wait_vm<0>(); push_flag(); }
            const int la = wide ? NT - 1 : min(max(na, RA + 1) + 2, NT) - 1, lb = wide ? NT - 1 : min(max(nb, RB + 1) + 4, NT) - 1;
            pollA(la); pollB(lb);
            wait_vm<0>();
            digestA(la + 1); digestB(lb + 1);
            plim = 0;
            if (__builtin_amdgcn_readfirstlane(rdreg<R_PA>()) == call) exa = (int)(short)__builtin_amdgcn_readfirstlane((int)rdreg<R_PA + 1>());
            if (RA >= na && RB >= nb) break;
            wide = false;
            const unsigned long long now = rtclock();
            if ((it & 15) == 0 || now - t0 > DEADLINE_TICKS) dead = give_up(tmo, call, t0);
            __builtin_amdgcn_s_sleep(SPINSLEEP);
        }
        if (pst == 0) retime(tcur);
    };
    acc_zero_all();
    const unsigned ldsb = (unsigned)(uintptr_t)lds + lane * 16, ldsh = ldsb + 65536;
#define GLDS_A(tile, st, i) __builtin_amdgcn_global_load_lds((const unsigned*)(gA + (size_t)(tile) * TB + ((i) >> 2) * 4096 + voff0), (LAS unsigned*)(lds + (st) * TB + wid * 8192 + ((i) >> 2) * 4096), 16, ((i) & 3) * 1024, 0)
#define LOAD_B(buf, tile, i) gload16<R_B + (buf) * 32 + (i) * 4, ((i) & 3) * 1024>(((i) >> 2) ? voff1 : voff0, gB + (size_t)(tile) * TB)
    sfor<8>([&](auto J) __attribute__((always_inline)) { constexpr int j = J; gload16<R_W + 4 * j, 0>(wvoff, wbase + (size_t)(1024 + j) * GN * 4); });
    spin(3, 3, 0);
    sfor<8>([&](auto I) __attribute__((always_inline)) { constexpr int i = I; GLDS_A(0, 0, i); });
    sfor<8>([&](auto I) __attribute__((always_inline)) { constexpr int i = I; LOAD_B(0, 0, i); });
    sfor<8>([&](auto I) __attribute__((always_inline)) { constexpr int i = I; GLDS_A(1, 1, i); });
    sfor<8>([&](auto I) __attribute__((always_inline)) { constexpr int i = I; LOAD_B(1, 1, i); });
    sfor<8>([&](auto I) __attribute__((always_inline)) { constexpr int i = I; GLDS_A(2, 2, i); });
    wait_vm<16>();
    __builtin_amdgcn_s_barrier();
    STAMP(5);
    sfor<APF>([&](auto G) __attribute__((always_inline)) { constexpr int g = G; ldsrd<R_AF + 4 * g, g * 1024>(ldsb, ldsh); });
    evt = pst == 5 ? 1000 : tsend;
    limits();
    float wmax = 0.f; unsigned wfl = 0;
    auto tile = [&](auto PH_, auto LASTP_, const int t) __attribute__((always_inline)) {
        constexpr int PH = PH_; constexpr bool LASTQ = LASTP_, LASTP = LASTQ;
        constexpr int cur = PH & 3, PP = (PH + 7) & 7;
        constexpr bool PREV_W = true;
        constexpr int e1p = (PREV_W && PP == 6 ? 8 : 0) + (PREV_W && PP == 2 ? 1 : 0), e2p = (PREV_W && PP == 0 ? 4 : 0) + ((PP & 1) ? 2 : 0);
        constexpr int base = 16 + e1p + e2p;
        constexpr bool FIN = LASTQ && PH == 4;
        if (__builtin_expect(t >= nslow, 0)) {
            wait_vm<base>();
            if (pst == 1) push_flag();
            const int na = t + 4 < NT ? t + 4 : NT, nb = FIN ? NT : t + 3;
            if (RA < na || RB < nb) spin(na, nb, t);
            if (pst == 0 && t >= tsend) push_all();
            evt = pst == 5 ? 1000 : (pst == 0 ? tsend : t + 1);
            limits();
        } else wait_vm<base>();
        const int slab = min((t >> 3) + 2, 7);
        sfor<32>([&](auto G) __attribute__((always_inline)) { constexpr int g = G;
            constexpr int RC = (g & 15) * 16, RAF = R_AF + 4 * (g & 7), RBF = R_B + cur * 32 + (g >> 4) * 16;
            if constexpr (g == 16) __builtin_amdgcn_s_barrier();
            constexpr int N0 = (g & 1) ? 12 : 0, DN = (g & 1) ? -4 : 4;
            mfma1w<RC + N0, (FIN && g + APF > 32) ? (32 - g - 1) : APF - 1, RAF, RBF + N0>();
            if constexpr (g + APF < 32) ldsrd<R_AF + 4 * ((g + APF) & 7), (PH & 3) * TB + (g + APF) * 1024>(ldsb, ldsh);
            else if constexpr (!FIN) ldsrd<R_AF + 4 * ((g + APF) & 7), ((PH + 1) & 3) * TB + (g + APF - 32) * 1024>(ldsb, ldsh);
            mfma1<RC + N0 + DN, RAF, RBF + N0 + DN>();
            if constexpr ((PH & 1) == 1 && g == 8) digestA(plim);
            if constexpr ((PH & 1) == 1 && g == 10) digestB(plim);
            if constexpr ((PH & 1) == 1 && g == 12) limits();
            if constexpr (!LASTP && PH == 0 && g >= 8 && g < 16) { if constexpr (g == 8) wmax = 0.f; amax4r<R_W + 4 * (g - 8)>(wmax); }
            if constexpr (!LASTP && PH == 0 && g == 17) { const float a = wmax * sw; wfl = call4 + ((__ballot(!(a < 60000.0f)) ? 1u : 0u) | (__ballot(a >= 0.015625f) ? 2u : 0u)); }
            u32x4 wq;
            if constexpr (!LASTP && PH == 0 && g >= 25 && g < 29) wq = cvt8r<R_W + (g - 25), 4>(sw);
            mfma1<RC + N0 + 2 * DN, RAF, RBF + N0 + 2 * DN>();
            if constexpr (g < 8) LOAD_B((cur + 2) & 3, t + 2, g);
            if constexpr (FIN && g >= 8 && g < 16) LOAD_B(3, t + 3, g - 8);
            if constexpr (FIN && g >= 26 && g < 30) gload16_sys<R_W + 4 * (g - 26)>(lane * 16 + (g - 26) * 1024, fA);
            if constexpr (FIN && g == 16) gload8_agent<R_PB>(lane * 8, fB);
            if constexpr (!LASTP && PH == 6 && g >= 8 && g < 16) gload16<R_W + 4 * (g - 8), 0>(wvoff, wbase + ((size_t)min(slab + 1, 7) * 512 + (g - 8)) * GN * 4);
            if constexpr (!LASTP && PH == 2 && g == 9) gstore4_agent(0u, wfl, fbdst + slab * 64);
            if constexpr (g >= 17 && g < 25) GLDS_A(t + 3, (PH + 3) & 3, g - 17);
            if constexpr (!LASTP && PH == 0 && g >= 25 && g < 29) gstore16_agent<(g - 25) * 1024>(voff0, wq, bdst + (size_t)slab * 8 * TB);
            if constexpr ((PH & 1) == 1 && g == 29) { pollA(NT - 1); plim = NT; }
            if constexpr ((PH & 1) == 1 && g == 30) pollB(NT - 1);
            mfma1<RC + N0 + 3 * DN, RAF, RBF + N0 + 3 * DN>();
        });
    };
    for (int s = 0;; ++s) {
        sfor<4>([&](auto PH) __attribute__((always_inline)) { tile(PH, std::false_type{}, s * 8 + PH); });
        if (__builtin_expect(s == 7, 0)) break;
        sfor<4>([&](auto PH) __attribute__((always_inline)) { tile(std::integral_constant<int, PH + 4>{}, std::false_type{}, s * 8 + 4 + PH); });
    }
    tile(std::integral_constant<int, 4>{}, std::true_type{}, 60);
    if (pst == 0) push_all();
    wait_vm<0>();
    if (pst != 5) push_flag();
    __builtin_amdgcn_s_barrier();
    STAMP(8);
    bool wbad;
    { const unsigned o = rdreg<R_PB>() | rdreg<R_PB + 1>(); wbad = __ballot(o & 1u) != 0ull || __ballot(o & 2u) == 0ull;
      unsigned xo = 0; sfor<8>([&](auto I) __attribute__((always_inline)) { xo |= rdreg<R_W + 2 * I + 1>(); }); wbad = wbad || __ballot(xo & 0x10000u) != 0ull; }
    float fx = exp2i(-exa), fw = exp2i(-ew);
    bool bad = false;
    float* cp = p.out + (size_t)(pm * 256 + fq * 4) * GN + pn * 256 + wid * 64 + fr * 4;
    auto store_rows = [&](auto M_) __attribute__((always_inline)) { constexpr int m = M_;
        sfor<4>([&](auto J) __attribute__((always_inline)) { constexpr int j = J;
            f32x4 v = acc_read_row<m * 16 + j>();
#pragma unroll
            for (int e = 0; e < 4; ++e) { float y = v[e]; y = y < 0.f ? 0.f : y; v[e] = (y * fx) * fw; }
            __builtin_nontemporal_store(v, (f32x4*)(cp + (size_t)(m * 16 + j) * GN));
        });
    };
    if (!wbad) {
        bad = dead || load_word(tmo) == call;
        if (bad) fw = __builtin_nanf("");
        sfor<APF>([&](auto I) __attribute__((always_inline)) { constexpr int i = I, m = i / 6, q = i % 6; ldsrd<R_AF + 4 * (i & 7), (1 + (q >> 1)) * TB + ((q & 1) * 16 + m) * 1024>(ldsb, ldsh); });
        sfor<96>([&](auto I) __attribute__((always_inline)) { constexpr int i = I, m = i / 6, q = i % 6, i2 = i + APF, m2 = i2 / 6, q2 = i2 % 6;
            if constexpr (i2 < 96) ldsrd<R_AF + 4 * (i2 & 7), (1 + (q2 >> 1)) * TB + ((q2 & 1) * 16 + m2) * 1024>(ldsb, ldsh);
            mfma_group<m * 16, (i2 < 96) ? APF : (95 - i), R_AF + 4 * (i & 7), R_B + (1 + (q >> 1)) * 32 + (q & 1) * 16>();
            if constexpr (q == 1 && m > 0) store_rows(std::integral_constant<int, m - 1>{});
        });
        asm volatile("s_nop 15\n\ts_nop 15" ::: "memory");
        store_rows(std::integral_constant<int, 15>{});
    } else {
        bad = dead || load_word(tmo) == call;
        asm volatile("s_nop 15\n\ts_nop 15" ::: "memory");
        acc_zero_all();
        const _Float16* ax = (const _Float16*)(wsl + abuf + (size_t)pm * PANEL_B);
        const float* x32 = (const float*)(wsl + WS_X32 + (size_t)(call & 1u) * 2 * OPER_B) + (size_t)(pm * 256 + fr) * GK;
        const unsigned* fl = (const unsigned*)fA;
        const float* wq = p.w + pn * 256 + wid * 64 + fr * 4;
        for (int k0 = 0; k0 < GK; k0 += 4) {
            const int k = k0 + fq;
            const f32x4 b = *(const f32x4*)(wq + (size_t)k * GN);
            const _Float16* ap = ax + (size_t)(k >> 5) * 8192 + (size_t)(((k >> 3) & 3) * 16 + fr) * 8 + (k & 7);
            sfor<4>([&](auto Q_) __attribute__((always_inline)) { constexpr int q = Q_;
                const bool f32src = (__hip_atomic_load(fl + ((k0 >> 5) * 4 + q) * 2 + 1, __ATOMIC_RELAXED, __HIP_MEMORY_SCOPE_SYSTEM) & 0x10000u) != 0u;
                sfor<4>([&](auto M_) __attribute__((always_inline)) { constexpr int m = q * 4 + M_;
                    const float a = f32src ? __hip_atomic_load(x32 + (size_t)m * 16 * GK + k, __ATOMIC_RELAXED, __HIP_MEMORY_SCOPE_SYSTEM) : (float)ap[m * 512] * fx;
                    mfma_f32<m * 16>(a, b[0]); mfma_f32<m * 16 + 4>(a, b[1]); mfma_f32<m * 16 + 8>(a, b[2]); mfma_f32<m * 16 + 12>(a, b[3]);
                });
            });
        }
        fx = 1.0f; fw = bad ? __builtin_nanf("") : 1.0f;
        asm volatile("s_nop 15\n\ts_nop 15" ::: "memory");
        sfor<16>([&](auto M_) __attribute__((always_inline)) { store_rows(M_); });
    }
    asm volatile(".Lagk_end:" ::: "memory");
    STAMP(9);
}

}

extern "C" void kernel_launch(int rank, int nranks, ncclComm_t comm,
                              hipStream_t stream,
                              void* const* d_in, const int* in_sizes, int n_in,
                              void* d_out, int out_size,
                              const int* in_dtypes, int out_dtype,
                              void* const* d_out_all,
                              void* const* d_ws_all, size_t ws_bytes) {
    if (nranks != WORLD || rank < 0 || rank >= WORLD || ws_bytes < WS_END || n_in < 2) { fprintf(stderr, "unsupported configuration\n"); return; }
    static bool init[WORLD] = {};
    if (!init[rank]) {
        HIP_CHECK(hipFuncSetAttribute((const void*)fused_kernel, hipFuncAttributeMaxDynamicSharedMemorySize, LDS_BYTES));
        init[rank] = true;
    }
    Params p;
    p.x = (const float*)d_in[0]; p.w = (const float*)d_in[1]; p.out = (float*)d_out; p.rank = rank;
    for (int i = 0; i < WORLD; ++i) p.ws[i] = (char*)d_ws_all[(rank + i) & 7];
    hipLaunchKernelGGL(fused_kernel, dim3(GRID), dim3(NTHREADS), LDS_BYTES, stream, p);
    HIP_CHECK(hipGetLastError());
}
```
